```python
import numpy as np

kernel_name = "gpu_a2a_gemm_m8192_k8192_n8192_f32_silu"


ABSMAX_RTOL = 0.001

COMPUTE_DTYPE = "float32"
EPILOGUE = "silu"
IN_QUANT = None
OUT_QUANT = None
IN_DTYPES = {"x": "float32", "w_mat": "float32"}
WORLD_SIZE = 8

def setup_inputs():

    x = np.random.randn(8192, 8192).astype(np.float32)
    w_mat = (np.random.randn(8192, 8192) / np.sqrt(8192)).astype(np.float32)
    return {"x": x, "w_mat": w_mat}

def _epilogue(y):
    return (y / (1.0 + np.exp(-np.clip(y, -60.0, 60.0)))).astype(np.float32)

def reference(x, w_mat):
    y = (x @ w_mat).astype(np.float32)
    return _epilogue(y)
```

```cpp
#include <hip/hip_runtime.h>
#include <rccl/rccl.h>
#include <cstdio>

typedef _Float16 h2 __attribute__((ext_vector_type(2)));
typedef _Float16 h8 __attribute__((ext_vector_type(8)));
typedef float f2 __attribute__((ext_vector_type(2)));
typedef float f4 __attribute__((ext_vector_type(4)));
typedef unsigned u4 __attribute__((ext_vector_type(4)));

constexpr int NR = 8;
constexpr int MLOC = 1024, KTOT = 8192, NTOT = 8192, KLOC = 1024;
constexpr size_t STAGE_BYTES = (size_t)MLOC * KTOT * 2;
constexpr size_t CTRL_OFF = 2 * STAGE_BYTES;
constexpr unsigned long long DEADLINE_TICKS = 50000000ull;

struct Params {
    const float* x;
    const float* w;
    float* out;
    char* ws[NR];
    int rank;
    int phase;
};

__device__ __forceinline__ unsigned cvtpk(float a, float b) {
    f2 v = {a, b};
    h2 h = __builtin_convertvector(v, h2);
    return __builtin_bit_cast(unsigned, h);
}

__global__ __launch_bounds__(256, 1) void fused_kernel(Params p) {
    __shared__ __attribute__((aligned(16))) char lds[16384 + 64];
    const int tid = threadIdx.x, lane = tid & 63, wave = tid >> 6;
    const int q = lane >> 4, c = lane & 15;
    char* myws = p.ws[p.rank];
    unsigned* ctrl = (unsigned*)(myws + CTRL_OFF);
    unsigned* s_call = (unsigned*)(lds + 16384);
    if (tid == 0) {
        unsigned t = __hip_atomic_fetch_add(ctrl + (4096 + 128 * p.phase) / 4, 1u, __ATOMIC_RELAXED, __HIP_MEMORY_SCOPE_AGENT);
        *s_call = t / gridDim.x + 1;
    }
    __syncthreads();
    const unsigned call = *s_call;
    const size_t stage_off = (size_t)(call & 1) * STAGE_BYTES;
    unsigned* tmo = ctrl + (4096 + 256) / 4;

    if (p.phase == 0) {
        const int dest = blockIdx.x & 7, part = blockIdx.x >> 3;
        char* dws = p.ws[dest] + stage_off;
        for (int i = 0; i < 16; ++i) {
            int f = part * 64 + wave * 16 + i;
            int kbl = f >> 6, mb = f & 63;
            const float* src = p.x + (size_t)(dest * MLOC + mb * 16 + c) * KLOC + kbl * 32 + q * 8;
            f4 a = *(const f4*)src, b = *(const f4*)(src + 4);
            u4 o = {cvtpk(a[0], a[1]), cvtpk(a[2], a[3]), cvtpk(b[0], b[1]), cvtpk(b[2], b[3])};
            int kb = p.rank * 32 + kbl;
            *(u4*)(dws + ((size_t)kb * 64 + mb) * 1024 + lane * 16) = o;
        }
        asm volatile("s_waitcnt vmcnt(0)" ::: "memory");
        __syncthreads();
        if (tid == 0) {
            __builtin_amdgcn_fence(__ATOMIC_RELEASE, "");
            asm volatile("s_waitcnt vmcnt(0)" ::: "memory");
            unsigned* flag = (unsigned*)(p.ws[dest] + CTRL_OFF) + p.rank * 32;
            __hip_atomic_fetch_add(flag, 1u, __ATOMIC_RELAXED, __HIP_MEMORY_SCOPE_SYSTEM);
        }
        return;
    }

    if (tid < NR) {
        unsigned* flag = ctrl + tid * 32;
        unsigned long long t0 = __builtin_amdgcn_s_memrealtime();
        while (__hip_atomic_load(flag, __ATOMIC_RELAXED, __HIP_MEMORY_SCOPE_SYSTEM) < call * 32u) {
            if (__hip_atomic_load(tmo, __ATOMIC_RELAXED, __HIP_MEMORY_SCOPE_AGENT) == call ||
                __builtin_amdgcn_s_memrealtime() - t0 > DEADLINE_TICKS) {
                __hip_atomic_store(tmo, call, __ATOMIC_RELAXED, __HIP_MEMORY_SCOPE_AGENT);
                break;
            }
            __builtin_amdgcn_s_sleep(2);
        }
        __builtin_amdgcn_fence(__ATOMIC_ACQUIRE, "");
        asm volatile("s_waitcnt vmcnt(0)" ::: "memory");
    }
    __syncthreads();
    const bool bad = __hip_atomic_load(tmo, __ATOMIC_RELAXED, __HIP_MEMORY_SCOPE_AGENT) == call;

    const int xcd = blockIdx.x & 7, j = blockIdx.x >> 3;
    const int nt = j & 7, mt = j >> 3;
    const int n0 = xcd * 1024 + nt * 128, m0 = mt * 256 + wave * 64;
    const char* As = myws + stage_off;
    f4 acc[4][8];
    for (int m = 0; m < 4; ++m)
        for (int n = 0; n < 8; ++n) acc[m][n] = f4{0.f, 0.f, 0.f, 0.f};
    const int g = wave & 1, sp = wave >> 1;
    for (int t = 0; t < KTOT / 64; ++t) {
        const float* wp = p.w + (size_t)(t * 64 + sp * 32 + q * 8) * NTOT + n0 + g * 64 + c * 4;
        f4 wv[8];
#pragma unroll
        for (int jj = 0; jj < 8; ++jj) wv[jj] = *(const f4*)(wp + (size_t)jj * NTOT);
        __syncthreads();
#pragma unroll
        for (int i = 0; i < 4; ++i) {
            u4 o = {cvtpk(wv[0][i], wv[1][i]), cvtpk(wv[2][i], wv[3][i]), cvtpk(wv[4][i], wv[5][i]), cvtpk(wv[6][i], wv[7][i])};
            *(u4*)(lds + ((sp * 8 + g * 4 + i) * 1024) + lane * 16) = o;
        }
        __syncthreads();
#pragma unroll
        for (int u = 0; u < 2; ++u) {
            h8 af[4], bf[8];
            int kb = t * 2 + u;
#pragma unroll
            for (int m = 0; m < 4; ++m) af[m] = *(const h8*)(As + ((size_t)kb * 64 + (m0 >> 4) + m) * 1024 + lane * 16);
#pragma unroll
            for (int n = 0; n < 8; ++n) bf[n] = *(const h8*)(lds + (u * 8 + n) * 1024 + lane * 16);
#pragma unroll
            for (int m = 0; m < 4; ++m)
#pragma unroll
                for (int n = 0; n < 8; ++n)
                    acc[m][n] = __builtin_amdgcn_mfma_f32_16x16x32_f16(af[m], bf[n], acc[m][n], 0, 0, 0);
        }
    }
    const float nanv = __builtin_nanf("");
#pragma unroll
    for (int m = 0; m < 4; ++m)
#pragma unroll
        for (int gg = 0; gg < 2; ++gg)
#pragma unroll
            for (int i = 0; i < 4; ++i) {
                f4 o;
#pragma unroll
                for (int ii = 0; ii < 4; ++ii) {
                    float y = acc[m][gg * 4 + ii][i];
                    float yc = fminf(fmaxf(y, -60.f), 60.f);
                    o[ii] = bad ? nanv : y / (1.f + __expf(-yc));
                }
                *(f4*)(p.out + (size_t)(m0 + m * 16 + q * 4 + i) * NTOT + n0 + gg * 64 + c * 4) = o;
            }
}

extern "C" void kernel_launch(int rank, int nranks, ncclComm_t comm, hipStream_t stream, void* const* d_in,
                              const int* in_sizes, int n_in, void* d_out, int out_size, const int* in_dtypes,
                              int out_dtype, void* const* d_out_all, void* const* d_ws_all, size_t ws_bytes) {
    if (nranks != NR || ws_bytes < CTRL_OFF + 65536) {
        fprintf(stderr, "unexpected config\n");
        return;
    }
    Params p;
    p.x = (const float*)d_in[0];
    p.w = (const float*)d_in[1];
    p.out = (float*)d_out;
    for (int i = 0; i < NR; ++i) p.ws[i] = (char*)d_ws_all[i];
    p.rank = rank;
    p.phase = 0;
    fused_kernel<<<256, 256, 0, stream>>>(p);
    p.phase = 1;
    fused_kernel<<<256, 256, 0, stream>>>(p);
}
```

```cpp
#include <hip/hip_runtime.h>
#include <rccl/rccl.h>
#include <cstdio>

#define FUSED_ASM \
  "s_lshl_b32 s72, %7, 7\n" \
  "s_add_u32 s72, s72, 67371008\n" \
  "s_load_dword s82, %6, s72 glc\n" \
  "v_readlane_b32 s76, %0, 62\n" \
  "v_readlane_b32 s77, %0, 63\n" \
  "s_mov_b32 s89, 0\n" \
  "s_mov_b32 s90, 67108864\n" \
  "s_cmp_eq_u32 %8, 0\n" \
  "s_cselect_b32 s91, -1, 0\n" \
  "s_mov_b64 s[32:33], %2\n" \
  "s_mov_b32 s34, 0x2000000\n" \
  "s_mov_b32 s35, 0x20000\n" \
  "v_mbcnt_lo_u32_b32 v224, -1, 0\n" \
  "v_mbcnt_hi_u32_b32 v224, -1, v224\n" \
  "v_lshlrev_b32 v248, 4, v224\n" \
  "v_lshrrev_b32 v225, 4, v224\n" \
  "v_and_b32 v226, 15, v224\n" \
  "v_lshlrev_b32 v250, 12, v226\n" \
  "v_lshl_add_u32 v250, v225, 5, v250\n" \
  "s_lshl_b32 s72, %8, 7\n" \
  "v_add_u32 v250, s72, v250\n" \
  "s_and_b32 s72, s76, 7\n" \
  "s_lshl_b32 s72, s72, 22\n" \
  "s_bfe_u32 s74, s76, 0x60004\n" \
  "s_lshl_b32 s74, s74, 16\n" \
  "s_or_b32 s72, s72, s74\n" \
  "s_bfe_u32 s74, s76, 0x3000a\n" \
  "s_lshl_b32 s74, s74, 9\n" \
  "s_or_b32 s72, s72, s74\n" \
  "s_and_b32 s73, s77, 7\n" \
  "s_lshl_b32 s73, s73, 22\n" \
  "s_bfe_u32 s74, s77, 0x60004\n" \
  "s_lshl_b32 s74, s74, 16\n" \
  "s_or_b32 s73, s73, s74\n" \
  "s_bfe_u32 s74, s77, 0x3000a\n" \
  "s_lshl_b32 s74, s74, 9\n" \
  "s_or_b32 s73, s73, s74\n" \
  "buffer_load_dwordx4 v[0:3], v250, s[32:35], s72 offen nt\n" \
  "buffer_load_dwordx4 v[4:7], v250, s[32:35], s72 offen offset:16 nt\n" \
  "buffer_load_dwordx4 v[8:11], v250, s[32:35], s73 offen nt\n" \
  "buffer_load_dwordx4 v[12:15], v250, s[32:35], s73 offen offset:16 nt\n" \
  "s_mov_b64 s[36:37], %6\n" \
  "s_mov_b32 s38, 67502080\n" \
  "s_mov_b32 s39, 0x20000\n" \
  "s_mov_b32 s42, 67502080\n" \
  "s_mov_b32 s43, 0x20000\n" \
  "s_mov_b64 s[28:29], %3\n" \
  "s_mov_b32 s30, 0x10000000\n" \
  "s_mov_b32 s31, 0x20000\n" \
  "v_lshlrev_b32 v227, 4, v226\n" \
  "s_and_b32 s75, %8, 1\n" \
  "s_and_b32 s72, %7, 7\n" \
  "s_lshl_b32 s72, s72, 12\n" \
  "s_bfe_u32 s73, %7, 0x20004\n" \
  "s_lshl_b32 s73, s73, 10\n" \
  "s_add_u32 s72, s72, s73\n" \
  "s_bfe_u32 s73, %7, 0x10003\n" \
  "s_lshl_b32 s73, s73, 8\n" \
  "s_add_u32 s72, s72, s73\n" \
  "s_lshl_b32 s73, s75, 9\n" \
  "s_add_u32 s72, s72, s73\n" \
  "v_add_u32 v249, s72, v227\n" \
  "v_lshl_add_u32 v249, v225, 18, v249\n" \
  "s_bfe_u32 s73, %7, 0x30003\n" \
  "s_bfe_u32 s74, %7, 0x10006\n" \
  "s_lshl_b32 s75, s74, 1\n" \
  "s_add_u32 s75, s75, s73\n" \
  "s_add_u32 s72, s75, 0\n" \
  "s_and_b32 s72, s72, 3\n" \
  "s_lshl_b32 s61, s72, 16\n" \
  "s_add_u32 s72, s75, 1\n" \
  "s_and_b32 s72, s72, 3\n" \
  "s_lshl_b32 s62, s72, 16\n" \
  "s_add_u32 s72, s75, 2\n" \
  "s_and_b32 s72, s72, 3\n" \
  "s_lshl_b32 s63, s72, 16\n" \
  "s_add_u32 s72, s75, 3\n" \
  "s_and_b32 s72, s72, 3\n" \
  "s_lshl_b32 s64, s72, 16\n" \
  "s_lshr_b32 s72, %8, 1\n" \
  "s_add_u32 s75, s75, s72\n" \
  "s_add_u32 s72, s75, 0\n" \
  "s_and_b32 s72, s72, 3\n" \
  "s_lshl_b32 s65, s72, 20\n" \
  "s_add_u32 s72, s75, 2\n" \
  "s_and_b32 s72, s72, 3\n" \
  "s_lshl_b32 s66, s72, 20\n" \
  "v_readlane_b32 s79, %1, 0\n" \
  "v_readlane_b32 s68, %1, 1\n" \
  "s_nop 1\n" \
  "s_and_b32 s79, s79, 63\n" \
  "s_and_b32 s68, s68, 63\n" \
  "s_lshl_b32 s80, s79, 22\n" \
  "s_lshl_b32 s69, s68, 22\n" \
  "s_lshl_b32 s72, %8, 12\n" \
  "s_add_u32 s72, s72, %9\n" \
  "v_add_u32 v252, s72, v248\n" \
  "v_add_u32 v251, %9, v248\n" \
  "s_mov_b32 s81, 0\n" \
  "s_lshr_b32 s83, %7, 7\n" \
  "s_lshl_b32 s83, s83, 12\n" \
  "s_xor_b32 s84, s83, 0x1000\n" \
  "s_waitcnt lgkmcnt(0)\n" \
  "s_add_u32 s82, s82, 1\n" \
  "s_and_b32 s85, s82, 1\n" \
  "s_mul_i32 s72, s85, 7168\n" \
  "s_mul_i32 s73, s85, 458759\n" \
  "s_mov_b32 exec_hi, 0x3fffffff\n" \
  "v_xor_b32 %0, s72, %0\n" \
  "s_mov_b32 exec_hi, -1\n" \
  "s_mov_b32 exec_lo, 4294967280\n" \
  "v_xor_b32 %1, s73, %1\n" \
  "s_mov_b32 exec_lo, -1\n" \
  "s_lshl_b32 s85, s85, 24\n" \
  "s_nop 0\n" \
  "v_readlane_b32 s88, %0, 0\n" \
  "v_readlane_b32 s58, %0, 1\n" \
  "s_nop 1\n" \
  "s_and_b32 s72, s88, 7\n" \
  "s_lshl_b32 s72, s72, 3\n" \
  "s_load_dwordx2 s[40:41], %5, s72\n" \
  "s_lshl_b32 s72, s74, 2\n" \
  "s_add_u32 s72, s72, %8\n" \
  "s_lshl_b32 s71, s72, 5\n" \
  "s_add_u32 s71, s71, 67108864\n" \
  "s_lshl_b32 s72, s72, 13\n" \
  "s_sub_u32 s26, 16777216, s72\n" \
  "s_add_u32 s72, s72, s85\n" \
  "s_mov_b64 s[24:25], %6\n" \
  "s_add_u32 s24, s24, s72\n" \
  "s_addc_u32 s25, s25, 0\n" \
  "s_mov_b32 s27, 0x20000\n" \
  "s_bfe_u32 s72, s76, 0x6000a\n" \
  "s_lshl_b32 s72, s72, 18\n" \
  "s_lshl_b32 s74, %8, 16\n" \
  "s_or_b32 s72, s72, s74\n" \
  "s_bfe_u32 s74, s76, 0x60004\n" \
  "s_lshl_b32 s74, s74, 10\n" \
  "s_or_b32 s72, s72, s74\n" \
  "s_add_u32 s72, s72, s85\n" \
  "s_bfe_u32 s73, s77, 0x6000a\n" \
  "s_lshl_b32 s73, s73, 18\n" \
  "s_lshl_b32 s74, %8, 16\n" \
  "s_or_b32 s73, s73, s74\n" \
  "s_bfe_u32 s74, s77, 0x60004\n" \
  "s_lshl_b32 s74, s74, 10\n" \
  "s_or_b32 s73, s73, s74\n" \
  "s_add_u32 s73, s73, s85\n" \
  "s_waitcnt vmcnt(2)\n" \
  "v_cvt_pk_f16_f32 v0, v0, v1\n" \
  "v_cvt_pk_f16_f32 v1, v2, v3\n" \
  "v_cvt_pk_f16_f32 v2, v4, v5\n" \
  "v_cvt_pk_f16_f32 v3, v6, v7\n" \
  "buffer_store_dwordx4 v[0:3], v248, s[36:39], s72 offen sc0 sc1\n" \
  "s_waitcnt vmcnt(1)\n" \
  "v_cvt_pk_f16_f32 v8, v8, v9\n" \
  "v_cvt_pk_f16_f32 v9, v10, v11\n" \
  "v_cvt_pk_f16_f32 v10, v12, v13\n" \
  "v_cvt_pk_f16_f32 v11, v14, v15\n" \
  "buffer_store_dwordx4 v[8:11], v248, s[36:39], s73 offen sc0 sc1\n" \
  "s_add_u32 s78, s80, s65\n" \
  "s_add_u32 s72, s78, 0\n" \
  "buffer_load_dwordx4 v[128:131], v249, s[28:31], s72 offen\n" \
  "s_add_u32 s73, s78, 32768\n" \
  "buffer_load_dwordx4 v[132:135], v249, s[28:31], s73 offen\n" \
  "s_add_u32 s72, s78, 65536\n" \
  "buffer_load_dwordx4 v[136:139], v249, s[28:31], s72 offen\n" \
  "s_add_u32 s73, s78, 98304\n" \
  "buffer_load_dwordx4 v[140:143], v249, s[28:31], s73 offen\n" \
  "s_add_u32 s72, s78, 131072\n" \
  "buffer_load_dwordx4 v[144:147], v249, s[28:31], s72 offen\n" \
  "s_add_u32 s73, s78, 163840\n" \
  "buffer_load_dwordx4 v[148:151], v249, s[28:31], s73 offen\n" \
  "s_add_u32 s72, s78, 196608\n" \
  "buffer_load_dwordx4 v[152:155], v249, s[28:31], s72 offen\n" \
  "s_add_u32 s73, s78, 229376\n" \
  "buffer_load_dwordx4 v[156:159], v249, s[28:31], s73 offen\n" \
  "s_bfe_u32 s44, s76, 0xc0004\n" \
  "s_lshl_b32 s44, s44, 2\n" \
  "s_add_u32 s44, s44, 67108864\n" \
  "s_bfe_u32 s45, s77, 0xc0004\n" \
  "s_lshl_b32 s45, s45, 2\n" \
  "s_add_u32 s45, s45, 67108864\n" \
  "s_and_b32 s86, s88, 7\n" \
  "s_lshl_b32 s86, s86, 22\n" \
  "s_bfe_u32 s74, s88, 0x60004\n" \
  "s_lshl_b32 s74, s74, 16\n" \
  "s_or_b32 s86, s86, s74\n" \
  "s_bfe_u32 s74, s88, 0x3000a\n" \
  "s_lshl_b32 s74, s74, 9\n" \
  "s_or_b32 s86, s86, s74\n" \
  "v_mov_b32 v224, s82\n" \
  "s_waitcnt vmcnt(8)\n" \
  "s_barrier\n" \
  "s_cmp_lg_u32 %8, 0\n" \
  "s_cbranch_scc1 W06_%=\n" \
  "s_mov_b64 exec, 1\n" \
  "buffer_store_dword v224, off, s[36:39], s44 sc0 sc1\n" \
  "buffer_store_dword v224, off, s[36:39], s45 sc0 sc1\n" \
  "s_mov_b64 exec, -1\n" \
  "W06_%=:\n" \
  "buffer_load_dwordx4 v[240:243], v250, s[32:35], s86 offen nt\n" \
  "buffer_load_dwordx4 v[244:247], v250, s[32:35], s86 offen offset:16 nt\n" \
  "s_add_u32 s78, s80, s66\n" \
  "s_add_u32 s72, s78, 0\n" \
  "buffer_load_dwordx4 v[160:163], v249, s[28:31], s72 offen\n" \
  "s_add_u32 s73, s78, 32768\n" \
  "buffer_load_dwordx4 v[164:167], v249, s[28:31], s73 offen\n" \
  "s_add_u32 s72, s78, 65536\n" \
  "buffer_load_dwordx4 v[168:171], v249, s[28:31], s72 offen\n" \
  "s_add_u32 s73, s78, 98304\n" \
  "buffer_load_dwordx4 v[172:175], v249, s[28:31], s73 offen\n" \
  "s_add_u32 s72, s78, 131072\n" \
  "buffer_load_dwordx4 v[176:179], v249, s[28:31], s72 offen\n" \
  "s_add_u32 s73, s78, 163840\n" \
  "buffer_load_dwordx4 v[180:183], v249, s[28:31], s73 offen\n" \
  "s_add_u32 s72, s78, 196608\n" \
  "buffer_load_dwordx4 v[184:187], v249, s[28:31], s72 offen\n" \
  "s_add_u32 s73, s78, 229376\n" \
  "buffer_load_dwordx4 v[188:191], v249, s[28:31], s73 offen\n" \
  "s_and_b32 s86, s58, 7\n" \
  "s_lshl_b32 s86, s86, 22\n" \
  "s_bfe_u32 s74, s58, 0x60004\n" \
  "s_lshl_b32 s74, s74, 16\n" \
  "s_or_b32 s86, s86, s74\n" \
  "s_bfe_u32 s74, s58, 0x3000a\n" \
  "s_lshl_b32 s74, s74, 9\n" \
  "s_or_b32 s86, s86, s74\n" \
  "s_bfe_u32 s87, s88, 0x6000a\n" \
  "s_lshl_b32 s87, s87, 18\n" \
  "s_lshl_b32 s74, %8, 16\n" \
  "s_or_b32 s87, s87, s74\n" \
  "s_bfe_u32 s74, s88, 0x60004\n" \
  "s_lshl_b32 s74, s74, 10\n" \
  "s_or_b32 s87, s87, s74\n" \
  "s_add_u32 s87, s87, s85\n" \
  "s_bfe_u32 s88, s88, 0xc0004\n" \
  "s_lshl_b32 s88, s88, 2\n" \
  "s_add_u32 s88, s88, 67108864\n" \
  "s_waitcnt vmcnt(10)\n" \
  "v_cvt_pk_f16_f32 v192, v128, v132\n" \
  "v_cvt_pk_f16_f32 v193, v136, v140\n" \
  "v_cvt_pk_f16_f32 v194, v144, v148\n" \
  "v_cvt_pk_f16_f32 v195, v152, v156\n" \
  "ds_write_b128 v252, v[192:195] offset:0\n" \
  "v_cvt_pk_f16_f32 v196, v129, v133\n" \
  "v_cvt_pk_f16_f32 v197, v137, v141\n" \
  "v_cvt_pk_f16_f32 v198, v145, v149\n" \
  "v_cvt_pk_f16_f32 v199, v153, v157\n" \
  "ds_write_b128 v252, v[196:199] offset:1024\n" \
  "v_cvt_pk_f16_f32 v200, v130, v134\n" \
  "v_cvt_pk_f16_f32 v201, v138, v142\n" \
  "v_cvt_pk_f16_f32 v202, v146, v150\n" \
  "v_cvt_pk_f16_f32 v203, v154, v158\n" \
  "ds_write_b128 v252, v[200:203] offset:2048\n" \
  "v_cvt_pk_f16_f32 v204, v131, v135\n" \
  "v_cvt_pk_f16_f32 v205, v139, v143\n" \
  "v_cvt_pk_f16_f32 v206, v147, v151\n" \
  "v_cvt_pk_f16_f32 v207, v155, v159\n" \
  "ds_write_b128 v252, v[204:207] offset:3072\n" \
  "v_add_u32 v252, 16384, v252\n" \
  "s_waitcnt lgkmcnt(0)\n" \
  "s_barrier\n" \
  "ds_read_b128 v[192:195], v251 offset:0\n" \
  "ds_read_b128 v[196:199], v251 offset:1024\n" \
  "ds_read_b128 v[200:203], v251 offset:2048\n" \
  "ds_read_b128 v[204:207], v251 offset:3072\n" \
  "s_lshl_b32 s74, s79, 8\n" \
  "s_add_u32 s74, s74, s71\n" \
  "s_branch SLOW8_%=\n" \
  "OK7_%=:\n" \
  "s_lshl_b32 s75, s79, 18\n" \
  "s_add_u32 s79, s75, s61\n" \
  "s_add_u32 s80, s79, s84\n" \
  "s_add_u32 s79, s79, s83\n" \
  "buffer_load_dwordx4 a[128:131], v248, s[24:27], s79 offen offset:0 sc1\n" \
  "buffer_load_dwordx4 a[132:135], v248, s[24:27], s79 offen offset:1024 sc1\n" \
  "buffer_load_dwordx4 a[136:139], v248, s[24:27], s79 offen offset:2048 sc1\n" \
  "buffer_load_dwordx4 a[140:143], v248, s[24:27], s79 offen offset:3072 sc1\n" \
  "buffer_load_dwordx4 a[144:147], v248, s[24:27], s80 offen offset:0 sc1\n" \
  "buffer_load_dwordx4 a[148:151], v248, s[24:27], s80 offen offset:1024 sc1\n" \
  "buffer_load_dwordx4 a[152:155], v248, s[24:27], s80 offen offset:2048 sc1\n" \
  "buffer_load_dwordx4 a[156:159], v248, s[24:27], s80 offen offset:3072 sc1\n" \
  "s_add_u32 s79, s75, s62\n" \
  "s_add_u32 s80, s79, s84\n" \
  "s_add_u32 s79, s79, s83\n" \
  "buffer_load_dwordx4 a[160:163], v248, s[24:27], s79 offen offset:0 sc1\n" \
  "buffer_load_dwordx4 a[164:167], v248, s[24:27], s79 offen offset:1024 sc1\n" \
  "buffer_load_dwordx4 a[168:171], v248, s[24:27], s79 offen offset:2048 sc1\n" \
  "buffer_load_dwordx4 a[172:175], v248, s[24:27], s79 offen offset:3072 sc1\n" \
  "buffer_load_dwordx4 a[176:179], v248, s[24:27], s80 offen offset:0 sc1\n" \
  "buffer_load_dwordx4 a[180:183], v248, s[24:27], s80 offen offset:1024 sc1\n" \
  "buffer_load_dwordx4 a[184:187], v248, s[24:27], s80 offen offset:2048 sc1\n" \
  "buffer_load_dwordx4 a[188:191], v248, s[24:27], s80 offen offset:3072 sc1\n" \
  "s_add_u32 s79, s75, s63\n" \
  "s_add_u32 s80, s79, s84\n" \
  "s_add_u32 s79, s79, s83\n" \
  "buffer_load_dwordx4 a[192:195], v248, s[24:27], s79 offen offset:0 sc1\n" \
  "buffer_load_dwordx4 a[196:199], v248, s[24:27], s79 offen offset:1024 sc1\n" \
  "buffer_load_dwordx4 a[200:203], v248, s[24:27], s79 offen offset:2048 sc1\n" \
  "buffer_load_dwordx4 a[204:207], v248, s[24:27], s79 offen offset:3072 sc1\n" \
  "buffer_load_dwordx4 a[208:211], v248, s[24:27], s80 offen offset:0 sc1\n" \
  "buffer_load_dwordx4 a[212:215], v248, s[24:27], s80 offen offset:1024 sc1\n" \
  "buffer_load_dwordx4 a[216:219], v248, s[24:27], s80 offen offset:2048 sc1\n" \
  "buffer_load_dwordx4 a[220:223], v248, s[24:27], s80 offen offset:3072 sc1\n" \
  "s_add_u32 s78, s69, s65\n" \
  "s_add_u32 s72, s78, 0\n" \
  "buffer_load_dwordx4 v[128:131], v249, s[28:31], s72 offen\n" \
  "s_add_u32 s73, s78, 32768\n" \
  "buffer_load_dwordx4 v[132:135], v249, s[28:31], s73 offen\n" \
  "s_add_u32 s72, s78, 65536\n" \
  "buffer_load_dwordx4 v[136:139], v249, s[28:31], s72 offen\n" \
  "s_add_u32 s73, s78, 98304\n" \
  "buffer_load_dwordx4 v[140:143], v249, s[28:31], s73 offen\n" \
  "s_add_u32 s72, s78, 131072\n" \
  "buffer_load_dwordx4 v[144:147], v249, s[28:31], s72 offen\n" \
  "s_add_u32 s73, s78, 163840\n" \
  "buffer_load_dwordx4 v[148:151], v249, s[28:31], s73 offen\n" \
  "s_add_u32 s72, s78, 196608\n" \
  "buffer_load_dwordx4 v[152:155], v249, s[28:31], s72 offen\n" \
  "s_add_u32 s73, s78, 229376\n" \
  "buffer_load_dwordx4 v[156:159], v249, s[28:31], s73 offen\n" \
  "s_add_u32 s79, s75, s64\n" \
  "s_add_u32 s80, s79, s84\n" \
  "s_add_u32 s79, s79, s83\n" \
  "buffer_load_dwordx4 a[224:227], v248, s[24:27], s79 offen offset:0 sc1\n" \
  "buffer_load_dwordx4 a[228:231], v248, s[24:27], s79 offen offset:1024 sc1\n" \
  "buffer_load_dwordx4 a[232:235], v248, s[24:27], s79 offen offset:2048 sc1\n" \
  "buffer_load_dwordx4 a[236:239], v248, s[24:27], s79 offen offset:3072 sc1\n" \
  "buffer_load_dwordx4 a[240:243], v248, s[24:27], s80 offen offset:0 sc1\n" \
  "buffer_load_dwordx4 a[244:247], v248, s[24:27], s80 offen offset:1024 sc1\n" \
  "buffer_load_dwordx4 a[248:251], v248, s[24:27], s80 offen offset:2048 sc1\n" \
  "buffer_load_dwordx4 a[252:255], v248, s[24:27], s80 offen offset:3072 sc1\n" \
  "s_lshl_b32 s68, s68, 18\n" \
  "s_mov_b32 s59, 2147483644\n" \
  "v_bfe_u32 v232, %1, 8, 16\n" \
  "s_and_b32 exec_lo, s59, s91\n" \
  "s_cbranch_scc0 PL12_%=\n" \
  "s_mov_b32 exec_hi, 0\n" \
  "buffer_load_dword v253, v232, s[36:39], s90 offen sc1\n" \
  "PL12_%=:\n" \
  "s_mov_b64 exec, -1\n" \
  "s_waitcnt lgkmcnt(0)\n" \
  "s_mov_b32 s67, 0\n" \
  "s_nop 1\n" \
  "s_waitcnt vmcnt(39)\n" \
  "v_mfma_f32_16x16x32_f16 v[0:3], a[128:131], v[192:195], 0\n" \
  "ds_read_b128 v[208:211], v251 offset:4096\n" \
  "v_mfma_f32_16x16x32_f16 v[4:7], a[128:131], v[196:199], 0\n" \
  "v_mfma_f32_16x16x32_f16 v[8:11], a[128:131], v[200:203], 0\n" \
  "ds_read_b128 v[212:215], v251 offset:5120\n" \
  "v_mfma_f32_16x16x32_f16 v[12:15], a[128:131], v[204:207], 0\n" \
  "s_waitcnt vmcnt(38)\n" \
  "v_mfma_f32_16x16x32_f16 v[44:47], a[132:135], v[204:207], 0\n" \
  "ds_read_b128 v[216:219], v251 offset:6144\n" \
  "v_mfma_f32_16x16x32_f16 v[40:43], a[132:135], v[200:203], 0\n" \
  "v_mfma_f32_16x16x32_f16 v[36:39], a[132:135], v[196:199], 0\n" \
  "ds_read_b128 v[220:223], v251 offset:7168\n" \
  "v_mfma_f32_16x16x32_f16 v[32:35], a[132:135], v[192:195], 0\n" \
  "s_waitcnt vmcnt(37)\n" \
  "v_mfma_f32_16x16x32_f16 v[64:67], a[136:139], v[192:195], 0\n" \
  "s_waitcnt vmcnt(40)\n" \
  "v_cvt_pk_f16_f32 v224, v160, v164\n" \
  "v_mfma_f32_16x16x32_f16 v[68:71], a[136:139], v[196:199], 0\n" \
  "v_cvt_pk_f16_f32 v225, v168, v172\n" \
  "v_mfma_f32_16x16x32_f16 v[72:75], a[136:139], v[200:203], 0\n" \
  "v_cvt_pk_f16_f32 v226, v176, v180\n" \
  "v_mfma_f32_16x16x32_f16 v[76:79], a[136:139], v[204:207], 0\n" \
  "v_cvt_pk_f16_f32 v227, v184, v188\n" \
  "s_waitcnt vmcnt(36)\n" \
  "v_mfma_f32_16x16x32_f16 v[108:111], a[140:143], v[204:207], 0\n" \
  "ds_write_b128 v252, v[224:227] offset:0\n" \
  "v_mfma_f32_16x16x32_f16 v[104:107], a[140:143], v[200:203], 0\n" \
  "v_cvt_pk_f16_f32 v228, v161, v165\n" \
  "v_mfma_f32_16x16x32_f16 v[100:103], a[140:143], v[196:199], 0\n" \
  "v_cvt_pk_f16_f32 v229, v169, v173\n" \
  "v_mfma_f32_16x16x32_f16 v[96:99], a[140:143], v[192:195], 0\n" \
  "v_cvt_pk_f16_f32 v230, v177, v181\n" \
  "s_waitcnt vmcnt(35)\n" \
  "v_mfma_f32_16x16x32_f16 a[0:3], a[144:147], v[192:195], 0\n" \
  "v_cvt_pk_f16_f32 v231, v185, v189\n" \
  "v_mfma_f32_16x16x32_f16 a[4:7], a[144:147], v[196:199], 0\n" \
  "ds_write_b128 v252, v[228:231] offset:1024\n" \
  "v_mfma_f32_16x16x32_f16 a[8:11], a[144:147], v[200:203], 0\n" \
  "v_cvt_pk_f16_f32 v232, v162, v166\n" \
  "v_mfma_f32_16x16x32_f16 a[12:15], a[144:147], v[204:207], 0\n" \
  "v_cvt_pk_f16_f32 v233, v170, v174\n" \
  "s_waitcnt vmcnt(34)\n" \
  "v_mfma_f32_16x16x32_f16 a[44:47], a[148:151], v[204:207], 0\n" \
  "v_cvt_pk_f16_f32 v234, v178, v182\n" \
  "v_mfma_f32_16x16x32_f16 a[40:43], a[148:151], v[200:203], 0\n" \
  "v_cvt_pk_f16_f32 v235, v186, v190\n" \
  "v_mfma_f32_16x16x32_f16 a[36:39], a[148:151], v[196:199], 0\n" \
  "ds_write_b128 v252, v[232:235] offset:2048\n" \
  "v_mfma_f32_16x16x32_f16 a[32:35], a[148:151], v[192:195], 0\n" \
  "v_cvt_pk_f16_f32 v236, v163, v167\n" \
  "s_waitcnt vmcnt(33)\n" \
  "v_mfma_f32_16x16x32_f16 a[64:67], a[152:155], v[192:195], 0\n" \
  "v_cvt_pk_f16_f32 v237, v171, v175\n" \
  "v_mfma_f32_16x16x32_f16 a[68:71], a[152:155], v[196:199], 0\n" \
  "v_cvt_pk_f16_f32 v238, v179, v183\n" \
  "v_mfma_f32_16x16x32_f16 a[72:75], a[152:155], v[200:203], 0\n" \
  "v_cvt_pk_f16_f32 v239, v187, v191\n" \
  "v_mfma_f32_16x16x32_f16 a[76:79], a[152:155], v[204:207], 0\n" \
  "ds_write_b128 v252, v[236:239] offset:3072\n" \
  "s_waitcnt vmcnt(32)\n" \
  "v_mfma_f32_16x16x32_f16 a[108:111], a[156:159], v[204:207], 0\n" \
  "v_mfma_f32_16x16x32_f16 a[104:107], a[156:159], v[200:203], 0\n" \
  "v_mfma_f32_16x16x32_f16 a[100:103], a[156:159], v[196:199], 0\n" \
  "v_mfma_f32_16x16x32_f16 a[96:99], a[156:159], v[192:195], 0\n" \
  "s_waitcnt lgkmcnt(0)\n" \
  "v_mfma_f32_16x16x32_f16 v[16:19], a[128:131], v[208:211], 0\n" \
  "v_mfma_f32_16x16x32_f16 v[20:23], a[128:131], v[212:215], 0\n" \
  "v_mfma_f32_16x16x32_f16 v[24:27], a[128:131], v[216:219], 0\n" \
  "v_mfma_f32_16x16x32_f16 v[28:31], a[128:131], v[220:223], 0\n" \
  "ds_read_b128 v[192:195], v251 offset:8192\n" \
  "v_mfma_f32_16x16x32_f16 v[60:63], a[132:135], v[220:223], 0\n" \
  "v_mfma_f32_16x16x32_f16 v[56:59], a[132:135], v[216:219], 0\n" \
  "ds_read_b128 v[196:199], v251 offset:9216\n" \
  "v_mfma_f32_16x16x32_f16 v[52:55], a[132:135], v[212:215], 0\n" \
  "v_mfma_f32_16x16x32_f16 v[48:51], a[132:135], v[208:211], 0\n" \
  "ds_read_b128 v[200:203], v251 offset:10240\n" \
  "v_mfma_f32_16x16x32_f16 v[80:83], a[136:139], v[208:211], 0\n" \
  "v_mfma_f32_16x16x32_f16 v[84:87], a[136:139], v[212:215], 0\n" \
  "ds_read_b128 v[204:207], v251 offset:11264\n" \
  "v_mfma_f32_16x16x32_f16 v[88:91], a[136:139], v[216:219], 0\n" \
  "v_mfma_f32_16x16x32_f16 v[92:95], a[136:139], v[220:223], 0\n" \
  "v_mfma_f32_16x16x32_f16 v[124:127], a[140:143], v[220:223], 0\n" \
  "v_mfma_f32_16x16x32_f16 v[120:123], a[140:143], v[216:219], 0\n" \
  "v_mfma_f32_16x16x32_f16 v[116:119], a[140:143], v[212:215], 0\n" \
  "v_mfma_f32_16x16x32_f16 v[112:115], a[140:143], v[208:211], 0\n" \
  "v_mfma_f32_16x16x32_f16 a[16:19], a[144:147], v[208:211], 0\n" \
  "v_mfma_f32_16x16x32_f16 a[20:23], a[144:147], v[212:215], 0\n" \
  "v_mfma_f32_16x16x32_f16 a[24:27], a[144:147], v[216:219], 0\n" \
  "v_mfma_f32_16x16x32_f16 a[28:31], a[144:147], v[220:223], 0\n" \
  "v_mfma_f32_16x16x32_f16 a[60:63], a[148:151], v[220:223], 0\n" \
  "v_mfma_f32_16x16x32_f16 a[56:59], a[148:151], v[216:219], 0\n" \
  "v_mfma_f32_16x16x32_f16 a[52:55], a[148:151], v[212:215], 0\n" \
  "v_mfma_f32_16x16x32_f16 a[48:51], a[148:151], v[208:211], 0\n" \
  "v_mfma_f32_16x16x32_f16 a[80:83], a[152:155], v[208:211], 0\n" \
  "v_mfma_f32_16x16x32_f16 a[84:87], a[152:155], v[212:215], 0\n" \
  "v_mfma_f32_16x16x32_f16 a[88:91], a[152:155], v[216:219], 0\n" \
  "v_mfma_f32_16x16x32_f16 a[92:95], a[152:155], v[220:223], 0\n" \
  "v_mfma_f32_16x16x32_f16 a[124:127], a[156:159], v[220:223], 0\n" \
  "v_mfma_f32_16x16x32_f16 a[120:123], a[156:159], v[216:219], 0\n" \
  "v_mfma_f32_16x16x32_f16 a[116:119], a[156:159], v[212:215], 0\n" \
  "v_mfma_f32_16x16x32_f16 a[112:115], a[156:159], v[208:211], 0\n" \
  "s_waitcnt lgkmcnt(0)\n" \
  "s_waitcnt vmcnt(48)\n" \
  "v_cvt_pk_f16_f32 v240, v240, v241\n" \
  "v_cvt_pk_f16_f32 v241, v242, v243\n" \
  "v_cvt_pk_f16_f32 v242, v244, v245\n" \
  "v_cvt_pk_f16_f32 v243, v246, v247\n" \
  "buffer_store_dwordx4 v[240:243], v248, s[40:43], s87 offen sc0 sc1\n" \
  "NS17_%=:\n" \
  "s_waitcnt vmcnt(31)\n" \
  "v_mfma_f32_16x16x32_f16 v[0:3], a[160:163], v[192:195], v[0:3]\n" \
  "ds_read_b128 v[208:211], v251 offset:12288\n" \
  "s_add_u32 s79, s68, s61\n" \
  "s_add_u32 s80, s79, s84\n" \
  "s_add_u32 s79, s79, s83\n" \
  "v_mfma_f32_16x16x32_f16 v[4:7], a[160:163], v[196:199], v[4:7]\n" \
  "buffer_load_dwordx4 a[128:131], v248, s[24:27], s79 offen offset:0 sc1\n" \
  "v_mfma_f32_16x16x32_f16 v[8:11], a[160:163], v[200:203], v[8:11]\n" \
  "ds_read_b128 v[212:215], v251 offset:13312\n" \
  "v_mfma_f32_16x16x32_f16 v[12:15], a[160:163], v[204:207], v[12:15]\n" \
  "s_waitcnt vmcnt(31)\n" \
  "v_mfma_f32_16x16x32_f16 v[44:47], a[164:167], v[204:207], v[44:47]\n" \
  "ds_read_b128 v[216:219], v251 offset:14336\n" \
  "v_mfma_f32_16x16x32_f16 v[40:43], a[164:167], v[200:203], v[40:43]\n" \
  "buffer_load_dwordx4 a[132:135], v248, s[24:27], s79 offen offset:1024 sc1\n" \
  "v_mfma_f32_16x16x32_f16 v[36:39], a[164:167], v[196:199], v[36:39]\n" \
  "ds_read_b128 v[220:223], v251 offset:15360\n" \
  "v_mfma_f32_16x16x32_f16 v[32:35], a[164:167], v[192:195], v[32:35]\n" \
  "s_waitcnt vmcnt(31)\n" \
  "v_mfma_f32_16x16x32_f16 v[64:67], a[168:171], v[192:195], v[64:67]\n" \
  "v_mfma_f32_16x16x32_f16 v[68:71], a[168:171], v[196:199], v[68:71]\n" \
  "buffer_load_dwordx4 a[136:139], v248, s[24:27], s79 offen offset:2048 sc1\n" \
  "v_mfma_f32_16x16x32_f16 v[72:75], a[168:171], v[200:203], v[72:75]\n" \
  "v_mfma_f32_16x16x32_f16 v[76:79], a[168:171], v[204:207], v[76:79]\n" \
  "s_waitcnt vmcnt(31)\n" \
  "v_mfma_f32_16x16x32_f16 v[108:111], a[172:175], v[204:207], v[108:111]\n" \
  "v_mfma_f32_16x16x32_f16 v[104:107], a[172:175], v[200:203], v[104:107]\n" \
  "buffer_load_dwordx4 a[140:143], v248, s[24:27], s79 offen offset:3072 sc1\n" \
  "v_mfma_f32_16x16x32_f16 v[100:103], a[172:175], v[196:199], v[100:103]\n" \
  "v_mfma_f32_16x16x32_f16 v[96:99], a[172:175], v[192:195], v[96:99]\n" \
  "s_waitcnt vmcnt(31)\n" \
  "v_mfma_f32_16x16x32_f16 a[0:3], a[176:179], v[192:195], a[0:3]\n" \
  "v_mfma_f32_16x16x32_f16 a[4:7], a[176:179], v[196:199], a[4:7]\n" \
  "buffer_load_dwordx4 a[144:147], v248, s[24:27], s80 offen offset:0 sc1\n" \
  "v_mfma_f32_16x16x32_f16 a[8:11], a[176:179], v[200:203], a[8:11]\n" \
  "v_mfma_f32_16x16x32_f16 a[12:15], a[176:179], v[204:207], a[12:15]\n" \
  "s_lshl_b32 s74, %8, 2\n" \
  "s_add_u32 s74, s74, %9\n" \
  "s_waitcnt vmcnt(31)\n" \
  "v_mfma_f32_16x16x32_f16 a[44:47], a[180:183], v[204:207], a[44:47]\n" \
  "v_mov_b32 v226, s74\n" \
  "v_mov_b32 v227, %9\n" \
  "v_readlane_b32 s75, %1, s67\n" \
  "v_mfma_f32_16x16x32_f16 a[40:43], a[180:183], v[200:203], a[40:43]\n" \
  "s_waitcnt vmcnt(5)\n" \
  "v_cmp_le_u32 vcc, s82, v253\n" \
  "buffer_load_dwordx4 a[148:151], v248, s[24:27], s80 offen offset:1024 sc1\n" \
  "v_mfma_f32_16x16x32_f16 a[36:39], a[180:183], v[196:199], a[36:39]\n" \
  "s_lshr_b32 s75, s75, 24\n" \
  "s_bfm_b64 s[52:53], s75, 0\n" \
  "v_mfma_f32_16x16x32_f16 a[32:35], a[180:183], v[192:195], a[32:35]\n" \
  "s_and_b32 s75, s52, s59\n" \
  "s_and_b32 s74, vcc_lo, s75\n" \
  "s_waitcnt vmcnt(31)\n" \
  "v_mfma_f32_16x16x32_f16 a[64:67], a[184:187], v[192:195], a[64:67]\n" \
  "s_cmp_eq_u32 s74, 0\n" \
  "s_cselect_b32 s76, s75, 0\n" \
  "s_and_b32 s76, s76, s91\n" \
  "s_cbranch_scc1 DSLOW14_%=\n" \
  "DOK13_%=:\n" \
  "v_mfma_f32_16x16x32_f16 a[68:71], a[184:187], v[196:199], a[68:71]\n" \
  "s_ff1_i32_b32 s74, s74\n" \
  "s_add_u32 s75, s67, 2\n" \
  "buffer_load_dwordx4 a[152:155], v248, s[24:27], s80 offen offset:2048 sc1\n" \
  "v_mfma_f32_16x16x32_f16 a[72:75], a[184:187], v[200:203], a[72:75]\n" \
  "s_cmp_eq_u32 s59, 0\n" \
  "s_cselect_b32 s74, s75, s74\n" \
  "v_mfma_f32_16x16x32_f16 a[76:79], a[184:187], v[204:207], a[76:79]\n" \
  "v_mov_b32 v224, s74\n" \
  "ds_write_b32 v226, v224 offset:49152\n" \
  "s_waitcnt vmcnt(31)\n" \
  "v_mfma_f32_16x16x32_f16 a[108:111], a[188:191], v[204:207], a[108:111]\n" \
  "v_mfma_f32_16x16x32_f16 a[104:107], a[188:191], v[200:203], a[104:107]\n" \
  "buffer_load_dwordx4 a[156:159], v248, s[24:27], s80 offen offset:3072 sc1\n" \
  "v_mfma_f32_16x16x32_f16 a[100:103], a[188:191], v[196:199], a[100:103]\n" \
  "v_mfma_f32_16x16x32_f16 a[96:99], a[188:191], v[192:195], a[96:99]\n" \
  "s_waitcnt lgkmcnt(0)\n" \
  "s_barrier\n" \
  "v_mfma_f32_16x16x32_f16 v[16:19], a[160:163], v[208:211], v[16:19]\n" \
  "s_add_u32 s81, s81, 1\n" \
  "s_cmp_eq_u32 s81, 3\n" \
  "s_cselect_b32 s81, 0, s81\n" \
  "s_movk_i32 s74, 0x4000\n" \
  "s_cmov_b32 s74, 0xffff8000\n" \
  "v_mfma_f32_16x16x32_f16 v[20:23], a[160:163], v[212:215], v[20:23]\n" \
  "ds_read_b32 v228, v227 offset:49152\n" \
  "v_add_u32 v251, s74, v251\n" \
  "s_cmp_eq_u32 s81, 2\n" \
  "s_movk_i32 s74, 0x4000\n" \
  "s_cmov_b32 s74, 0xffff8000\n" \
  "v_mfma_f32_16x16x32_f16 v[24:27], a[160:163], v[216:219], v[24:27]\n" \
  "v_add_u32 v252, s74, v252\n" \
  "s_add_u32 s79, s68, s62\n" \
  "s_add_u32 s80, s79, s84\n" \
  "s_add_u32 s79, s79, s83\n" \
  "v_mfma_f32_16x16x32_f16 v[28:31], a[160:163], v[220:223], v[28:31]\n" \
  "ds_read_b128 v[192:195], v251 offset:0\n" \
  "buffer_load_dwordx4 a[160:163], v248, s[24:27], s79 offen offset:0 sc1\n" \
  "v_mfma_f32_16x16x32_f16 v[60:63], a[164:167], v[220:223], v[60:63]\n" \
  "v_mfma_f32_16x16x32_f16 v[56:59], a[164:167], v[216:219], v[56:59]\n" \
  "ds_read_b128 v[196:199], v251 offset:1024\n" \
  "v_mfma_f32_16x16x32_f16 v[52:55], a[164:167], v[212:215], v[52:55]\n" \
  "v_mfma_f32_16x16x32_f16 v[48:51], a[164:167], v[208:211], v[48:51]\n" \
  "ds_read_b128 v[200:203], v251 offset:2048\n" \
  "buffer_load_dwordx4 a[164:167], v248, s[24:27], s79 offen offset:1024 sc1\n" \
  "v_mfma_f32_16x16x32_f16 v[80:83], a[168:171], v[208:211], v[80:83]\n" \
  "v_mfma_f32_16x16x32_f16 v[84:87], a[168:171], v[212:215], v[84:87]\n" \
  "ds_read_b128 v[204:207], v251 offset:3072\n" \
  "v_mfma_f32_16x16x32_f16 v[88:91], a[168:171], v[216:219], v[88:91]\n" \
  "v_mfma_f32_16x16x32_f16 v[92:95], a[168:171], v[220:223], v[92:95]\n" \
  "buffer_load_dwordx4 a[168:171], v248, s[24:27], s79 offen offset:2048 sc1\n" \
  "v_mfma_f32_16x16x32_f16 v[124:127], a[172:175], v[220:223], v[124:127]\n" \
  "v_mfma_f32_16x16x32_f16 v[120:123], a[172:175], v[216:219], v[120:123]\n" \
  "v_mfma_f32_16x16x32_f16 v[116:119], a[172:175], v[212:215], v[116:119]\n" \
  "v_mfma_f32_16x16x32_f16 v[112:115], a[172:175], v[208:211], v[112:115]\n" \
  "buffer_load_dwordx4 a[172:175], v248, s[24:27], s79 offen offset:3072 sc1\n" \
  "v_mfma_f32_16x16x32_f16 a[16:19], a[176:179], v[208:211], a[16:19]\n" \
  "v_mfma_f32_16x16x32_f16 a[20:23], a[176:179], v[212:215], a[20:23]\n" \
  "v_mfma_f32_16x16x32_f16 a[24:27], a[176:179], v[216:219], a[24:27]\n" \
  "v_mfma_f32_16x16x32_f16 a[28:31], a[176:179], v[220:223], a[28:31]\n" \
  "buffer_load_dwordx4 a[176:179], v248, s[24:27], s80 offen offset:0 sc1\n" \
  "v_mfma_f32_16x16x32_f16 a[60:63], a[180:183], v[220:223], a[60:63]\n" \
  "v_mfma_f32_16x16x32_f16 a[56:59], a[180:183], v[216:219], a[56:59]\n" \
  "v_mfma_f32_16x16x32_f16 a[52:55], a[180:183], v[212:215], a[52:55]\n" \
  "v_mfma_f32_16x16x32_f16 a[48:51], a[180:183], v[208:211], a[48:51]\n" \
  "buffer_load_dwordx4 a[180:183], v248, s[24:27], s80 offen offset:1024 sc1\n" \
  "v_mfma_f32_16x16x32_f16 a[80:83], a[184:187], v[208:211], a[80:83]\n" \
  "v_mfma_f32_16x16x32_f16 a[84:87], a[184:187], v[212:215], a[84:87]\n" \
  "v_mfma_f32_16x16x32_f16 a[88:91], a[184:187], v[216:219], a[88:91]\n" \
  "v_mfma_f32_16x16x32_f16 a[92:95], a[184:187], v[220:223], a[92:95]\n" \
  "buffer_load_dwordx4 a[184:187], v248, s[24:27], s80 offen offset:2048 sc1\n" \
  "v_mfma_f32_16x16x32_f16 a[124:127], a[188:191], v[220:223], a[124:127]\n" \
  "v_mfma_f32_16x16x32_f16 a[120:123], a[188:191], v[216:219], a[120:123]\n" \
  "v_mfma_f32_16x16x32_f16 a[116:119], a[188:191], v[212:215], a[116:119]\n" \
  "v_mfma_f32_16x16x32_f16 a[112:115], a[188:191], v[208:211], a[112:115]\n" \
  "buffer_load_dwordx4 a[188:191], v248, s[24:27], s80 offen offset:3072 sc1\n" \
  "s_waitcnt lgkmcnt(0)\n" \
  "v_readfirstlane_b32 s60, v228\n" \
  "s_waitcnt vmcnt(39)\n" \
  "v_mfma_f32_16x16x32_f16 v[0:3], a[192:195], v[192:195], v[0:3]\n" \
  "ds_read_b128 v[208:211], v251 offset:4096\n" \
  "v_mfma_f32_16x16x32_f16 v[4:7], a[192:195], v[196:199], v[4:7]\n" \
  "s_add_u32 s78, s69, s66\n" \
  "s_add_u32 s72, s78, 0\n" \
  "buffer_load_dwordx4 v[160:163], v249, s[28:31], s72 offen\n" \
  "v_mfma_f32_16x16x32_f16 v[8:11], a[192:195], v[200:203], v[8:11]\n" \
  "v_readlane_b32 s56, %1, s60\n" \
  "s_bitset0_b32 s59, s60\n" \
  "ds_read_b128 v[212:215], v251 offset:5120\n" \
  "v_mfma_f32_16x16x32_f16 v[12:15], a[192:195], v[204:207], v[12:15]\n" \
  "s_waitcnt vmcnt(39)\n" \
  "v_mfma_f32_16x16x32_f16 v[44:47], a[196:199], v[204:207], v[44:47]\n" \
  "s_and_b32 s56, s56, 63\n" \
  "ds_read_b128 v[216:219], v251 offset:6144\n" \
  "v_mfma_f32_16x16x32_f16 v[40:43], a[196:199], v[200:203], v[40:43]\n" \
  "s_lshl_b32 s57, s56, 22\n" \
  "s_lshl_b32 s58, s56, 8\n" \
  "s_add_u32 s73, s78, 32768\n" \
  "buffer_load_dwordx4 v[164:167], v249, s[28:31], s73 offen\n" \
  "v_mfma_f32_16x16x32_f16 v[36:39], a[196:199], v[196:199], v[36:39]\n" \
  "s_lshl_b32 s56, s56, 18\n" \
  "s_add_u32 s58, s58, s71\n" \
  "ds_read_b128 v[220:223], v251 offset:7168\n" \
  "v_mfma_f32_16x16x32_f16 v[32:35], a[196:199], v[192:195], v[32:35]\n" \
  "s_waitcnt vmcnt(39)\n" \
  "v_mfma_f32_16x16x32_f16 v[64:67], a[200:203], v[192:195], v[64:67]\n" \
  "s_waitcnt vmcnt(26)\n" \
  "v_cvt_pk_f16_f32 v224, v128, v132\n" \
  "v_mfma_f32_16x16x32_f16 v[68:71], a[200:203], v[196:199], v[68:71]\n" \
  "v_cvt_pk_f16_f32 v225, v136, v140\n" \
  "s_add_u32 s72, s78, 65536\n" \
  "buffer_load_dwordx4 v[168:171], v249, s[28:31], s72 offen\n" \
  "v_mfma_f32_16x16x32_f16 v[72:75], a[200:203], v[200:203], v[72:75]\n" \
  "v_cvt_pk_f16_f32 v226, v144, v148\n" \
  "v_mfma_f32_16x16x32_f16 v[76:79], a[200:203], v[204:207], v[76:79]\n" \
  "v_cvt_pk_f16_f32 v227, v152, v156\n" \
  "s_waitcnt vmcnt(39)\n" \
  "v_mfma_f32_16x16x32_f16 v[108:111], a[204:207], v[204:207], v[108:111]\n" \
  "ds_write_b128 v252, v[224:227] offset:0\n" \
  "v_mfma_f32_16x16x32_f16 v[104:107], a[204:207], v[200:203], v[104:107]\n" \
  "v_cvt_pk_f16_f32 v228, v129, v133\n" \
  "s_add_u32 s73, s78, 98304\n" \
  "buffer_load_dwordx4 v[172:175], v249, s[28:31], s73 offen\n" \
  "v_mfma_f32_16x16x32_f16 v[100:103], a[204:207], v[196:199], v[100:103]\n" \
  "v_cvt_pk_f16_f32 v229, v137, v141\n" \
  "v_mfma_f32_16x16x32_f16 v[96:99], a[204:207], v[192:195], v[96:99]\n" \
  "v_cvt_pk_f16_f32 v230, v145, v149\n" \
  "s_waitcnt vmcnt(39)\n" \
  "v_mfma_f32_16x16x32_f16 a[0:3], a[208:211], v[192:195], a[0:3]\n" \
  "v_cvt_pk_f16_f32 v231, v153, v157\n" \
  "v_mfma_f32_16x16x32_f16 a[4:7], a[208:211], v[196:199], a[4:7]\n" \
  "ds_write_b128 v252, v[228:231] offset:1024\n" \
  "s_add_u32 s72, s78, 131072\n" \
  "buffer_load_dwordx4 v[176:179], v249, s[28:31], s72 offen\n" \
  "v_mfma_f32_16x16x32_f16 a[8:11], a[208:211], v[200:203], a[8:11]\n" \
  "v_cvt_pk_f16_f32 v232, v130, v134\n" \
  "v_mfma_f32_16x16x32_f16 a[12:15], a[208:211], v[204:207], a[12:15]\n" \
  "v_cvt_pk_f16_f32 v233, v138, v142\n" \
  "s_waitcnt vmcnt(39)\n" \
  "v_mfma_f32_16x16x32_f16 a[44:47], a[212:215], v[204:207], a[44:47]\n" \
  "v_cvt_pk_f16_f32 v234, v146, v150\n" \
  "v_mfma_f32_16x16x32_f16 a[40:43], a[212:215], v[200:203], a[40:43]\n" \
  "v_cvt_pk_f16_f32 v235, v154, v158\n" \
  "s_add_u32 s73, s78, 163840\n" \
  "buffer_load_dwordx4 v[180:183], v249, s[28:31], s73 offen\n" \
  "v_mfma_f32_16x16x32_f16 a[36:39], a[212:215], v[196:199], a[36:39]\n" \
  "ds_write_b128 v252, v[232:235] offset:2048\n" \
  "v_mfma_f32_16x16x32_f16 a[32:35], a[212:215], v[192:195], a[32:35]\n" \
  "v_cvt_pk_f16_f32 v236, v131, v135\n" \
  "s_waitcnt vmcnt(39)\n" \
  "v_mfma_f32_16x16x32_f16 a[64:67], a[216:219], v[192:195], a[64:67]\n" \
  "v_cvt_pk_f16_f32 v237, v139, v143\n" \
  "v_mfma_f32_16x16x32_f16 a[68:71], a[216:219], v[196:199], a[68:71]\n" \
  "v_cvt_pk_f16_f32 v238, v147, v151\n" \
  "s_add_u32 s72, s78, 196608\n" \
  "buffer_load_dwordx4 v[184:187], v249, s[28:31], s72 offen\n" \
  "v_mfma_f32_16x16x32_f16 a[72:75], a[216:219], v[200:203], a[72:75]\n" \
  "v_cvt_pk_f16_f32 v239, v155, v159\n" \
  "v_mfma_f32_16x16x32_f16 a[76:79], a[216:219], v[204:207], a[76:79]\n" \
  "ds_write_b128 v252, v[236:239] offset:3072\n" \
  "s_waitcnt vmcnt(39)\n" \
  "v_mfma_f32_16x16x32_f16 a[108:111], a[220:223], v[204:207], a[108:111]\n" \
  "v_mfma_f32_16x16x32_f16 a[104:107], a[220:223], v[200:203], a[104:107]\n" \
  "s_add_u32 s73, s78, 229376\n" \
  "buffer_load_dwordx4 v[188:191], v249, s[28:31], s73 offen\n" \
  "v_mfma_f32_16x16x32_f16 a[100:103], a[220:223], v[196:199], a[100:103]\n" \
  "v_mfma_f32_16x16x32_f16 a[96:99], a[220:223], v[192:195], a[96:99]\n" \
  "s_waitcnt lgkmcnt(0)\n" \
  "v_mfma_f32_16x16x32_f16 v[16:19], a[192:195], v[208:211], v[16:19]\n" \
  "v_mfma_f32_16x16x32_f16 v[20:23], a[192:195], v[212:215], v[20:23]\n" \
  "v_mfma_f32_16x16x32_f16 v[24:27], a[192:195], v[216:219], v[24:27]\n" \
  "s_add_u32 s79, s68, s63\n" \
  "s_add_u32 s80, s79, s84\n" \
  "s_add_u32 s79, s79, s83\n" \
  "v_mfma_f32_16x16x32_f16 v[28:31], a[192:195], v[220:223], v[28:31]\n" \
  "ds_read_b128 v[192:195], v251 offset:8192\n" \
  "buffer_load_dwordx4 a[192:195], v248, s[24:27], s79 offen offset:0 sc1\n" \
  "v_mfma_f32_16x16x32_f16 v[60:63], a[196:199], v[220:223], v[60:63]\n" \
  "v_mfma_f32_16x16x32_f16 v[56:59], a[196:199], v[216:219], v[56:59]\n" \
  "ds_read_b128 v[196:199], v251 offset:9216\n" \
  "v_mfma_f32_16x16x32_f16 v[52:55], a[196:199], v[212:215], v[52:55]\n" \
  "v_mfma_f32_16x16x32_f16 v[48:51], a[196:199], v[208:211], v[48:51]\n" \
  "ds_read_b128 v[200:203], v251 offset:10240\n" \
  "buffer_load_dwordx4 a[196:199], v248, s[24:27], s79 offen offset:1024 sc1\n" \
  "v_mfma_f32_16x16x32_f16 v[80:83], a[200:203], v[208:211], v[80:83]\n" \
  "v_mfma_f32_16x16x32_f16 v[84:87], a[200:203], v[212:215], v[84:87]\n" \
  "ds_read_b128 v[204:207], v251 offset:11264\n" \
  "v_mfma_f32_16x16x32_f16 v[88:91], a[200:203], v[216:219], v[88:91]\n" \
  "v_mfma_f32_16x16x32_f16 v[92:95], a[200:203], v[220:223], v[92:95]\n" \
  "buffer_load_dwordx4 a[200:203], v248, s[24:27], s79 offen offset:2048 sc1\n" \
  "v_mfma_f32_16x16x32_f16 v[124:127], a[204:207], v[220:223], v[124:127]\n" \
  "v_mfma_f32_16x16x32_f16 v[120:123], a[204:207], v[216:219], v[120:123]\n" \
  "v_mfma_f32_16x16x32_f16 v[116:119], a[204:207], v[212:215], v[116:119]\n" \
  "v_mfma_f32_16x16x32_f16 v[112:115], a[204:207], v[208:211], v[112:115]\n" \
  "buffer_load_dwordx4 a[204:207], v248, s[24:27], s79 offen offset:3072 sc1\n" \
  "v_mfma_f32_16x16x32_f16 a[16:19], a[208:211], v[208:211], a[16:19]\n" \
  "v_mfma_f32_16x16x32_f16 a[20:23], a[208:211], v[212:215], a[20:23]\n" \
  "v_mfma_f32_16x16x32_f16 a[24:27], a[208:211], v[216:219], a[24:27]\n" \
  "v_mfma_f32_16x16x32_f16 a[28:31], a[208:211], v[220:223], a[28:31]\n" \
  "buffer_load_dwordx4 a[208:211], v248, s[24:27], s80 offen offset:0 sc1\n" \
  "v_mfma_f32_16x16x32_f16 a[60:63], a[212:215], v[220:223], a[60:63]\n" \
  "v_mfma_f32_16x16x32_f16 a[56:59], a[212:215], v[216:219], a[56:59]\n" \
  "v_mfma_f32_16x16x32_f16 a[52:55], a[212:215], v[212:215], a[52:55]\n" \
  "v_mfma_f32_16x16x32_f16 a[48:51], a[212:215], v[208:211], a[48:51]\n" \
  "buffer_load_dwordx4 a[212:215], v248, s[24:27], s80 offen offset:1024 sc1\n" \
  "v_mfma_f32_16x16x32_f16 a[80:83], a[216:219], v[208:211], a[80:83]\n" \
  "v_mfma_f32_16x16x32_f16 a[84:87], a[216:219], v[212:215], a[84:87]\n" \
  "v_mfma_f32_16x16x32_f16 a[88:91], a[216:219], v[216:219], a[88:91]\n" \
  "v_mfma_f32_16x16x32_f16 a[92:95], a[216:219], v[220:223], a[92:95]\n" \
  "buffer_load_dwordx4 a[216:219], v248, s[24:27], s80 offen offset:2048 sc1\n" \
  "v_mfma_f32_16x16x32_f16 a[124:127], a[220:223], v[220:223], a[124:127]\n" \
  "v_mfma_f32_16x16x32_f16 a[120:123], a[220:223], v[216:219], a[120:123]\n" \
  "v_mfma_f32_16x16x32_f16 a[116:119], a[220:223], v[212:215], a[116:119]\n" \
  "v_mfma_f32_16x16x32_f16 a[112:115], a[220:223], v[208:211], a[112:115]\n" \
  "buffer_load_dwordx4 a[220:223], v248, s[24:27], s80 offen offset:3072 sc1\n" \
  "s_waitcnt lgkmcnt(0)\n" \
  "s_waitcnt vmcnt(39)\n" \
  "v_mfma_f32_16x16x32_f16 v[0:3], a[224:227], v[192:195], v[0:3]\n" \
  "ds_read_b128 v[208:211], v251 offset:12288\n" \
  "s_add_u32 s78, s57, s65\n" \
  "s_add_u32 s72, s78, 0\n" \
  "buffer_load_dwordx4 v[128:131], v249, s[28:31], s72 offen\n" \
  "v_mfma_f32_16x16x32_f16 v[4:7], a[224:227], v[196:199], v[4:7]\n" \
  "v_bfe_u32 v232, %1, 8, 16\n" \
  "s_and_b32 exec_lo, s59, s91\n" \
  "s_cbranch_scc0 PL21_%=\n" \
  "s_mov_b32 exec_hi, 0\n" \
  "buffer_load_dword v253, v232, s[36:39], s90 offen sc1\n" \
  "PL21_%=:\n" \
  "s_mov_b64 exec, -1\n" \
  "v_mfma_f32_16x16x32_f16 v[8:11], a[224:227], v[200:203], v[8:11]\n" \
  "ds_read_b128 v[212:215], v251 offset:13312\n" \
  "v_mfma_f32_16x16x32_f16 v[12:15], a[224:227], v[204:207], v[12:15]\n" \
  "s_waitcnt vmcnt(39)\n" \
  "v_mfma_f32_16x16x32_f16 v[44:47], a[228:231], v[204:207], v[44:47]\n" \
  "ds_read_b128 v[216:219], v251 offset:14336\n" \
  "s_add_u32 s73, s78, 32768\n" \
  "buffer_load_dwordx4 v[132:135], v249, s[28:31], s73 offen\n" \
  "v_mfma_f32_16x16x32_f16 v[40:43], a[228:231], v[200:203], v[40:43]\n" \
  "v_mfma_f32_16x16x32_f16 v[36:39], a[228:231], v[196:199], v[36:39]\n" \
  "ds_read_b128 v[220:223], v251 offset:15360\n" \
  "v_mfma_f32_16x16x32_f16 v[32:35], a[228:231], v[192:195], v[32:35]\n" \
  "s_waitcnt vmcnt(39)\n" \
  "v_mfma_f32_16x16x32_f16 v[64:67], a[232:235], v[192:195], v[64:67]\n" \
  "s_add_u32 s72, s78, 65536\n" \
  "buffer_load_dwordx4 v[136:139], v249, s[28:31], s72 offen\n" \
  "v_mfma_f32_16x16x32_f16 v[68:71], a[232:235], v[196:199], v[68:71]\n" \
  "v_mfma_f32_16x16x32_f16 v[72:75], a[232:235], v[200:203], v[72:75]\n" \
  "v_mfma_f32_16x16x32_f16 v[76:79], a[232:235], v[204:207], v[76:79]\n" \
  "s_waitcnt vmcnt(39)\n" \
  "v_mfma_f32_16x16x32_f16 v[108:111], a[236:239], v[204:207], v[108:111]\n" \
  "s_add_u32 s73, s78, 98304\n" \
  "buffer_load_dwordx4 v[140:143], v249, s[28:31], s73 offen\n" \
  "v_mfma_f32_16x16x32_f16 v[104:107], a[236:239], v[200:203], v[104:107]\n" \
  "v_mfma_f32_16x16x32_f16 v[100:103], a[236:239], v[196:199], v[100:103]\n" \
  "v_mfma_f32_16x16x32_f16 v[96:99], a[236:239], v[192:195], v[96:99]\n" \
  "s_waitcnt vmcnt(39)\n" \
  "v_mfma_f32_16x16x32_f16 a[0:3], a[240:243], v[192:195], a[0:3]\n" \
  "s_add_u32 s72, s78, 131072\n" \
  "buffer_load_dwordx4 v[144:147], v249, s[28:31], s72 offen\n" \
  "v_mfma_f32_16x16x32_f16 a[4:7], a[240:243], v[196:199], a[4:7]\n" \
  "v_mfma_f32_16x16x32_f16 a[8:11], a[240:243], v[200:203], a[8:11]\n" \
  "v_mfma_f32_16x16x32_f16 a[12:15], a[240:243], v[204:207], a[12:15]\n" \
  "s_waitcnt vmcnt(39)\n" \
  "v_mfma_f32_16x16x32_f16 a[44:47], a[244:247], v[204:207], a[44:47]\n" \
  "s_add_u32 s73, s78, 163840\n" \
  "buffer_load_dwordx4 v[148:151], v249, s[28:31], s73 offen\n" \
  "v_mfma_f32_16x16x32_f16 a[40:43], a[244:247], v[200:203], a[40:43]\n" \
  "v_mfma_f32_16x16x32_f16 a[36:39], a[244:247], v[196:199], a[36:39]\n" \
  "v_mfma_f32_16x16x32_f16 a[32:35], a[244:247], v[192:195], a[32:35]\n" \
  "s_waitcnt vmcnt(39)\n" \
  "v_mfma_f32_16x16x32_f16 a[64:67], a[248:251], v[192:195], a[64:67]\n" \
  "s_add_u32 s72, s78, 196608\n" \
  "buffer_load_dwordx4 v[152:155], v249, s[28:31], s72 offen\n" \
  "v_mfma_f32_16x16x32_f16 a[68:71], a[248:251], v[196:199], a[68:71]\n" \
  "v_mfma_f32_16x16x32_f16 a[72:75], a[248:251], v[200:203], a[72:75]\n" \
  "v_mfma_f32_16x16x32_f16 a[76:79], a[248:251], v[204:207], a[76:79]\n" \
  "s_waitcnt vmcnt(39)\n" \
  "v_mfma_f32_16x16x32_f16 a[108:111], a[252:255], v[204:207], a[108:111]\n" \
  "s_add_u32 s73, s78, 229376\n" \
  "buffer_load_dwordx4 v[156:159], v249, s[28:31], s73 offen\n" \
  "v_mfma_f32_16x16x32_f16 a[104:107], a[252:255], v[200:203], a[104:107]\n" \
  "v_mfma_f32_16x16x32_f16 a[100:103], a[252:255], v[196:199], a[100:103]\n" \
  "v_mfma_f32_16x16x32_f16 a[96:99], a[252:255], v[192:195], a[96:99]\n" \
  "s_waitcnt vmcnt(40)\n" \
  "s_waitcnt lgkmcnt(0)\n" \
  "s_barrier\n" \
  "v_mfma_f32_16x16x32_f16 v[16:19], a[224:227], v[208:211], v[16:19]\n" \
  "s_add_u32 s81, s81, 1\n" \
  "s_cmp_eq_u32 s81, 3\n" \
  "s_cselect_b32 s81, 0, s81\n" \
  "s_movk_i32 s74, 0x4000\n" \
  "s_cmov_b32 s74, 0xffff8000\n" \
  "buffer_load_dwordx4 v[240:243], v250, s[32:35], s86 offen nt\n" \
  "buffer_load_dwordx4 v[244:247], v250, s[32:35], s86 offen offset:16 nt\n" \
  "NX18_%=:\n" \
  "v_mfma_f32_16x16x32_f16 v[20:23], a[224:227], v[212:215], v[20:23]\n" \
  "v_add_u32 v251, s74, v251\n" \
  "s_cmp_eq_u32 s81, 2\n" \
  "s_movk_i32 s74, 0x4000\n" \
  "s_cmov_b32 s74, 0xffff8000\n" \
  "v_mfma_f32_16x16x32_f16 v[24:27], a[224:227], v[216:219], v[24:27]\n" \
  "v_add_u32 v252, s74, v252\n" \
  "v_mfma_f32_16x16x32_f16 v[28:31], a[224:227], v[220:223], v[28:31]\n" \
  "ds_read_b128 v[192:195], v251 offset:0\n" \
  "v_mfma_f32_16x16x32_f16 v[60:63], a[228:231], v[220:223], v[60:63]\n" \
  "v_mov_b32 v224, s82\n" \
  "s_cmp_lg_u32 %8, 0\n" \
  "s_cbranch_scc1 W020_%=\n" \
  "s_mov_b64 exec, 1\n" \
  "buffer_store_dword v224, off, s[40:43], s88 sc0 sc1\n" \
  "s_mov_b64 exec, -1\n" \
  "W020_%=:\n" \
  "NA19_%=:\n" \
  "v_mfma_f32_16x16x32_f16 v[56:59], a[228:231], v[216:219], v[56:59]\n" \
  "s_add_u32 s74, s67, 1\n" \
  "s_add_u32 s75, s67, 2\n" \
  "ds_read_b128 v[196:199], v251 offset:1024\n" \
  "v_mfma_f32_16x16x32_f16 v[52:55], a[228:231], v[212:215], v[52:55]\n" \
  "v_readlane_b32 s76, %0, s74\n" \
  "v_readlane_b32 s77, %0, s75\n" \
  "v_mfma_f32_16x16x32_f16 v[48:51], a[228:231], v[208:211], v[48:51]\n" \
  "ds_read_b128 v[200:203], v251 offset:2048\n" \
  "v_mfma_f32_16x16x32_f16 v[80:83], a[232:235], v[208:211], v[80:83]\n" \
  "s_and_b32 s74, s76, 7\n" \
  "s_lshl_b32 s74, s74, 3\n" \
  "v_mfma_f32_16x16x32_f16 v[84:87], a[232:235], v[212:215], v[84:87]\n" \
  "s_load_dwordx2 s[40:41], %5, s74\n" \
  "s_bfe_u32 s87, s76, 0x6000a\n" \
  "ds_read_b128 v[204:207], v251 offset:3072\n" \
  "v_mfma_f32_16x16x32_f16 v[88:91], a[232:235], v[216:219], v[88:91]\n" \
  "s_lshl_b32 s87, s87, 18\n" \
  "s_lshl_b32 s74, %8, 16\n" \
  "v_mfma_f32_16x16x32_f16 v[92:95], a[232:235], v[220:223], v[92:95]\n" \
  "s_or_b32 s87, s87, s74\n" \
  "s_bfe_u32 s74, s76, 0x60004\n" \
  "v_mfma_f32_16x16x32_f16 v[124:127], a[236:239], v[220:223], v[124:127]\n" \
  "s_lshl_b32 s74, s74, 10\n" \
  "s_or_b32 s87, s87, s74\n" \
  "v_mfma_f32_16x16x32_f16 v[120:123], a[236:239], v[216:219], v[120:123]\n" \
  "s_add_u32 s87, s87, s85\n" \
  "s_bfe_u32 s88, s76, 0xc0004\n" \
  "v_mfma_f32_16x16x32_f16 v[116:119], a[236:239], v[212:215], v[116:119]\n" \
  "s_lshl_b32 s88, s88, 2\n" \
  "s_add_u32 s88, s88, 67108864\n" \
  "v_mfma_f32_16x16x32_f16 v[112:115], a[236:239], v[208:211], v[112:115]\n" \
  "s_and_b32 s86, s77, 7\n" \
  "s_lshl_b32 s86, s86, 22\n" \
  "v_mfma_f32_16x16x32_f16 a[16:19], a[240:243], v[208:211], a[16:19]\n" \
  "s_bfe_u32 s74, s77, 0x60004\n" \
  "s_lshl_b32 s74, s74, 16\n" \
  "v_mfma_f32_16x16x32_f16 a[20:23], a[240:243], v[212:215], a[20:23]\n" \
  "s_or_b32 s86, s86, s74\n" \
  "s_bfe_u32 s74, s77, 0x3000a\n" \
  "v_mfma_f32_16x16x32_f16 a[24:27], a[240:243], v[216:219], a[24:27]\n" \
  "s_lshl_b32 s74, s74, 9\n" \
  "s_or_b32 s86, s86, s74\n" \
  "v_mfma_f32_16x16x32_f16 a[28:31], a[240:243], v[220:223], a[28:31]\n" \
  "v_mfma_f32_16x16x32_f16 a[60:63], a[244:247], v[220:223], a[60:63]\n" \
  "v_mfma_f32_16x16x32_f16 a[56:59], a[244:247], v[216:219], a[56:59]\n" \
  "v_mfma_f32_16x16x32_f16 a[52:55], a[244:247], v[212:215], a[52:55]\n" \
  "v_mfma_f32_16x16x32_f16 a[48:51], a[244:247], v[208:211], a[48:51]\n" \
  "v_mfma_f32_16x16x32_f16 a[80:83], a[248:251], v[208:211], a[80:83]\n" \
  "v_mfma_f32_16x16x32_f16 a[84:87], a[248:251], v[212:215], a[84:87]\n" \
  "v_mfma_f32_16x16x32_f16 a[88:91], a[248:251], v[216:219], a[88:91]\n" \
  "v_mfma_f32_16x16x32_f16 a[92:95], a[248:251], v[220:223], a[92:95]\n" \
  "v_mfma_f32_16x16x32_f16 a[124:127], a[252:255], v[220:223], a[124:127]\n" \
  "v_mfma_f32_16x16x32_f16 a[120:123], a[252:255], v[216:219], a[120:123]\n" \
  "v_mfma_f32_16x16x32_f16 a[116:119], a[252:255], v[212:215], a[116:119]\n" \
  "v_mfma_f32_16x16x32_f16 a[112:115], a[252:255], v[208:211], a[112:115]\n" \
  "s_mov_b32 s92, s68\n" \
  "s_mov_b32 s68, s56\n" \
  "s_mov_b32 s69, s57\n" \
  "s_mov_b32 s70, s58\n" \
  "s_waitcnt lgkmcnt(0)\n" \
  "s_load_dwordx8 s[44:51], %6, s70\n" \
  "s_mov_b32 s67, 1\n" \
  ".p2align 6\n" \
  "LOOP_%=:\n" \
  "s_waitcnt vmcnt(39)\n" \
  "v_mfma_f32_16x16x32_f16 v[0:3], a[128:131], v[192:195], v[0:3]\n" \
  "ds_read_b128 v[208:211], v251 offset:4096\n" \
  "v_mfma_f32_16x16x32_f16 v[4:7], a[128:131], v[196:199], v[4:7]\n" \
  "v_mfma_f32_16x16x32_f16 v[8:11], a[128:131], v[200:203], v[8:11]\n" \
  "ds_read_b128 v[212:215], v251 offset:5120\n" \
  "v_mfma_f32_16x16x32_f16 v[12:15], a[128:131], v[204:207], v[12:15]\n" \
  "s_waitcnt vmcnt(38)\n" \
  "v_mfma_f32_16x16x32_f16 v[44:47], a[132:135], v[204:207], v[44:47]\n" \
  "ds_read_b128 v[216:219], v251 offset:6144\n" \
  "v_mfma_f32_16x16x32_f16 v[40:43], a[132:135], v[200:203], v[40:43]\n" \
  "v_mfma_f32_16x16x32_f16 v[36:39], a[132:135], v[196:199], v[36:39]\n" \
  "ds_read_b128 v[220:223], v251 offset:7168\n" \
  "v_mfma_f32_16x16x32_f16 v[32:35], a[132:135], v[192:195], v[32:35]\n" \
  "s_waitcnt vmcnt(37)\n" \
  "v_mfma_f32_16x16x32_f16 v[64:67], a[136:139], v[192:195], v[64:67]\n" \
  "s_waitcnt vmcnt(16)\n" \
  "v_cvt_pk_f16_f32 v224, v160, v164\n" \
  "v_mfma_f32_16x16x32_f16 v[68:71], a[136:139], v[196:199], v[68:71]\n" \
  "v_cvt_pk_f16_f32 v225, v168, v172\n" \
  "v_mfma_f32_16x16x32_f16 v[72:75], a[136:139], v[200:203], v[72:75]\n" \
  "v_cvt_pk_f16_f32 v226, v176, v180\n" \
  "v_mfma_f32_16x16x32_f16 v[76:79], a[136:139], v[204:207], v[76:79]\n" \
  "v_cvt_pk_f16_f32 v227, v184, v188\n" \
  "s_waitcnt vmcnt(36)\n" \
  "v_mfma_f32_16x16x32_f16 v[108:111], a[140:143], v[204:207], v[108:111]\n" \
  "ds_write_b128 v252, v[224:227] offset:0\n" \
  "v_mfma_f32_16x16x32_f16 v[104:107], a[140:143], v[200:203], v[104:107]\n" \
  "v_cvt_pk_f16_f32 v228, v161, v165\n" \
  "v_mfma_f32_16x16x32_f16 v[100:103], a[140:143], v[196:199], v[100:103]\n" \
  "v_cvt_pk_f16_f32 v229, v169, v173\n" \
  "v_mfma_f32_16x16x32_f16 v[96:99], a[140:143], v[192:195], v[96:99]\n" \
  "v_cvt_pk_f16_f32 v230, v177, v181\n" \
  "s_waitcnt vmcnt(35)\n" \
  "v_mfma_f32_16x16x32_f16 a[0:3], a[144:147], v[192:195], a[0:3]\n" \
  "v_cvt_pk_f16_f32 v231, v185, v189\n" \
  "v_mfma_f32_16x16x32_f16 a[4:7], a[144:147], v[196:199], a[4:7]\n" \
  "ds_write_b128 v252, v[228:231] offset:1024\n" \
  "v_mfma_f32_16x16x32_f16 a[8:11], a[144:147], v[200:203], a[8:11]\n" \
  "v_cvt_pk_f16_f32 v232, v162, v166\n" \
  "v_mfma_f32_16x16x32_f16 a[12:15], a[144:147], v[204:207], a[12:15]\n" \
  "v_cvt_pk_f16_f32 v233, v170, v174\n" \
  "s_waitcnt vmcnt(34)\n" \
  "v_mfma_f32_16x16x32_f16 a[44:47], a[148:151], v[204:207], a[44:47]\n" \
  "v_cvt_pk_f16_f32 v234, v178, v182\n" \
  "v_mfma_f32_16x16x32_f16 a[40:43], a[148:151], v[200:203], a[40:43]\n" \
  "v_cvt_pk_f16_f32 v235, v186, v190\n" \
  "v_mfma_f32_16x16x32_f16 a[36:39], a[148:151], v[196:199], a[36:39]\n" \
  "ds_write_b128 v252, v[232:235] offset:2048\n" \
  "v_mfma_f32_16x16x32_f16 a[32:35], a[148:151], v[192:195], a[32:35]\n" \
  "v_cvt_pk_f16_f32 v236, v163, v167\n" \
  "s_waitcnt vmcnt(33)\n" \
  "v_mfma_f32_16x16x32_f16 a[64:67], a[152:155], v[192:195], a[64:67]\n" \
  "v_cvt_pk_f16_f32 v237, v171, v175\n" \
  "v_mfma_f32_16x16x32_f16 a[68:71], a[152:155], v[196:199], a[68:71]\n" \
  "v_cvt_pk_f16_f32 v238, v179, v183\n" \
  "v_mfma_f32_16x16x32_f16 a[72:75], a[152:155], v[200:203], a[72:75]\n" \
  "v_cvt_pk_f16_f32 v239, v187, v191\n" \
  "v_mfma_f32_16x16x32_f16 a[76:79], a[152:155], v[204:207], a[76:79]\n" \
  "ds_write_b128 v252, v[236:239] offset:3072\n" \
  "s_waitcnt vmcnt(32)\n" \
  "v_mfma_f32_16x16x32_f16 a[108:111], a[156:159], v[204:207], a[108:111]\n" \
  "v_mfma_f32_16x16x32_f16 a[104:107], a[156:159], v[200:203], a[104:107]\n" \
  "v_mfma_f32_16x16x32_f16 a[100:103], a[156:159], v[196:199], a[100:103]\n" \
  "v_mfma_f32_16x16x32_f16 a[96:99], a[156:159], v[192:195], a[96:99]\n" \
  "s_waitcnt lgkmcnt(0)\n" \
  "v_mfma_f32_16x16x32_f16 v[16:19], a[128:131], v[208:211], v[16:19]\n" \
  "s_min_u32 s44, s44, s45\n" \
  "s_min_u32 s46, s46, s47\n" \
  "s_min_u32 s48, s48, s49\n" \
  "s_min_u32 s50, s50, s51\n" \
  "v_mfma_f32_16x16x32_f16 v[20:23], a[128:131], v[212:215], v[20:23]\n" \
  "s_min_u32 s44, s44, s46\n" \
  "s_min_u32 s48, s48, s50\n" \
  "s_min_u32 s44, s44, s48\n" \
  "s_cmp_lt_u32 s44, s82\n" \
  "s_cbranch_scc1 SLOW23_%=\n" \
  "OK22_%=:\n" \
  "v_mfma_f32_16x16x32_f16 v[24:27], a[128:131], v[216:219], v[24:27]\n" \
  "s_add_u32 s79, s92, s64\n" \
  "s_add_u32 s80, s79, s84\n" \
  "s_add_u32 s79, s79, s83\n" \
  "buffer_load_dwordx4 a[224:227], v248, s[24:27], s79 offen offset:0 sc1\n" \
  "v_mfma_f32_16x16x32_f16 v[28:31], a[128:131], v[220:223], v[28:31]\n" \
  "ds_read_b128 v[192:195], v251 offset:8192\n" \
  "v_mfma_f32_16x16x32_f16 v[60:63], a[132:135], v[220:223], v[60:63]\n" \
  "v_mfma_f32_16x16x32_f16 v[56:59], a[132:135], v[216:219], v[56:59]\n" \
  "ds_read_b128 v[196:199], v251 offset:9216\n" \
  "v_mfma_f32_16x16x32_f16 v[52:55], a[132:135], v[212:215], v[52:55]\n" \
  "buffer_load_dwordx4 a[228:231], v248, s[24:27], s79 offen offset:1024 sc1\n" \
  "v_mfma_f32_16x16x32_f16 v[48:51], a[132:135], v[208:211], v[48:51]\n" \
  "ds_read_b128 v[200:203], v251 offset:10240\n" \
  "v_mfma_f32_16x16x32_f16 v[80:83], a[136:139], v[208:211], v[80:83]\n" \
  "v_mfma_f32_16x16x32_f16 v[84:87], a[136:139], v[212:215], v[84:87]\n" \
  "ds_read_b128 v[204:207], v251 offset:11264\n" \
  "v_mfma_f32_16x16x32_f16 v[88:91], a[136:139], v[216:219], v[88:91]\n" \
  "buffer_load_dwordx4 a[232:235], v248, s[24:27], s79 offen offset:2048 sc1\n" \
  "v_mfma_f32_16x16x32_f16 v[92:95], a[136:139], v[220:223], v[92:95]\n" \
  "v_mfma_f32_16x16x32_f16 v[124:127], a[140:143], v[220:223], v[124:127]\n" \
  "v_mfma_f32_16x16x32_f16 v[120:123], a[140:143], v[216:219], v[120:123]\n" \
  "v_mfma_f32_16x16x32_f16 v[116:119], a[140:143], v[212:215], v[116:119]\n" \
  "buffer_load_dwordx4 a[236:239], v248, s[24:27], s79 offen offset:3072 sc1\n" \
  "v_mfma_f32_16x16x32_f16 v[112:115], a[140:143], v[208:211], v[112:115]\n" \
  "v_mfma_f32_16x16x32_f16 a[16:19], a[144:147], v[208:211], a[16:19]\n" \
  "v_mfma_f32_16x16x32_f16 a[20:23], a[144:147], v[212:215], a[20:23]\n" \
  "v_mfma_f32_16x16x32_f16 a[24:27], a[144:147], v[216:219], a[24:27]\n" \
  "buffer_load_dwordx4 a[240:243], v248, s[24:27], s80 offen offset:0 sc1\n" \
  "v_mfma_f32_16x16x32_f16 a[28:31], a[144:147], v[220:223], a[28:31]\n" \
  "v_mfma_f32_16x16x32_f16 a[60:63], a[148:151], v[220:223], a[60:63]\n" \
  "v_mfma_f32_16x16x32_f16 a[56:59], a[148:151], v[216:219], a[56:59]\n" \
  "v_mfma_f32_16x16x32_f16 a[52:55], a[148:151], v[212:215], a[52:55]\n" \
  "buffer_load_dwordx4 a[244:247], v248, s[24:27], s80 offen offset:1024 sc1\n" \
  "v_mfma_f32_16x16x32_f16 a[48:51], a[148:151], v[208:211], a[48:51]\n" \
  "v_mfma_f32_16x16x32_f16 a[80:83], a[152:155], v[208:211], a[80:83]\n" \
  "v_mfma_f32_16x16x32_f16 a[84:87], a[152:155], v[212:215], a[84:87]\n" \
  "v_mfma_f32_16x16x32_f16 a[88:91], a[152:155], v[216:219], a[88:91]\n" \
  "buffer_load_dwordx4 a[248:251], v248, s[24:27], s80 offen offset:2048 sc1\n" \
  "v_mfma_f32_16x16x32_f16 a[92:95], a[152:155], v[220:223], a[92:95]\n" \
  "v_mfma_f32_16x16x32_f16 a[124:127], a[156:159], v[220:223], a[124:127]\n" \
  "v_mfma_f32_16x16x32_f16 a[120:123], a[156:159], v[216:219], a[120:123]\n" \
  "v_mfma_f32_16x16x32_f16 a[116:119], a[156:159], v[212:215], a[116:119]\n" \
  "buffer_load_dwordx4 a[252:255], v248, s[24:27], s80 offen offset:3072 sc1\n" \
  "v_mfma_f32_16x16x32_f16 a[112:115], a[156:159], v[208:211], a[112:115]\n" \
  "s_waitcnt lgkmcnt(0)\n" \
  "s_cmp_gt_u32 s67, %11\n" \
  "s_cbranch_scc1 NS31_%=\n" \
  "s_waitcnt vmcnt(8)\n" \
  "v_cvt_pk_f16_f32 v240, v240, v241\n" \
  "v_cvt_pk_f16_f32 v241, v242, v243\n" \
  "v_cvt_pk_f16_f32 v242, v244, v245\n" \
  "v_cvt_pk_f16_f32 v243, v246, v247\n" \
  "buffer_store_dwordx4 v[240:243], v248, s[40:43], s87 offen sc0 sc1\n" \
  "NS31_%=:\n" \
  "s_waitcnt vmcnt(39)\n" \
  "v_mfma_f32_16x16x32_f16 v[0:3], a[160:163], v[192:195], v[0:3]\n" \
  "ds_read_b128 v[208:211], v251 offset:12288\n" \
  "s_add_u32 s79, s68, s61\n" \
  "s_add_u32 s80, s79, s84\n" \
  "s_add_u32 s79, s79, s83\n" \
  "v_mfma_f32_16x16x32_f16 v[4:7], a[160:163], v[196:199], v[4:7]\n" \
  "buffer_load_dwordx4 a[128:131], v248, s[24:27], s79 offen offset:0 sc1\n" \
  "v_mfma_f32_16x16x32_f16 v[8:11], a[160:163], v[200:203], v[8:11]\n" \
  "ds_read_b128 v[212:215], v251 offset:13312\n" \
  "v_mfma_f32_16x16x32_f16 v[12:15], a[160:163], v[204:207], v[12:15]\n" \
  "s_waitcnt vmcnt(39)\n" \
  "v_mfma_f32_16x16x32_f16 v[44:47], a[164:167], v[204:207], v[44:47]\n" \
  "ds_read_b128 v[216:219], v251 offset:14336\n" \
  "v_mfma_f32_16x16x32_f16 v[40:43], a[164:167], v[200:203], v[40:43]\n" \
  "buffer_load_dwordx4 a[132:135], v248, s[24:27], s79 offen offset:1024 sc1\n" \
  "v_mfma_f32_16x16x32_f16 v[36:39], a[164:167], v[196:199], v[36:39]\n" \
  "ds_read_b128 v[220:223], v251 offset:15360\n" \
  "v_mfma_f32_16x16x32_f16 v[32:35], a[164:167], v[192:195], v[32:35]\n" \
  "s_waitcnt vmcnt(39)\n" \
  "v_mfma_f32_16x16x32_f16 v[64:67], a[168:171], v[192:195], v[64:67]\n" \
  "v_mfma_f32_16x16x32_f16 v[68:71], a[168:171], v[196:199], v[68:71]\n" \
  "buffer_load_dwordx4 a[136:139], v248, s[24:27], s79 offen offset:2048 sc1\n" \
  "v_mfma_f32_16x16x32_f16 v[72:75], a[168:171], v[200:203], v[72:75]\n" \
  "v_mfma_f32_16x16x32_f16 v[76:79], a[168:171], v[204:207], v[76:79]\n" \
  "s_waitcnt vmcnt(39)\n" \
  "v_mfma_f32_16x16x32_f16 v[108:111], a[172:175], v[204:207], v[108:111]\n" \
  "v_mfma_f32_16x16x32_f16 v[104:107], a[172:175], v[200:203], v[104:107]\n" \
  "buffer_load_dwordx4 a[140:143], v248, s[24:27], s79 offen offset:3072 sc1\n" \
  "v_mfma_f32_16x16x32_f16 v[100:103], a[172:175], v[196:199], v[100:103]\n" \
  "v_mfma_f32_16x16x32_f16 v[96:99], a[172:175], v[192:195], v[96:99]\n" \
  "s_waitcnt vmcnt(39)\n" \
  "v_mfma_f32_16x16x32_f16 a[0:3], a[176:179], v[192:195], a[0:3]\n" \
  "v_mfma_f32_16x16x32_f16 a[4:7], a[176:179], v[196:199], a[4:7]\n" \
  "buffer_load_dwordx4 a[144:147], v248, s[24:27], s80 offen offset:0 sc1\n" \
  "v_mfma_f32_16x16x32_f16 a[8:11], a[176:179], v[200:203], a[8:11]\n" \
  "v_mfma_f32_16x16x32_f16 a[12:15], a[176:179], v[204:207], a[12:15]\n" \
  "s_lshl_b32 s74, %8, 2\n" \
  "s_add_u32 s74, s74, %9\n" \
  "s_waitcnt vmcnt(39)\n" \
  "v_mfma_f32_16x16x32_f16 a[44:47], a[180:183], v[204:207], a[44:47]\n" \
  "v_mov_b32 v226, s74\n" \
  "v_mov_b32 v227, %9\n" \
  "v_readlane_b32 s75, %1, s67\n" \
  "v_mfma_f32_16x16x32_f16 a[40:43], a[180:183], v[200:203], a[40:43]\n" \
  "s_waitcnt vmcnt(20)\n" \
  "v_cmp_le_u32 vcc, s82, v253\n" \
  "buffer_load_dwordx4 a[148:151], v248, s[24:27], s80 offen offset:1024 sc1\n" \
  "v_mfma_f32_16x16x32_f16 a[36:39], a[180:183], v[196:199], a[36:39]\n" \
  "s_lshr_b32 s75, s75, 24\n" \
  "s_bfm_b64 s[52:53], s75, 0\n" \
  "v_mfma_f32_16x16x32_f16 a[32:35], a[180:183], v[192:195], a[32:35]\n" \
  "s_and_b32 s75, s52, s59\n" \
  "s_and_b32 s74, vcc_lo, s75\n" \
  "s_waitcnt vmcnt(39)\n" \
  "v_mfma_f32_16x16x32_f16 a[64:67], a[184:187], v[192:195], a[64:67]\n" \
  "s_cmp_eq_u32 s74, 0\n" \
  "s_cselect_b32 s76, s75, 0\n" \
  "s_and_b32 s76, s76, s91\n" \
  "s_cbranch_scc1 DSLOW28_%=\n" \
  "DOK27_%=:\n" \
  "v_mfma_f32_16x16x32_f16 a[68:71], a[184:187], v[196:199], a[68:71]\n" \
  "s_ff1_i32_b32 s74, s74\n" \
  "s_add_u32 s75, s67, 2\n" \
  "buffer_load_dwordx4 a[152:155], v248, s[24:27], s80 offen offset:2048 sc1\n" \
  "v_mfma_f32_16x16x32_f16 a[72:75], a[184:187], v[200:203], a[72:75]\n" \
  "s_cmp_eq_u32 s59, 0\n" \
  "s_cselect_b32 s74, s75, s74\n" \
  "v_mfma_f32_16x16x32_f16 a[76:79], a[184:187], v[204:207], a[76:79]\n" \
  "v_mov_b32 v224, s74\n" \
  "ds_write_b32 v226, v224 offset:49152\n" \
  "s_waitcnt vmcnt(39)\n" \
  "v_mfma_f32_16x16x32_f16 a[108:111], a[188:191], v[204:207], a[108:111]\n" \
  "v_mfma_f32_16x16x32_f16 a[104:107], a[188:191], v[200:203], a[104:107]\n" \
  "buffer_load_dwordx4 a[156:159], v248, s[24:27], s80 offen offset:3072 sc1\n" \
  "v_mfma_f32_16x16x32_f16 a[100:103], a[188:191], v[196:199], a[100:103]\n" \
  "v_mfma_f32_16x16x32_f16 a[96:99], a[188:191], v[192:195], a[96:99]\n" \
  "s_waitcnt lgkmcnt(0)\n" \
  "s_barrier\n" \
  "v_mfma_f32_16x16x32_f16 v[16:19], a[160:163], v[208:211], v[16:19]\n" \
  "s_add_u32 s81, s81, 1\n" \
  "s_cmp_eq_u32 s81, 3\n" \
  "s_cselect_b32 s81, 0, s81\n" \
  "s_movk_i32 s74, 0x4000\n" \
  "s_cmov_b32 s74, 0xffff8000\n" \
  "v_mfma_f32_16x16x32_f16 v[20:23], a[160:163], v[212:215], v[20:23]\n" \
  "ds_read_b32 v228, v227 offset:49152\n" \
  "v_add_u32 v251, s74, v251\n" \
  "s_cmp_eq_u32 s81, 2\n" \
  "s_movk_i32 s74, 0x4000\n" \
  "s_cmov_b32 s74, 0xffff8000\n" \
  "v_mfma_f32_16x16x32_f16 v[24:27], a[160:163], v[216:219], v[24:27]\n" \
  "v_add_u32 v252, s74, v252\n" \
  "s_add_u32 s79, s68, s62\n" \
  "s_add_u32 s80, s79, s84\n" \
  "s_add_u32 s79, s79, s83\n" \
  "v_mfma_f32_16x16x32_f16 v[28:31], a[160:163], v[220:223], v[28:31]\n" \
  "ds_read_b128 v[192:195], v251 offset:0\n" \
  "buffer_load_dwordx4 a[160:163], v248, s[24:27], s79 offen offset:0 sc1\n" \
  "v_mfma_f32_16x16x32_f16 v[60:63], a[164:167], v[220:223], v[60:63]\n" \
  "v_mfma_f32_16x16x32_f16 v[56:59], a[164:167], v[216:219], v[56:59]\n" \
  "ds_read_b128 v[196:199], v251 offset:1024\n" \
  "v_mfma_f32_16x16x32_f16 v[52:55], a[164:167], v[212:215], v[52:55]\n" \
  "v_mfma_f32_16x16x32_f16 v[48:51], a[164:167], v[208:211], v[48:51]\n" \
  "ds_read_b128 v[200:203], v251 offset:2048\n" \
  "buffer_load_dwordx4 a[164:167], v248, s[24:27], s79 offen offset:1024 sc1\n" \
  "v_mfma_f32_16x16x32_f16 v[80:83], a[168:171], v[208:211], v[80:83]\n" \
  "v_mfma_f32_16x16x32_f16 v[84:87], a[168:171], v[212:215], v[84:87]\n" \
  "ds_read_b128 v[204:207], v251 offset:3072\n" \
  "v_mfma_f32_16x16x32_f16 v[88:91], a[168:171], v[216:219], v[88:91]\n" \
  "v_mfma_f32_16x16x32_f16 v[92:95], a[168:171], v[220:223], v[92:95]\n" \
  "buffer_load_dwordx4 a[168:171], v248, s[24:27], s79 offen offset:2048 sc1\n" \
  "v_mfma_f32_16x16x32_f16 v[124:127], a[172:175], v[220:223], v[124:127]\n" \
  "v_mfma_f32_16x16x32_f16 v[120:123], a[172:175], v[216:219], v[120:123]\n" \
  "v_mfma_f32_16x16x32_f16 v[116:119], a[172:175], v[212:215], v[116:119]\n" \
  "v_mfma_f32_16x16x32_f16 v[112:115], a[172:175], v[208:211], v[112:115]\n" \
  "buffer_load_dwordx4 a[172:175], v248, s[24:27], s79 offen offset:3072 sc1\n" \
  "v_mfma_f32_16x16x32_f16 a[16:19], a[176:179], v[208:211], a[16:19]\n" \
  "v_mfma_f32_16x16x32_f16 a[20:23], a[176:179], v[212:215], a[20:23]\n" \
  "v_mfma_f32_16x16x32_f16 a[24:27], a[176:179], v[216:219], a[24:27]\n" \
  "v_mfma_f32_16x16x32_f16 a[28:31], a[176:179], v[220:223], a[28:31]\n" \
  "buffer_load_dwordx4 a[176:179], v248, s[24:27], s80 offen offset:0 sc1\n" \
  "v_mfma_f32_16x16x32_f16 a[60:63], a[180:183], v[220:223], a[60:63]\n" \
  "v_mfma_f32_16x16x32_f16 a[56:59], a[180:183], v[216:219], a[56:59]\n" \
  "v_mfma_f32_16x16x32_f16 a[52:55], a[180:183], v[212:215], a[52:55]\n" \
  "v_mfma_f32_16x16x32_f16 a[48:51], a[180:183], v[208:211], a[48:51]\n" \
  "buffer_load_dwordx4 a[180:183], v248, s[24:27], s80 offen offset:1024 sc1\n" \
  "v_mfma_f32_16x16x32_f16 a[80:83], a[184:187], v[208:211], a[80:83]\n" \
  "v_mfma_f32_16x16x32_f16 a[84:87], a[184:187], v[212:215], a[84:87]\n" \
  "v_mfma_f32_16x16x32_f16 a[88:91], a[184:187], v[216:219], a[88:91]\n" \
  "v_mfma_f32_16x16x32_f16 a[92:95], a[184:187], v[220:223], a[92:95]\n" \
  "buffer_load_dwordx4 a[184:187], v248, s[24:27], s80 offen offset:2048 sc1\n" \
  "v_mfma_f32_16x16x32_f16 a[124:127], a[188:191], v[220:223], a[124:127]\n" \
  "v_mfma_f32_16x16x32_f16 a[120:123], a[188:191], v[216:219], a[120:123]\n" \
  "v_mfma_f32_16x16x32_f16 a[116:119], a[188:191], v[212:215], a[116:119]\n" \
  "v_mfma_f32_16x16x32_f16 a[112:115], a[188:191], v[208:211], a[112:115]\n" \
  "buffer_load_dwordx4 a[188:191], v248, s[24:27], s80 offen offset:3072 sc1\n" \
  "s_waitcnt lgkmcnt(0)\n" \
  "v_readfirstlane_b32 s60, v228\n" \
  "s_waitcnt vmcnt(39)\n" \
  "v_mfma_f32_16x16x32_f16 v[0:3], a[192:195], v[192:195], v[0:3]\n" \
  "ds_read_b128 v[208:211], v251 offset:4096\n" \
  "v_mfma_f32_16x16x32_f16 v[4:7], a[192:195], v[196:199], v[4:7]\n" \
  "s_add_u32 s78, s69, s66\n" \
  "s_add_u32 s72, s78, 0\n" \
  "buffer_load_dwordx4 v[160:163], v249, s[28:31], s72 offen\n" \
  "v_mfma_f32_16x16x32_f16 v[8:11], a[192:195], v[200:203], v[8:11]\n" \
  "v_readlane_b32 s56, %1, s60\n" \
  "s_bitset0_b32 s59, s60\n" \
  "ds_read_b128 v[212:215], v251 offset:5120\n" \
  "v_mfma_f32_16x16x32_f16 v[12:15], a[192:195], v[204:207], v[12:15]\n" \
  "s_waitcnt vmcnt(39)\n" \
  "v_mfma_f32_16x16x32_f16 v[44:47], a[196:199], v[204:207], v[44:47]\n" \
  "s_and_b32 s56, s56, 63\n" \
  "ds_read_b128 v[216:219], v251 offset:6144\n" \
  "v_mfma_f32_16x16x32_f16 v[40:43], a[196:199], v[200:203], v[40:43]\n" \
  "s_lshl_b32 s57, s56, 22\n" \
  "s_lshl_b32 s58, s56, 8\n" \
  "s_add_u32 s73, s78, 32768\n" \
  "buffer_load_dwordx4 v[164:167], v249, s[28:31], s73 offen\n" \
  "v_mfma_f32_16x16x32_f16 v[36:39], a[196:199], v[196:199], v[36:39]\n" \
  "s_lshl_b32 s56, s56, 18\n" \
  "s_add_u32 s58, s58, s71\n" \
  "ds_read_b128 v[220:223], v251 offset:7168\n" \
  "v_mfma_f32_16x16x32_f16 v[32:35], a[196:199], v[192:195], v[32:35]\n" \
  "s_waitcnt vmcnt(39)\n" \
  "v_mfma_f32_16x16x32_f16 v[64:67], a[200:203], v[192:195], v[64:67]\n" \
  "s_waitcnt vmcnt(26)\n" \
  "v_cvt_pk_f16_f32 v224, v128, v132\n" \
  "v_mfma_f32_16x16x32_f16 v[68:71], a[200:203], v[196:199], v[68:71]\n" \
  "v_cvt_pk_f16_f32 v225, v136, v140\n" \
  "s_add_u32 s72, s78, 65536\n" \
  "buffer_load_dwordx4 v[168:171], v249, s[28:31], s72 offen\n" \
  "v_mfma_f32_16x16x32_f16 v[72:75], a[200:203], v[200:203], v[72:75]\n" \
  "v_cvt_pk_f16_f32 v226, v144, v148\n" \
  "v_mfma_f32_16x16x32_f16 v[76:79], a[200:203], v[204:207], v[76:79]\n" \
  "v_cvt_pk_f16_f32 v227, v152, v156\n" \
  "s_waitcnt vmcnt(39)\n" \
  "v_mfma_f32_16x16x32_f16 v[108:111], a[204:207], v[204:207], v[108:111]\n" \
  "ds_write_b128 v252, v[224:227] offset:0\n" \
  "v_mfma_f32_16x16x32_f16 v[104:107], a[204:207], v[200:203], v[104:107]\n" \
  "v_cvt_pk_f16_f32 v228, v129, v133\n" \
  "s_add_u32 s73, s78, 98304\n" \
  "buffer_load_dwordx4 v[172:175], v249, s[28:31], s73 offen\n" \
  "v_mfma_f32_16x16x32_f16 v[100:103], a[204:207], v[196:199], v[100:103]\n" \
  "v_cvt_pk_f16_f32 v229, v137, v141\n" \
  "v_mfma_f32_16x16x32_f16 v[96:99], a[204:207], v[192:195], v[96:99]\n" \
  "v_cvt_pk_f16_f32 v230, v145, v149\n" \
  "s_waitcnt vmcnt(39)\n" \
  "v_mfma_f32_16x16x32_f16 a[0:3], a[208:211], v[192:195], a[0:3]\n" \
  "v_cvt_pk_f16_f32 v231, v153, v157\n" \
  "v_mfma_f32_16x16x32_f16 a[4:7], a[208:211], v[196:199], a[4:7]\n" \
  "ds_write_b128 v252, v[228:231] offset:1024\n" \
  "s_add_u32 s72, s78, 131072\n" \
  "buffer_load_dwordx4 v[176:179], v249, s[28:31], s72 offen\n" \
  "v_mfma_f32_16x16x32_f16 a[8:11], a[208:211], v[200:203], a[8:11]\n" \
  "v_cvt_pk_f16_f32 v232, v130, v134\n" \
  "v_mfma_f32_16x16x32_f16 a[12:15], a[208:211], v[204:207], a[12:15]\n" \
  "v_cvt_pk_f16_f32 v233, v138, v142\n" \
  "s_waitcnt vmcnt(39)\n" \
  "v_mfma_f32_16x16x32_f16 a[44:47], a[212:215], v[204:207], a[44:47]\n" \
  "v_cvt_pk_f16_f32 v234, v146, v150\n" \
  "v_mfma_f32_16x16x32_f16 a[40:43], a[212:215], v[200:203], a[40:43]\n" \
  "v_cvt_pk_f16_f32 v235, v154, v158\n" \
  "s_add_u32 s73, s78, 163840\n" \
  "buffer_load_dwordx4 v[180:183], v249, s[28:31], s73 offen\n" \
  "v_mfma_f32_16x16x32_f16 a[36:39], a[212:215], v[196:199], a[36:39]\n" \
  "ds_write_b128 v252, v[232:235] offset:2048\n" \
  "v_mfma_f32_16x16x32_f16 a[32:35], a[212:215], v[192:195], a[32:35]\n" \
  "v_cvt_pk_f16_f32 v236, v131, v135\n" \
  "s_waitcnt vmcnt(39)\n" \
  "v_mfma_f32_16x16x32_f16 a[64:67], a[216:219], v[192:195], a[64:67]\n" \
  "v_cvt_pk_f16_f32 v237, v139, v143\n" \
  "v_mfma_f32_16x16x32_f16 a[68:71], a[216:219], v[196:199], a[68:71]\n" \
  "v_cvt_pk_f16_f32 v238, v147, v151\n" \
  "s_add_u32 s72, s78, 196608\n" \
  "buffer_load_dwordx4 v[184:187], v249, s[28:31], s72 offen\n" \
  "v_mfma_f32_16x16x32_f16 a[72:75], a[216:219], v[200:203], a[72:75]\n" \
  "v_cvt_pk_f16_f32 v239, v155, v159\n" \
  "v_mfma_f32_16x16x32_f16 a[76:79], a[216:219], v[204:207], a[76:79]\n" \
  "ds_write_b128 v252, v[236:239] offset:3072\n" \
  "s_waitcnt vmcnt(39)\n" \
  "v_mfma_f32_16x16x32_f16 a[108:111], a[220:223], v[204:207], a[108:111]\n" \
  "v_mfma_f32_16x16x32_f16 a[104:107], a[220:223], v[200:203], a[104:107]\n" \
  "s_add_u32 s73, s78, 229376\n" \
  "buffer_load_dwordx4 v[188:191], v249, s[28:31], s73 offen\n" \
  "v_mfma_f32_16x16x32_f16 a[100:103], a[220:223], v[196:199], a[100:103]\n" \
  "v_mfma_f32_16x16x32_f16 a[96:99], a[220:223], v[192:195], a[96:99]\n" \
  "s_waitcnt lgkmcnt(0)\n" \
  "v_mfma_f32_16x16x32_f16 v[16:19], a[192:195], v[208:211], v[16:19]\n" \
  "v_mfma_f32_16x16x32_f16 v[20:23], a[192:195], v[212:215], v[20:23]\n" \
  "v_mfma_f32_16x16x32_f16 v[24:27], a[192:195], v[216:219], v[24:27]\n" \
  "s_add_u32 s79, s68, s63\n" \
  "s_add_u32 s80, s79, s84\n" \
  "s_add_u32 s79, s79, s83\n" \
  "v_mfma_f32_16x16x32_f16 v[28:31], a[192:195], v[220:223], v[28:31]\n" \
  "ds_read_b128 v[192:195], v251 offset:8192\n" \
  "buffer_load_dwordx4 a[192:195], v248, s[24:27], s79 offen offset:0 sc1\n" \
  "v_mfma_f32_16x16x32_f16 v[60:63], a[196:199], v[220:223], v[60:63]\n" \
  "v_mfma_f32_16x16x32_f16 v[56:59], a[196:199], v[216:219], v[56:59]\n" \
  "ds_read_b128 v[196:199], v251 offset:9216\n" \
  "v_mfma_f32_16x16x32_f16 v[52:55], a[196:199], v[212:215], v[52:55]\n" \
  "v_mfma_f32_16x16x32_f16 v[48:51], a[196:199], v[208:211], v[48:51]\n" \
  "ds_read_b128 v[200:203], v251 offset:10240\n" \
  "buffer_load_dwordx4 a[196:199], v248, s[24:27], s79 offen offset:1024 sc1\n" \
  "v_mfma_f32_16x16x32_f16 v[80:83], a[200:203], v[208:211], v[80:83]\n" \
  "v_mfma_f32_16x16x32_f16 v[84:87], a[200:203], v[212:215], v[84:87]\n" \
  "ds_read_b128 v[204:207], v251 offset:11264\n" \
  "v_mfma_f32_16x16x32_f16 v[88:91], a[200:203], v[216:219], v[88:91]\n" \
  "v_mfma_f32_16x16x32_f16 v[92:95], a[200:203], v[220:223], v[92:95]\n" \
  "buffer_load_dwordx4 a[200:203], v248, s[24:27], s79 offen offset:2048 sc1\n" \
  "v_mfma_f32_16x16x32_f16 v[124:127], a[204:207], v[220:223], v[124:127]\n" \
  "v_mfma_f32_16x16x32_f16 v[120:123], a[204:207], v[216:219], v[120:123]\n" \
  "v_mfma_f32_16x16x32_f16 v[116:119], a[204:207], v[212:215], v[116:119]\n" \
  "v_mfma_f32_16x16x32_f16 v[112:115], a[204:207], v[208:211], v[112:115]\n" \
  "buffer_load_dwordx4 a[204:207], v248, s[24:27], s79 offen offset:3072 sc1\n" \
  "v_mfma_f32_16x16x32_f16 a[16:19], a[208:211], v[208:211], a[16:19]\n" \
  "v_mfma_f32_16x16x32_f16 a[20:23], a[208:211], v[212:215], a[20:23]\n" \
  "v_mfma_f32_16x16x32_f16 a[24:27], a[208:211], v[216:219], a[24:27]\n" \
  "v_mfma_f32_16x16x32_f16 a[28:31], a[208:211], v[220:223], a[28:31]\n" \
  "buffer_load_dwordx4 a[208:211], v248, s[24:27], s80 offen offset:0 sc1\n" \
  "v_mfma_f32_16x16x32_f16 a[60:63], a[212:215], v[220:223], a[60:63]\n" \
  "v_mfma_f32_16x16x32_f16 a[56:59], a[212:215], v[216:219], a[56:59]\n" \
  "v_mfma_f32_16x16x32_f16 a[52:55], a[212:215], v[212:215], a[52:55]\n" \
  "v_mfma_f32_16x16x32_f16 a[48:51], a[212:215], v[208:211], a[48:51]\n" \
  "buffer_load_dwordx4 a[212:215], v248, s[24:27], s80 offen offset:1024 sc1\n" \
  "v_mfma_f32_16x16x32_f16 a[80:83], a[216:219], v[208:211], a[80:83]\n" \
  "v_mfma_f32_16x16x32_f16 a[84:87], a[216:219], v[212:215], a[84:87]\n" \
  "v_mfma_f32_16x16x32_f16 a[88:91], a[216:219], v[216:219], a[88:91]\n" \
  "v_mfma_f32_16x16x32_f16 a[92:95], a[216:219], v[220:223], a[92:95]\n" \
  "buffer_load_dwordx4 a[216:219], v248, s[24:27], s80 offen offset:2048 sc1\n" \
  "v_mfma_f32_16x16x32_f16 a[124:127], a[220:223], v[220:223], a[124:127]\n" \
  "v_mfma_f32_16x16x32_f16 a[120:123], a[220:223], v[216:219], a[120:123]\n" \
  "v_mfma_f32_16x16x32_f16 a[116:119], a[220:223], v[212:215], a[116:119]\n" \
  "v_mfma_f32_16x16x32_f16 a[112:115], a[220:223], v[208:211], a[112:115]\n" \
  "buffer_load_dwordx4 a[220:223], v248, s[24:27], s80 offen offset:3072 sc1\n" \
  "s_waitcnt lgkmcnt(0)\n" \
  "s_waitcnt vmcnt(39)\n" \
  "v_mfma_f32_16x16x32_f16 v[0:3], a[224:227], v[192:195], v[0:3]\n" \
  "ds_read_b128 v[208:211], v251 offset:12288\n" \
  "s_add_u32 s78, s57, s65\n" \
  "s_add_u32 s72, s78, 0\n" \
  "buffer_load_dwordx4 v[128:131], v249, s[28:31], s72 offen\n" \
  "v_mfma_f32_16x16x32_f16 v[4:7], a[224:227], v[196:199], v[4:7]\n" \
  "v_bfe_u32 v232, %1, 8, 16\n" \
  "s_and_b32 exec_lo, s59, s91\n" \
  "s_cbranch_scc0 PL35_%=\n" \
  "s_mov_b32 exec_hi, 0\n" \
  "buffer_load_dword v253, v232, s[36:39], s90 offen sc1\n" \
  "PL35_%=:\n" \
  "s_mov_b64 exec, -1\n" \
  "v_mfma_f32_16x16x32_f16 v[8:11], a[224:227], v[200:203], v[8:11]\n" \
  "ds_read_b128 v[212:215], v251 offset:13312\n" \
  "v_mfma_f32_16x16x32_f16 v[12:15], a[224:227], v[204:207], v[12:15]\n" \
  "s_waitcnt vmcnt(39)\n" \
  "v_mfma_f32_16x16x32_f16 v[44:47], a[228:231], v[204:207], v[44:47]\n" \
  "ds_read_b128 v[216:219], v251 offset:14336\n" \
  "s_add_u32 s73, s78, 32768\n" \
  "buffer_load_dwordx4 v[132:135], v249, s[28:31], s73 offen\n" \
  "v_mfma_f32_16x16x32_f16 v[40:43], a[228:231], v[200:203], v[40:43]\n" \
  "v_mfma_f32_16x16x32_f16 v[36:39], a[228:231], v[196:199], v[36:39]\n" \
  "ds_read_b128 v[220:223], v251 offset:15360\n" \
  "v_mfma_f32_16x16x32_f16 v[32:35], a[228:231], v[192:195], v[32:35]\n" \
  "s_waitcnt vmcnt(39)\n" \
  "v_mfma_f32_16x16x32_f16 v[64:67], a[232:235], v[192:195], v[64:67]\n" \
  "s_add_u32 s72, s78, 65536\n" \
  "buffer_load_dwordx4 v[136:139], v249, s[28:31], s72 offen\n" \
  "v_mfma_f32_16x16x32_f16 v[68:71], a[232:235], v[196:199], v[68:71]\n" \
  "v_mfma_f32_16x16x32_f16 v[72:75], a[232:235], v[200:203], v[72:75]\n" \
  "v_mfma_f32_16x16x32_f16 v[76:79], a[232:235], v[204:207], v[76:79]\n" \
  "s_waitcnt vmcnt(39)\n" \
  "v_mfma_f32_16x16x32_f16 v[108:111], a[236:239], v[204:207], v[108:111]\n" \
  "s_add_u32 s73, s78, 98304\n" \
  "buffer_load_dwordx4 v[140:143], v249, s[28:31], s73 offen\n" \
  "v_mfma_f32_16x16x32_f16 v[104:107], a[236:239], v[200:203], v[104:107]\n" \
  "v_mfma_f32_16x16x32_f16 v[100:103], a[236:239], v[196:199], v[100:103]\n" \
  "v_mfma_f32_16x16x32_f16 v[96:99], a[236:239], v[192:195], v[96:99]\n" \
  "s_waitcnt vmcnt(39)\n" \
  "v_mfma_f32_16x16x32_f16 a[0:3], a[240:243], v[192:195], a[0:3]\n" \
  "s_add_u32 s72, s78, 131072\n" \
  "buffer_load_dwordx4 v[144:147], v249, s[28:31], s72 offen\n" \
  "v_mfma_f32_16x16x32_f16 a[4:7], a[240:243], v[196:199], a[4:7]\n" \
  "v_mfma_f32_16x16x32_f16 a[8:11], a[240:243], v[200:203], a[8:11]\n" \
  "v_mfma_f32_16x16x32_f16 a[12:15], a[240:243], v[204:207], a[12:15]\n" \
  "s_waitcnt vmcnt(39)\n" \
  "v_mfma_f32_16x16x32_f16 a[44:47], a[244:247], v[204:207], a[44:47]\n" \
  "s_add_u32 s73, s78, 163840\n" \
  "buffer_load_dwordx4 v[148:151], v249, s[28:31], s73 offen\n" \
  "v_mfma_f32_16x16x32_f16 a[40:43], a[244:247], v[200:203], a[40:43]\n" \
  "v_mfma_f32_16x16x32_f16 a[36:39], a[244:247], v[196:199], a[36:39]\n" \
  "v_mfma_f32_16x16x32_f16 a[32:35], a[244:247], v[192:195], a[32:35]\n" \
  "s_waitcnt vmcnt(39)\n" \
  "v_mfma_f32_16x16x32_f16 a[64:67], a[248:251], v[192:195], a[64:67]\n" \
  "s_add_u32 s72, s78, 196608\n" \
  "buffer_load_dwordx4 v[152:155], v249, s[28:31], s72 offen\n" \
  "v_mfma_f32_16x16x32_f16 a[68:71], a[248:251], v[196:199], a[68:71]\n" \
  "v_mfma_f32_16x16x32_f16 a[72:75], a[248:251], v[200:203], a[72:75]\n" \
  "v_mfma_f32_16x16x32_f16 a[76:79], a[248:251], v[204:207], a[76:79]\n" \
  "s_waitcnt vmcnt(39)\n" \
  "v_mfma_f32_16x16x32_f16 a[108:111], a[252:255], v[204:207], a[108:111]\n" \
  "s_add_u32 s73, s78, 229376\n" \
  "buffer_load_dwordx4 v[156:159], v249, s[28:31], s73 offen\n" \
  "v_mfma_f32_16x16x32_f16 a[104:107], a[252:255], v[200:203], a[104:107]\n" \
  "v_mfma_f32_16x16x32_f16 a[100:103], a[252:255], v[196:199], a[100:103]\n" \
  "v_mfma_f32_16x16x32_f16 a[96:99], a[252:255], v[192:195], a[96:99]\n" \
  "s_waitcnt vmcnt(40)\n" \
  "s_waitcnt lgkmcnt(0)\n" \
  "s_barrier\n" \
  "v_mfma_f32_16x16x32_f16 v[16:19], a[224:227], v[208:211], v[16:19]\n" \
  "s_add_u32 s81, s81, 1\n" \
  "s_cmp_eq_u32 s81, 3\n" \
  "s_cselect_b32 s81, 0, s81\n" \
  "s_movk_i32 s74, 0x4000\n" \
  "s_cmov_b32 s74, 0xffff8000\n" \
  "s_cmp_ge_u32 s67, %11\n" \
  "s_cbranch_scc1 NX32_%=\n" \
  "buffer_load_dwordx4 v[240:243], v250, s[32:35], s86 offen nt\n" \
  "buffer_load_dwordx4 v[244:247], v250, s[32:35], s86 offen offset:16 nt\n" \
  "NX32_%=:\n" \
  "v_mfma_f32_16x16x32_f16 v[20:23], a[224:227], v[212:215], v[20:23]\n" \
  "v_add_u32 v251, s74, v251\n" \
  "s_cmp_eq_u32 s81, 2\n" \
  "s_movk_i32 s74, 0x4000\n" \
  "s_cmov_b32 s74, 0xffff8000\n" \
  "v_mfma_f32_16x16x32_f16 v[24:27], a[224:227], v[216:219], v[24:27]\n" \
  "v_add_u32 v252, s74, v252\n" \
  "v_mfma_f32_16x16x32_f16 v[28:31], a[224:227], v[220:223], v[28:31]\n" \
  "ds_read_b128 v[192:195], v251 offset:0\n" \
  "v_mfma_f32_16x16x32_f16 v[60:63], a[228:231], v[220:223], v[60:63]\n" \
  "s_cmp_gt_u32 s67, %11\n" \
  "s_cbranch_scc1 NA33_%=\n" \
  "v_mov_b32 v224, s82\n" \
  "s_cmp_lg_u32 %8, 0\n" \
  "s_cbranch_scc1 W034_%=\n" \
  "s_mov_b64 exec, 1\n" \
  "buffer_store_dword v224, off, s[40:43], s88 sc0 sc1\n" \
  "s_mov_b64 exec, -1\n" \
  "W034_%=:\n" \
  "NA33_%=:\n" \
  "v_mfma_f32_16x16x32_f16 v[56:59], a[228:231], v[216:219], v[56:59]\n" \
  "s_add_u32 s74, s67, 1\n" \
  "s_add_u32 s75, s67, 2\n" \
  "ds_read_b128 v[196:199], v251 offset:1024\n" \
  "v_mfma_f32_16x16x32_f16 v[52:55], a[228:231], v[212:215], v[52:55]\n" \
  "v_readlane_b32 s76, %0, s74\n" \
  "v_readlane_b32 s77, %0, s75\n" \
  "v_mfma_f32_16x16x32_f16 v[48:51], a[228:231], v[208:211], v[48:51]\n" \
  "ds_read_b128 v[200:203], v251 offset:2048\n" \
  "v_mfma_f32_16x16x32_f16 v[80:83], a[232:235], v[208:211], v[80:83]\n" \
  "s_and_b32 s74, s76, 7\n" \
  "s_lshl_b32 s74, s74, 3\n" \
  "v_mfma_f32_16x16x32_f16 v[84:87], a[232:235], v[212:215], v[84:87]\n" \
  "s_load_dwordx2 s[40:41], %5, s74\n" \
  "s_bfe_u32 s87, s76, 0x6000a\n" \
  "ds_read_b128 v[204:207], v251 offset:3072\n" \
  "v_mfma_f32_16x16x32_f16 v[88:91], a[232:235], v[216:219], v[88:91]\n" \
  "s_lshl_b32 s87, s87, 18\n" \
  "s_lshl_b32 s74, %8, 16\n" \
  "v_mfma_f32_16x16x32_f16 v[92:95], a[232:235], v[220:223], v[92:95]\n" \
  "s_or_b32 s87, s87, s74\n" \
  "s_bfe_u32 s74, s76, 0x60004\n" \
  "v_mfma_f32_16x16x32_f16 v[124:127], a[236:239], v[220:223], v[124:127]\n" \
  "s_lshl_b32 s74, s74, 10\n" \
  "s_or_b32 s87, s87, s74\n" \
  "v_mfma_f32_16x16x32_f16 v[120:123], a[236:239], v[216:219], v[120:123]\n" \
  "s_add_u32 s87, s87, s85\n" \
  "s_bfe_u32 s88, s76, 0xc0004\n" \
  "v_mfma_f32_16x16x32_f16 v[116:119], a[236:239], v[212:215], v[116:119]\n" \
  "s_lshl_b32 s88, s88, 2\n" \
  "s_add_u32 s88, s88, 67108864\n" \
  "v_mfma_f32_16x16x32_f16 v[112:115], a[236:239], v[208:211], v[112:115]\n" \
  "s_and_b32 s86, s77, 7\n" \
  "s_lshl_b32 s86, s86, 22\n" \
  "v_mfma_f32_16x16x32_f16 a[16:19], a[240:243], v[208:211], a[16:19]\n" \
  "s_bfe_u32 s74, s77, 0x60004\n" \
  "s_lshl_b32 s74, s74, 16\n" \
  "v_mfma_f32_16x16x32_f16 a[20:23], a[240:243], v[212:215], a[20:23]\n" \
  "s_or_b32 s86, s86, s74\n" \
  "s_bfe_u32 s74, s77, 0x3000a\n" \
  "v_mfma_f32_16x16x32_f16 a[24:27], a[240:243], v[216:219], a[24:27]\n" \
  "s_lshl_b32 s74, s74, 9\n" \
  "s_or_b32 s86, s86, s74\n" \
  "v_mfma_f32_16x16x32_f16 a[28:31], a[240:243], v[220:223], a[28:31]\n" \
  "v_mfma_f32_16x16x32_f16 a[60:63], a[244:247], v[220:223], a[60:63]\n" \
  "v_mfma_f32_16x16x32_f16 a[56:59], a[244:247], v[216:219], a[56:59]\n" \
  "v_mfma_f32_16x16x32_f16 a[52:55], a[244:247], v[212:215], a[52:55]\n" \
  "v_mfma_f32_16x16x32_f16 a[48:51], a[244:247], v[208:211], a[48:51]\n" \
  "v_mfma_f32_16x16x32_f16 a[80:83], a[248:251], v[208:211], a[80:83]\n" \
  "v_mfma_f32_16x16x32_f16 a[84:87], a[248:251], v[212:215], a[84:87]\n" \
  "v_mfma_f32_16x16x32_f16 a[88:91], a[248:251], v[216:219], a[88:91]\n" \
  "v_mfma_f32_16x16x32_f16 a[92:95], a[248:251], v[220:223], a[92:95]\n" \
  "v_mfma_f32_16x16x32_f16 a[124:127], a[252:255], v[220:223], a[124:127]\n" \
  "v_mfma_f32_16x16x32_f16 a[120:123], a[252:255], v[216:219], a[120:123]\n" \
  "v_mfma_f32_16x16x32_f16 a[116:119], a[252:255], v[212:215], a[116:119]\n" \
  "v_mfma_f32_16x16x32_f16 a[112:115], a[252:255], v[208:211], a[112:115]\n" \
  "s_mov_b32 s92, s68\n" \
  "s_mov_b32 s68, s56\n" \
  "s_mov_b32 s69, s57\n" \
  "s_mov_b32 s70, s58\n" \
  "s_waitcnt lgkmcnt(0)\n" \
  "s_load_dwordx8 s[44:51], %6, s70\n" \
  "s_add_u32 s67, s67, 1\n" \
  "s_cmp_lt_u32 s67, 30\n" \
  "s_cbranch_scc1 LOOP_%=\n" \
  "s_waitcnt vmcnt(39)\n" \
  "v_mfma_f32_16x16x32_f16 v[0:3], a[128:131], v[192:195], v[0:3]\n" \
  "ds_read_b128 v[208:211], v251 offset:4096\n" \
  "v_mfma_f32_16x16x32_f16 v[4:7], a[128:131], v[196:199], v[4:7]\n" \
  "v_mfma_f32_16x16x32_f16 v[8:11], a[128:131], v[200:203], v[8:11]\n" \
  "ds_read_b128 v[212:215], v251 offset:5120\n" \
  "v_mfma_f32_16x16x32_f16 v[12:15], a[128:131], v[204:207], v[12:15]\n" \
  "s_waitcnt vmcnt(38)\n" \
  "v_mfma_f32_16x16x32_f16 v[44:47], a[132:135], v[204:207], v[44:47]\n" \
  "ds_read_b128 v[216:219], v251 offset:6144\n" \
  "v_mfma_f32_16x16x32_f16 v[40:43], a[132:135], v[200:203], v[40:43]\n" \
  "v_mfma_f32_16x16x32_f16 v[36:39], a[132:135], v[196:199], v[36:39]\n" \
  "ds_read_b128 v[220:223], v251 offset:7168\n" \
  "v_mfma_f32_16x16x32_f16 v[32:35], a[132:135], v[192:195], v[32:35]\n" \
  "s_waitcnt vmcnt(37)\n" \
  "v_mfma_f32_16x16x32_f16 v[64:67], a[136:139], v[192:195], v[64:67]\n" \
  "s_waitcnt vmcnt(16)\n" \
  "v_cvt_pk_f16_f32 v224, v160, v164\n" \
  "v_mfma_f32_16x16x32_f16 v[68:71], a[136:139], v[196:199], v[68:71]\n" \
  "v_cvt_pk_f16_f32 v225, v168, v172\n" \
  "v_mfma_f32_16x16x32_f16 v[72:75], a[136:139], v[200:203], v[72:75]\n" \
  "v_cvt_pk_f16_f32 v226, v176, v180\n" \
  "v_mfma_f32_16x16x32_f16 v[76:79], a[136:139], v[204:207], v[76:79]\n" \
  "v_cvt_pk_f16_f32 v227, v184, v188\n" \
  "s_waitcnt vmcnt(36)\n" \
  "v_mfma_f32_16x16x32_f16 v[108:111], a[140:143], v[204:207], v[108:111]\n" \
  "ds_write_b128 v252, v[224:227] offset:0\n" \
  "v_mfma_f32_16x16x32_f16 v[104:107], a[140:143], v[200:203], v[104:107]\n" \
  "v_cvt_pk_f16_f32 v228, v161, v165\n" \
  "v_mfma_f32_16x16x32_f16 v[100:103], a[140:143], v[196:199], v[100:103]\n" \
  "v_cvt_pk_f16_f32 v229, v169, v173\n" \
  "v_mfma_f32_16x16x32_f16 v[96:99], a[140:143], v[192:195], v[96:99]\n" \
  "v_cvt_pk_f16_f32 v230, v177, v181\n" \
  "s_waitcnt vmcnt(35)\n" \
  "v_mfma_f32_16x16x32_f16 a[0:3], a[144:147], v[192:195], a[0:3]\n" \
  "v_cvt_pk_f16_f32 v231, v185, v189\n" \
  "v_mfma_f32_16x16x32_f16 a[4:7], a[144:147], v[196:199], a[4:7]\n" \
  "ds_write_b128 v252, v[228:231] offset:1024\n" \
  "v_mfma_f32_16x16x32_f16 a[8:11], a[144:147], v[200:203], a[8:11]\n" \
  "v_cvt_pk_f16_f32 v232, v162, v166\n" \
  "v_mfma_f32_16x16x32_f16 a[12:15], a[144:147], v[204:207], a[12:15]\n" \
  "v_cvt_pk_f16_f32 v233, v170, v174\n" \
  "s_waitcnt vmcnt(34)\n" \
  "v_mfma_f32_16x16x32_f16 a[44:47], a[148:151], v[204:207], a[44:47]\n" \
  "v_cvt_pk_f16_f32 v234, v178, v182\n" \
  "v_mfma_f32_16x16x32_f16 a[40:43], a[148:151], v[200:203], a[40:43]\n" \
  "v_cvt_pk_f16_f32 v235, v186, v190\n" \
  "v_mfma_f32_16x16x32_f16 a[36:39], a[148:151], v[196:199], a[36:39]\n" \
  "ds_write_b128 v252, v[232:235] offset:2048\n" \
  "v_mfma_f32_16x16x32_f16 a[32:35], a[148:151], v[192:195], a[32:35]\n" \
  "v_cvt_pk_f16_f32 v236, v163, v167\n" \
  "s_waitcnt vmcnt(33)\n" \
  "v_mfma_f32_16x16x32_f16 a[64:67], a[152:155], v[192:195], a[64:67]\n" \
  "v_cvt_pk_f16_f32 v237, v171, v175\n" \
  "v_mfma_f32_16x16x32_f16 a[68:71], a[152:155], v[196:199], a[68:71]\n" \
  "v_cvt_pk_f16_f32 v238, v179, v183\n" \
  "v_mfma_f32_16x16x32_f16 a[72:75], a[152:155], v[200:203], a[72:75]\n" \
  "v_cvt_pk_f16_f32 v239, v187, v191\n" \
  "v_mfma_f32_16x16x32_f16 a[76:79], a[152:155], v[204:207], a[76:79]\n" \
  "ds_write_b128 v252, v[236:239] offset:3072\n" \
  "s_waitcnt vmcnt(32)\n" \
  "v_mfma_f32_16x16x32_f16 a[108:111], a[156:159], v[204:207], a[108:111]\n" \
  "v_mfma_f32_16x16x32_f16 a[104:107], a[156:159], v[200:203], a[104:107]\n" \
  "v_mfma_f32_16x16x32_f16 a[100:103], a[156:159], v[196:199], a[100:103]\n" \
  "v_mfma_f32_16x16x32_f16 a[96:99], a[156:159], v[192:195], a[96:99]\n" \
  "s_waitcnt lgkmcnt(0)\n" \
  "v_mfma_f32_16x16x32_f16 v[16:19], a[128:131], v[208:211], v[16:19]\n" \
  "s_min_u32 s44, s44, s45\n" \
  "s_min_u32 s46, s46, s47\n" \
  "s_min_u32 s48, s48, s49\n" \
  "s_min_u32 s50, s50, s51\n" \
  "v_mfma_f32_16x16x32_f16 v[20:23], a[128:131], v[212:215], v[20:23]\n" \
  "s_min_u32 s44, s44, s46\n" \
  "s_min_u32 s48, s48, s50\n" \
  "s_min_u32 s44, s44, s48\n" \
  "s_cmp_lt_u32 s44, s82\n" \
  "s_cbranch_scc1 SLOW37_%=\n" \
  "OK36_%=:\n" \
  "v_mfma_f32_16x16x32_f16 v[24:27], a[128:131], v[216:219], v[24:27]\n" \
  "s_add_u32 s79, s92, s64\n" \
  "s_add_u32 s80, s79, s84\n" \
  "s_add_u32 s79, s79, s83\n" \
  "buffer_load_dwordx4 a[224:227], v248, s[24:27], s79 offen offset:0 sc1\n" \
  "v_mfma_f32_16x16x32_f16 v[28:31], a[128:131], v[220:223], v[28:31]\n" \
  "ds_read_b128 v[192:195], v251 offset:8192\n" \
  "v_mfma_f32_16x16x32_f16 v[60:63], a[132:135], v[220:223], v[60:63]\n" \
  "v_mfma_f32_16x16x32_f16 v[56:59], a[132:135], v[216:219], v[56:59]\n" \
  "ds_read_b128 v[196:199], v251 offset:9216\n" \
  "v_mfma_f32_16x16x32_f16 v[52:55], a[132:135], v[212:215], v[52:55]\n" \
  "buffer_load_dwordx4 a[228:231], v248, s[24:27], s79 offen offset:1024 sc1\n" \
  "v_mfma_f32_16x16x32_f16 v[48:51], a[132:135], v[208:211], v[48:51]\n" \
  "ds_read_b128 v[200:203], v251 offset:10240\n" \
  "v_mfma_f32_16x16x32_f16 v[80:83], a[136:139], v[208:211], v[80:83]\n" \
  "v_mfma_f32_16x16x32_f16 v[84:87], a[136:139], v[212:215], v[84:87]\n" \
  "ds_read_b128 v[204:207], v251 offset:11264\n" \
  "v_mfma_f32_16x16x32_f16 v[88:91], a[136:139], v[216:219], v[88:91]\n" \
  "buffer_load_dwordx4 a[232:235], v248, s[24:27], s79 offen offset:2048 sc1\n" \
  "v_mfma_f32_16x16x32_f16 v[92:95], a[136:139], v[220:223], v[92:95]\n" \
  "v_mfma_f32_16x16x32_f16 v[124:127], a[140:143], v[220:223], v[124:127]\n" \
  "v_mfma_f32_16x16x32_f16 v[120:123], a[140:143], v[216:219], v[120:123]\n" \
  "v_mfma_f32_16x16x32_f16 v[116:119], a[140:143], v[212:215], v[116:119]\n" \
  "buffer_load_dwordx4 a[236:239], v248, s[24:27], s79 offen offset:3072 sc1\n" \
  "v_mfma_f32_16x16x32_f16 v[112:115], a[140:143], v[208:211], v[112:115]\n" \
  "v_mfma_f32_16x16x32_f16 a[16:19], a[144:147], v[208:211], a[16:19]\n" \
  "v_mfma_f32_16x16x32_f16 a[20:23], a[144:147], v[212:215], a[20:23]\n" \
  "v_mfma_f32_16x16x32_f16 a[24:27], a[144:147], v[216:219], a[24:27]\n" \
  "buffer_load_dwordx4 a[240:243], v248, s[24:27], s80 offen offset:0 sc1\n" \
  "v_mfma_f32_16x16x32_f16 a[28:31], a[144:147], v[220:223], a[28:31]\n" \
  "v_mfma_f32_16x16x32_f16 a[60:63], a[148:151], v[220:223], a[60:63]\n" \
  "v_mfma_f32_16x16x32_f16 a[56:59], a[148:151], v[216:219], a[56:59]\n" \
  "v_mfma_f32_16x16x32_f16 a[52:55], a[148:151], v[212:215], a[52:55]\n" \
  "buffer_load_dwordx4 a[244:247], v248, s[24:27], s80 offen offset:1024 sc1\n" \
  "v_mfma_f32_16x16x32_f16 a[48:51], a[148:151], v[208:211], a[48:51]\n" \
  "v_mfma_f32_16x16x32_f16 a[80:83], a[152:155], v[208:211], a[80:83]\n" \
  "v_mfma_f32_16x16x32_f16 a[84:87], a[152:155], v[212:215], a[84:87]\n" \
  "v_mfma_f32_16x16x32_f16 a[88:91], a[152:155], v[216:219], a[88:91]\n" \
  "buffer_load_dwordx4 a[248:251], v248, s[24:27], s80 offen offset:2048 sc1\n" \
  "v_mfma_f32_16x16x32_f16 a[92:95], a[152:155], v[220:223], a[92:95]\n" \
  "v_mfma_f32_16x16x32_f16 a[124:127], a[156:159], v[220:223], a[124:127]\n" \
  "v_mfma_f32_16x16x32_f16 a[120:123], a[156:159], v[216:219], a[120:123]\n" \
  "v_mfma_f32_16x16x32_f16 a[116:119], a[156:159], v[212:215], a[116:119]\n" \
  "buffer_load_dwordx4 a[252:255], v248, s[24:27], s80 offen offset:3072 sc1\n" \
  "v_mfma_f32_16x16x32_f16 a[112:115], a[156:159], v[208:211], a[112:115]\n" \
  "s_waitcnt lgkmcnt(0)\n" \
  "s_waitcnt vmcnt(39)\n" \
  "v_mfma_f32_16x16x32_f16 v[0:3], a[160:163], v[192:195], v[0:3]\n" \
  "ds_read_b128 v[208:211], v251 offset:12288\n" \
  "s_add_u32 s79, s68, s61\n" \
  "s_add_u32 s80, s79, s84\n" \
  "s_add_u32 s79, s79, s83\n" \
  "v_mfma_f32_16x16x32_f16 v[4:7], a[160:163], v[196:199], v[4:7]\n" \
  "buffer_load_dwordx4 a[128:131], v248, s[24:27], s79 offen offset:0 sc1\n" \
  "v_mfma_f32_16x16x32_f16 v[8:11], a[160:163], v[200:203], v[8:11]\n" \
  "ds_read_b128 v[212:215], v251 offset:13312\n" \
  "v_mfma_f32_16x16x32_f16 v[12:15], a[160:163], v[204:207], v[12:15]\n" \
  "s_waitcnt vmcnt(39)\n" \
  "v_mfma_f32_16x16x32_f16 v[44:47], a[164:167], v[204:207], v[44:47]\n" \
  "ds_read_b128 v[216:219], v251 offset:14336\n" \
  "v_mfma_f32_16x16x32_f16 v[40:43], a[164:167], v[200:203], v[40:43]\n" \
  "buffer_load_dwordx4 a[132:135], v248, s[24:27], s79 offen offset:1024 sc1\n" \
  "v_mfma_f32_16x16x32_f16 v[36:39], a[164:167], v[196:199], v[36:39]\n" \
  "ds_read_b128 v[220:223], v251 offset:15360\n" \
  "v_mfma_f32_16x16x32_f16 v[32:35], a[164:167], v[192:195], v[32:35]\n" \
  "s_waitcnt vmcnt(39)\n" \
  "v_mfma_f32_16x16x32_f16 v[64:67], a[168:171], v[192:195], v[64:67]\n" \
  "v_mfma_f32_16x16x32_f16 v[68:71], a[168:171], v[196:199], v[68:71]\n" \
  "buffer_load_dwordx4 a[136:139], v248, s[24:27], s79 offen offset:2048 sc1\n" \
  "v_mfma_f32_16x16x32_f16 v[72:75], a[168:171], v[200:203], v[72:75]\n" \
  "v_mfma_f32_16x16x32_f16 v[76:79], a[168:171], v[204:207], v[76:79]\n" \
  "s_waitcnt vmcnt(39)\n" \
  "v_mfma_f32_16x16x32_f16 v[108:111], a[172:175], v[204:207], v[108:111]\n" \
  "v_mfma_f32_16x16x32_f16 v[104:107], a[172:175], v[200:203], v[104:107]\n" \
  "buffer_load_dwordx4 a[140:143], v248, s[24:27], s79 offen offset:3072 sc1\n" \
  "v_mfma_f32_16x16x32_f16 v[100:103], a[172:175], v[196:199], v[100:103]\n" \
  "v_mfma_f32_16x16x32_f16 v[96:99], a[172:175], v[192:195], v[96:99]\n" \
  "s_waitcnt vmcnt(39)\n" \
  "v_mfma_f32_16x16x32_f16 a[0:3], a[176:179], v[192:195], a[0:3]\n" \
  "v_mfma_f32_16x16x32_f16 a[4:7], a[176:179], v[196:199], a[4:7]\n" \
  "v_mfma_f32_16x16x32_f16 a[8:11], a[176:179], v[200:203], a[8:11]\n" \
  "v_mfma_f32_16x16x32_f16 a[12:15], a[176:179], v[204:207], a[12:15]\n" \
  "s_lshl_b32 s74, %8, 2\n" \
  "s_add_u32 s74, s74, %9\n" \
  "s_waitcnt vmcnt(38)\n" \
  "v_mfma_f32_16x16x32_f16 a[44:47], a[180:183], v[204:207], a[44:47]\n" \
  "v_mov_b32 v226, s74\n" \
  "v_mov_b32 v227, %9\n" \
  "v_mfma_f32_16x16x32_f16 a[40:43], a[180:183], v[200:203], a[40:43]\n" \
  "v_mfma_f32_16x16x32_f16 a[36:39], a[180:183], v[196:199], a[36:39]\n" \
  "v_mfma_f32_16x16x32_f16 a[32:35], a[180:183], v[192:195], a[32:35]\n" \
  "s_waitcnt vmcnt(37)\n" \
  "v_mfma_f32_16x16x32_f16 a[64:67], a[184:187], v[192:195], a[64:67]\n" \
  "v_mfma_f32_16x16x32_f16 a[68:71], a[184:187], v[196:199], a[68:71]\n" \
  "s_add_u32 s74, s67, 2\n" \
  "v_mfma_f32_16x16x32_f16 a[72:75], a[184:187], v[200:203], a[72:75]\n" \
  "v_mfma_f32_16x16x32_f16 a[76:79], a[184:187], v[204:207], a[76:79]\n" \
  "v_mov_b32 v224, s74\n" \
  "ds_write_b32 v226, v224 offset:49152\n" \
  "s_waitcnt vmcnt(36)\n" \
  "v_mfma_f32_16x16x32_f16 a[108:111], a[188:191], v[204:207], a[108:111]\n" \
  "v_mfma_f32_16x16x32_f16 a[104:107], a[188:191], v[200:203], a[104:107]\n" \
  "v_mfma_f32_16x16x32_f16 a[100:103], a[188:191], v[196:199], a[100:103]\n" \
  "v_mfma_f32_16x16x32_f16 a[96:99], a[188:191], v[192:195], a[96:99]\n" \
  "s_waitcnt lgkmcnt(0)\n" \
  "s_barrier\n" \
  "v_mfma_f32_16x16x32_f16 v[16:19], a[160:163], v[208:211], v[16:19]\n" \
  "s_add_u32 s81, s81, 1\n" \
  "s_cmp_eq_u32 s81, 3\n" \
  "s_cselect_b32 s81, 0, s81\n" \
  "s_movk_i32 s74, 0x4000\n" \
  "s_cmov_b32 s74, 0xffff8000\n" \
  "v_mfma_f32_16x16x32_f16 v[20:23], a[160:163], v[212:215], v[20:23]\n" \
  "ds_read_b32 v228, v227 offset:49152\n" \
  "v_add_u32 v251, s74, v251\n" \
  "s_cmp_eq_u32 s81, 2\n" \
  "s_movk_i32 s74, 0x4000\n" \
  "s_cmov_b32 s74, 0xffff8000\n" \
  "v_mfma_f32_16x16x32_f16 v[24:27], a[160:163], v[216:219], v[24:27]\n" \
  "v_add_u32 v252, s74, v252\n" \
  "s_add_u32 s79, s68, s62\n" \
  "s_add_u32 s80, s79, s84\n" \
  "s_add_u32 s79, s79, s83\n" \
  "v_mfma_f32_16x16x32_f16 v[28:31], a[160:163], v[220:223], v[28:31]\n" \
  "ds_read_b128 v[192:195], v251 offset:0\n" \
  "buffer_load_dwordx4 a[160:163], v248, s[24:27], s79 offen offset:0 sc1\n" \
  "v_mfma_f32_16x16x32_f16 v[60:63], a[164:167], v[220:223], v[60:63]\n" \
  "v_mfma_f32_16x16x32_f16 v[56:59], a[164:167], v[216:219], v[56:59]\n" \
  "ds_read_b128 v[196:199], v251 offset:1024\n" \
  "v_mfma_f32_16x16x32_f16 v[52:55], a[164:167], v[212:215], v[52:55]\n" \
  "v_mfma_f32_16x16x32_f16 v[48:51], a[164:167], v[208:211], v[48:51]\n" \
  "ds_read_b128 v[200:203], v251 offset:2048\n" \
  "buffer_load_dwordx4 a[164:167], v248, s[24:27], s79 offen offset:1024 sc1\n" \
  "v_mfma_f32_16x16x32_f16 v[80:83], a[168:171], v[208:211], v[80:83]\n" \
  "v_mfma_f32_16x16x32_f16 v[84:87], a[168:171], v[212:215], v[84:87]\n" \
  "ds_read_b128 v[204:207], v251 offset:3072\n" \
  "v_mfma_f32_16x16x32_f16 v[88:91], a[168:171], v[216:219], v[88:91]\n" \
  "v_mfma_f32_16x16x32_f16 v[92:95], a[168:171], v[220:223], v[92:95]\n" \
  "buffer_load_dwordx4 a[168:171], v248, s[24:27], s79 offen offset:2048 sc1\n" \
  "v_mfma_f32_16x16x32_f16 v[124:127], a[172:175], v[220:223], v[124:127]\n" \
  "v_mfma_f32_16x16x32_f16 v[120:123], a[172:175], v[216:219], v[120:123]\n" \
  "v_mfma_f32_16x16x32_f16 v[116:119], a[172:175], v[212:215], v[116:119]\n" \
  "v_mfma_f32_16x16x32_f16 v[112:115], a[172:175], v[208:211], v[112:115]\n" \
  "buffer_load_dwordx4 a[172:175], v248, s[24:27], s79 offen offset:3072 sc1\n" \
  "v_mfma_f32_16x16x32_f16 a[16:19], a[176:179], v[208:211], a[16:19]\n" \
  "v_mfma_f32_16x16x32_f16 a[20:23], a[176:179], v[212:215], a[20:23]\n" \
  "v_mfma_f32_16x16x32_f16 a[24:27], a[176:179], v[216:219], a[24:27]\n" \
  "v_mfma_f32_16x16x32_f16 a[28:31], a[176:179], v[220:223], a[28:31]\n" \
  "v_mfma_f32_16x16x32_f16 a[60:63], a[180:183], v[220:223], a[60:63]\n" \
  "v_mfma_f32_16x16x32_f16 a[56:59], a[180:183], v[216:219], a[56:59]\n" \
  "v_mfma_f32_16x16x32_f16 a[52:55], a[180:183], v[212:215], a[52:55]\n" \
  "v_mfma_f32_16x16x32_f16 a[48:51], a[180:183], v[208:211], a[48:51]\n" \
  "v_mfma_f32_16x16x32_f16 a[80:83], a[184:187], v[208:211], a[80:83]\n" \
  "v_mfma_f32_16x16x32_f16 a[84:87], a[184:187], v[212:215], a[84:87]\n" \
  "v_mfma_f32_16x16x32_f16 a[88:91], a[184:187], v[216:219], a[88:91]\n" \
  "v_mfma_f32_16x16x32_f16 a[92:95], a[184:187], v[220:223], a[92:95]\n" \
  "v_mfma_f32_16x16x32_f16 a[124:127], a[188:191], v[220:223], a[124:127]\n" \
  "v_mfma_f32_16x16x32_f16 a[120:123], a[188:191], v[216:219], a[120:123]\n" \
  "v_mfma_f32_16x16x32_f16 a[116:119], a[188:191], v[212:215], a[116:119]\n" \
  "v_mfma_f32_16x16x32_f16 a[112:115], a[188:191], v[208:211], a[112:115]\n" \
  "s_waitcnt lgkmcnt(0)\n" \
  "v_readfirstlane_b32 s60, v228\n" \
  "s_waitcnt vmcnt(31)\n" \
  "v_mfma_f32_16x16x32_f16 v[0:3], a[192:195], v[192:195], v[0:3]\n" \
  "ds_read_b128 v[208:211], v251 offset:4096\n" \
  "v_mfma_f32_16x16x32_f16 v[4:7], a[192:195], v[196:199], v[4:7]\n" \
  "s_add_u32 s78, s69, s66\n" \
  "s_add_u32 s72, s78, 0\n" \
  "buffer_load_dwordx4 v[160:163], v249, s[28:31], s72 offen\n" \
  "v_mfma_f32_16x16x32_f16 v[8:11], a[192:195], v[200:203], v[8:11]\n" \
  "v_readlane_b32 s56, %1, s60\n" \
  "s_bitset0_b32 s59, s60\n" \
  "ds_read_b128 v[212:215], v251 offset:5120\n" \
  "v_mfma_f32_16x16x32_f16 v[12:15], a[192:195], v[204:207], v[12:15]\n" \
  "s_waitcnt vmcnt(31)\n" \
  "v_mfma_f32_16x16x32_f16 v[44:47], a[196:199], v[204:207], v[44:47]\n" \
  "s_and_b32 s56, s56, 63\n" \
  "ds_read_b128 v[216:219], v251 offset:6144\n" \
  "v_mfma_f32_16x16x32_f16 v[40:43], a[196:199], v[200:203], v[40:43]\n" \
  "s_lshl_b32 s57, s56, 22\n" \
  "s_lshl_b32 s58, s56, 8\n" \
  "s_add_u32 s73, s78, 32768\n" \
  "buffer_load_dwordx4 v[164:167], v249, s[28:31], s73 offen\n" \
  "v_mfma_f32_16x16x32_f16 v[36:39], a[196:199], v[196:199], v[36:39]\n" \
  "s_lshl_b32 s56, s56, 18\n" \
  "s_add_u32 s58, s58, s71\n" \
  "ds_read_b128 v[220:223], v251 offset:7168\n" \
  "v_mfma_f32_16x16x32_f16 v[32:35], a[196:199], v[192:195], v[32:35]\n" \
  "s_waitcnt vmcnt(31)\n" \
  "v_mfma_f32_16x16x32_f16 v[64:67], a[200:203], v[192:195], v[64:67]\n" \
  "s_waitcnt vmcnt(18)\n" \
  "v_cvt_pk_f16_f32 v224, v128, v132\n" \
  "v_mfma_f32_16x16x32_f16 v[68:71], a[200:203], v[196:199], v[68:71]\n" \
  "v_cvt_pk_f16_f32 v225, v136, v140\n" \
  "s_add_u32 s72, s78, 65536\n" \
  "buffer_load_dwordx4 v[168:171], v249, s[28:31], s72 offen\n" \
  "v_mfma_f32_16x16x32_f16 v[72:75], a[200:203], v[200:203], v[72:75]\n" \
  "v_cvt_pk_f16_f32 v226, v144, v148\n" \
  "v_mfma_f32_16x16x32_f16 v[76:79], a[200:203], v[204:207], v[76:79]\n" \
  "v_cvt_pk_f16_f32 v227, v152, v156\n" \
  "s_waitcnt vmcnt(31)\n" \
  "v_mfma_f32_16x16x32_f16 v[108:111], a[204:207], v[204:207], v[108:111]\n" \
  "ds_write_b128 v252, v[224:227] offset:0\n" \
  "v_mfma_f32_16x16x32_f16 v[104:107], a[204:207], v[200:203], v[104:107]\n" \
  "v_cvt_pk_f16_f32 v228, v129, v133\n" \
  "s_add_u32 s73, s78, 98304\n" \
  "buffer_load_dwordx4 v[172:175], v249, s[28:31], s73 offen\n" \
  "v_mfma_f32_16x16x32_f16 v[100:103], a[204:207], v[196:199], v[100:103]\n" \
  "v_cvt_pk_f16_f32 v229, v137, v141\n" \
  "v_mfma_f32_16x16x32_f16 v[96:99], a[204:207], v[192:195], v[96:99]\n" \
  "v_cvt_pk_f16_f32 v230, v145, v149\n" \
  "s_waitcnt vmcnt(31)\n" \
  "v_mfma_f32_16x16x32_f16 a[0:3], a[208:211], v[192:195], a[0:3]\n" \
  "v_cvt_pk_f16_f32 v231, v153, v157\n" \
  "v_mfma_f32_16x16x32_f16 a[4:7], a[208:211], v[196:199], a[4:7]\n" \
  "ds_write_b128 v252, v[228:231] offset:1024\n" \
  "s_add_u32 s72, s78, 131072\n" \
  "buffer_load_dwordx4 v[176:179], v249, s[28:31], s72 offen\n" \
  "v_mfma_f32_16x16x32_f16 a[8:11], a[208:211], v[200:203], a[8:11]\n" \
  "v_cvt_pk_f16_f32 v232, v130, v134\n" \
  "v_mfma_f32_16x16x32_f16 a[12:15], a[208:211], v[204:207], a[12:15]\n" \
  "v_cvt_pk_f16_f32 v233, v138, v142\n" \
  "s_waitcnt vmcnt(31)\n" \
  "v_mfma_f32_16x16x32_f16 a[44:47], a[212:215], v[204:207], a[44:47]\n" \
  "v_cvt_pk_f16_f32 v234, v146, v150\n" \
  "v_mfma_f32_16x16x32_f16 a[40:43], a[212:215], v[200:203], a[40:43]\n" \
  "v_cvt_pk_f16_f32 v235, v154, v158\n" \
  "s_add_u32 s73, s78, 163840\n" \
  "buffer_load_dwordx4 v[180:183], v249, s[28:31], s73 offen\n" \
  "v_mfma_f32_16x16x32_f16 a[36:39], a[212:215], v[196:199], a[36:39]\n" \
  "ds_write_b128 v252, v[232:235] offset:2048\n" \
  "v_mfma_f32_16x16x32_f16 a[32:35], a[212:215], v[192:195], a[32:35]\n" \
  "v_cvt_pk_f16_f32 v236, v131, v135\n" \
  "s_waitcnt vmcnt(31)\n" \
  "v_mfma_f32_16x16x32_f16 a[64:67], a[216:219], v[192:195], a[64:67]\n" \
  "v_cvt_pk_f16_f32 v237, v139, v143\n" \
  "v_mfma_f32_16x16x32_f16 a[68:71], a[216:219], v[196:199], a[68:71]\n" \
  "v_cvt_pk_f16_f32 v238, v147, v151\n" \
  "s_add_u32 s72, s78, 196608\n" \
  "buffer_load_dwordx4 v[184:187], v249, s[28:31], s72 offen\n" \
  "v_mfma_f32_16x16x32_f16 a[72:75], a[216:219], v[200:203], a[72:75]\n" \
  "v_cvt_pk_f16_f32 v239, v155, v159\n" \
  "v_mfma_f32_16x16x32_f16 a[76:79], a[216:219], v[204:207], a[76:79]\n" \
  "ds_write_b128 v252, v[236:239] offset:3072\n" \
  "s_waitcnt vmcnt(31)\n" \
  "v_mfma_f32_16x16x32_f16 a[108:111], a[220:223], v[204:207], a[108:111]\n" \
  "v_mfma_f32_16x16x32_f16 a[104:107], a[220:223], v[200:203], a[104:107]\n" \
  "s_add_u32 s73, s78, 229376\n" \
  "buffer_load_dwordx4 v[188:191], v249, s[28:31], s73 offen\n" \
  "v_mfma_f32_16x16x32_f16 a[100:103], a[220:223], v[196:199], a[100:103]\n" \
  "v_mfma_f32_16x16x32_f16 a[96:99], a[220:223], v[192:195], a[96:99]\n" \
  "s_waitcnt lgkmcnt(0)\n" \
  "v_mfma_f32_16x16x32_f16 v[16:19], a[192:195], v[208:211], v[16:19]\n" \
  "v_mfma_f32_16x16x32_f16 v[20:23], a[192:195], v[212:215], v[20:23]\n" \
  "v_mfma_f32_16x16x32_f16 v[24:27], a[192:195], v[216:219], v[24:27]\n" \
  "s_add_u32 s79, s68, s63\n" \
  "s_add_u32 s80, s79, s84\n" \
  "s_add_u32 s79, s79, s83\n" \
  "v_mfma_f32_16x16x32_f16 v[28:31], a[192:195], v[220:223], v[28:31]\n" \
  "ds_read_b128 v[192:195], v251 offset:8192\n" \
  "buffer_load_dwordx4 a[192:195], v248, s[24:27], s79 offen offset:0 sc1\n" \
  "v_mfma_f32_16x16x32_f16 v[60:63], a[196:199], v[220:223], v[60:63]\n" \
  "v_mfma_f32_16x16x32_f16 v[56:59], a[196:199], v[216:219], v[56:59]\n" \
  "ds_read_b128 v[196:199], v251 offset:9216\n" \
  "v_mfma_f32_16x16x32_f16 v[52:55], a[196:199], v[212:215], v[52:55]\n" \
  "v_mfma_f32_16x16x32_f16 v[48:51], a[196:199], v[208:211], v[48:51]\n" \
  "ds_read_b128 v[200:203], v251 offset:10240\n" \
  "buffer_load_dwordx4 a[196:199], v248, s[24:27], s79 offen offset:1024 sc1\n" \
  "v_mfma_f32_16x16x32_f16 v[80:83], a[200:203], v[208:211], v[80:83]\n" \
  "v_mfma_f32_16x16x32_f16 v[84:87], a[200:203], v[212:215], v[84:87]\n" \
  "ds_read_b128 v[204:207], v251 offset:11264\n" \
  "v_mfma_f32_16x16x32_f16 v[88:91], a[200:203], v[216:219], v[88:91]\n" \
  "v_mfma_f32_16x16x32_f16 v[92:95], a[200:203], v[220:223], v[92:95]\n" \
  "buffer_load_dwordx4 a[200:203], v248, s[24:27], s79 offen offset:2048 sc1\n" \
  "v_mfma_f32_16x16x32_f16 v[124:127], a[204:207], v[220:223], v[124:127]\n" \
  "v_mfma_f32_16x16x32_f16 v[120:123], a[204:207], v[216:219], v[120:123]\n" \
  "v_mfma_f32_16x16x32_f16 v[116:119], a[204:207], v[212:215], v[116:119]\n" \
  "v_mfma_f32_16x16x32_f16 v[112:115], a[204:207], v[208:211], v[112:115]\n" \
  "buffer_load_dwordx4 a[204:207], v248, s[24:27], s79 offen offset:3072 sc1\n" \
  "v_mfma_f32_16x16x32_f16 a[16:19], a[208:211], v[208:211], a[16:19]\n" \
  "v_mfma_f32_16x16x32_f16 a[20:23], a[208:211], v[212:215], a[20:23]\n" \
  "v_mfma_f32_16x16x32_f16 a[24:27], a[208:211], v[216:219], a[24:27]\n" \
  "v_mfma_f32_16x16x32_f16 a[28:31], a[208:211], v[220:223], a[28:31]\n" \
  "v_mfma_f32_16x16x32_f16 a[60:63], a[212:215], v[220:223], a[60:63]\n" \
  "v_mfma_f32_16x16x32_f16 a[56:59], a[212:215], v[216:219], a[56:59]\n" \
  "v_mfma_f32_16x16x32_f16 a[52:55], a[212:215], v[212:215], a[52:55]\n" \
  "v_mfma_f32_16x16x32_f16 a[48:51], a[212:215], v[208:211], a[48:51]\n" \
  "v_mfma_f32_16x16x32_f16 a[80:83], a[216:219], v[208:211], a[80:83]\n" \
  "v_mfma_f32_16x16x32_f16 a[84:87], a[216:219], v[212:215], a[84:87]\n" \
  "v_mfma_f32_16x16x32_f16 a[88:91], a[216:219], v[216:219], a[88:91]\n" \
  "v_mfma_f32_16x16x32_f16 a[92:95], a[216:219], v[220:223], a[92:95]\n" \
  "v_mfma_f32_16x16x32_f16 a[124:127], a[220:223], v[220:223], a[124:127]\n" \
  "v_mfma_f32_16x16x32_f16 a[120:123], a[220:223], v[216:219], a[120:123]\n" \
  "v_mfma_f32_16x16x32_f16 a[116:119], a[220:223], v[212:215], a[116:119]\n" \
  "v_mfma_f32_16x16x32_f16 a[112:115], a[220:223], v[208:211], a[112:115]\n" \
  "s_waitcnt lgkmcnt(0)\n" \
  "s_waitcnt vmcnt(27)\n" \
  "v_mfma_f32_16x16x32_f16 v[0:3], a[224:227], v[192:195], v[0:3]\n" \
  "ds_read_b128 v[208:211], v251 offset:12288\n" \
  "s_add_u32 s78, s57, s65\n" \
  "s_add_u32 s72, s78, 0\n" \
  "buffer_load_dwordx4 v[128:131], v249, s[28:31], s72 offen\n" \
  "v_mfma_f32_16x16x32_f16 v[4:7], a[224:227], v[196:199], v[4:7]\n" \
  "v_mfma_f32_16x16x32_f16 v[8:11], a[224:227], v[200:203], v[8:11]\n" \
  "ds_read_b128 v[212:215], v251 offset:13312\n" \
  "v_mfma_f32_16x16x32_f16 v[12:15], a[224:227], v[204:207], v[12:15]\n" \
  "s_waitcnt vmcnt(27)\n" \
  "v_mfma_f32_16x16x32_f16 v[44:47], a[228:231], v[204:207], v[44:47]\n" \
  "ds_read_b128 v[216:219], v251 offset:14336\n" \
  "s_add_u32 s73, s78, 32768\n" \
  "buffer_load_dwordx4 v[132:135], v249, s[28:31], s73 offen\n" \
  "v_mfma_f32_16x16x32_f16 v[40:43], a[228:231], v[200:203], v[40:43]\n" \
  "v_mfma_f32_16x16x32_f16 v[36:39], a[228:231], v[196:199], v[36:39]\n" \
  "ds_read_b128 v[220:223], v251 offset:15360\n" \
  "v_mfma_f32_16x16x32_f16 v[32:35], a[228:231], v[192:195], v[32:35]\n" \
  "s_waitcnt vmcnt(27)\n" \
  "v_mfma_f32_16x16x32_f16 v[64:67], a[232:235], v[192:195], v[64:67]\n" \
  "s_add_u32 s72, s78, 65536\n" \
  "buffer_load_dwordx4 v[136:139], v249, s[28:31], s72 offen\n" \
  "v_mfma_f32_16x16x32_f16 v[68:71], a[232:235], v[196:199], v[68:71]\n" \
  "v_mfma_f32_16x16x32_f16 v[72:75], a[232:235], v[200:203], v[72:75]\n" \
  "v_mfma_f32_16x16x32_f16 v[76:79], a[232:235], v[204:207], v[76:79]\n" \
  "s_waitcnt vmcnt(27)\n" \
  "v_mfma_f32_16x16x32_f16 v[108:111], a[236:239], v[204:207], v[108:111]\n" \
  "s_add_u32 s73, s78, 98304\n" \
  "buffer_load_dwordx4 v[140:143], v249, s[28:31], s73 offen\n" \
  "v_mfma_f32_16x16x32_f16 v[104:107], a[236:239], v[200:203], v[104:107]\n" \
  "v_mfma_f32_16x16x32_f16 v[100:103], a[236:239], v[196:199], v[100:103]\n" \
  "v_mfma_f32_16x16x32_f16 v[96:99], a[236:239], v[192:195], v[96:99]\n" \
  "s_waitcnt vmcnt(27)\n" \
  "v_mfma_f32_16x16x32_f16 a[0:3], a[240:243], v[192:195], a[0:3]\n" \
  "s_add_u32 s72, s78, 131072\n" \
  "buffer_load_dwordx4 v[144:147], v249, s[28:31], s72 offen\n" \
  "v_mfma_f32_16x16x32_f16 a[4:7], a[240:243], v[196:199], a[4:7]\n" \
  "v_mfma_f32_16x16x32_f16 a[8:11], a[240:243], v[200:203], a[8:11]\n" \
  "v_mfma_f32_16x16x32_f16 a[12:15], a[240:243], v[204:207], a[12:15]\n" \
  "s_waitcnt vmcnt(27)\n" \
  "v_mfma_f32_16x16x32_f16 a[44:47], a[244:247], v[204:207], a[44:47]\n" \
  "s_add_u32 s73, s78, 163840\n" \
  "buffer_load_dwordx4 v[148:151], v249, s[28:31], s73 offen\n" \
  "v_mfma_f32_16x16x32_f16 a[40:43], a[244:247], v[200:203], a[40:43]\n" \
  "v_mfma_f32_16x16x32_f16 a[36:39], a[244:247], v[196:199], a[36:39]\n" \
  "v_mfma_f32_16x16x32_f16 a[32:35], a[244:247], v[192:195], a[32:35]\n" \
  "s_waitcnt vmcnt(27)\n" \
  "v_mfma_f32_16x16x32_f16 a[64:67], a[248:251], v[192:195], a[64:67]\n" \
  "s_add_u32 s72, s78, 196608\n" \
  "buffer_load_dwordx4 v[152:155], v249, s[28:31], s72 offen\n" \
  "v_mfma_f32_16x16x32_f16 a[68:71], a[248:251], v[196:199], a[68:71]\n" \
  "v_mfma_f32_16x16x32_f16 a[72:75], a[248:251], v[200:203], a[72:75]\n" \
  "v_mfma_f32_16x16x32_f16 a[76:79], a[248:251], v[204:207], a[76:79]\n" \
  "s_waitcnt vmcnt(27)\n" \
  "v_mfma_f32_16x16x32_f16 a[108:111], a[252:255], v[204:207], a[108:111]\n" \
  "s_add_u32 s73, s78, 229376\n" \
  "buffer_load_dwordx4 v[156:159], v249, s[28:31], s73 offen\n" \
  "v_mfma_f32_16x16x32_f16 a[104:107], a[252:255], v[200:203], a[104:107]\n" \
  "v_mfma_f32_16x16x32_f16 a[100:103], a[252:255], v[196:199], a[100:103]\n" \
  "v_mfma_f32_16x16x32_f16 a[96:99], a[252:255], v[192:195], a[96:99]\n" \
  "s_waitcnt lgkmcnt(0)\n" \
  "s_barrier\n" \
  "v_mfma_f32_16x16x32_f16 v[16:19], a[224:227], v[208:211], v[16:19]\n" \
  "s_add_u32 s81, s81, 1\n" \
  "s_cmp_eq_u32 s81, 3\n" \
  "s_cselect_b32 s81, 0, s81\n" \
  "s_movk_i32 s74, 0x4000\n" \
  "s_cmov_b32 s74, 0xffff8000\n" \
  "v_mfma_f32_16x16x32_f16 v[20:23], a[224:227], v[212:215], v[20:23]\n" \
  "v_add_u32 v251, s74, v251\n" \
  "s_cmp_eq_u32 s81, 2\n" \
  "s_movk_i32 s74, 0x4000\n" \
  "s_cmov_b32 s74, 0xffff8000\n" \
  "v_mfma_f32_16x16x32_f16 v[24:27], a[224:227], v[216:219], v[24:27]\n" \
  "v_add_u32 v252, s74, v252\n" \
  "v_mfma_f32_16x16x32_f16 v[28:31], a[224:227], v[220:223], v[28:31]\n" \
  "ds_read_b128 v[192:195], v251 offset:0\n" \
  "v_mfma_f32_16x16x32_f16 v[60:63], a[228:231], v[220:223], v[60:63]\n" \
  "v_mfma_f32_16x16x32_f16 v[56:59], a[228:231], v[216:219], v[56:59]\n" \
  "ds_read_b128 v[196:199], v251 offset:1024\n" \
  "v_mfma_f32_16x16x32_f16 v[52:55], a[228:231], v[212:215], v[52:55]\n" \
  "v_mfma_f32_16x16x32_f16 v[48:51], a[228:231], v[208:211], v[48:51]\n" \
  "ds_read_b128 v[200:203], v251 offset:2048\n" \
  "v_mfma_f32_16x16x32_f16 v[80:83], a[232:235], v[208:211], v[80:83]\n" \
  "v_mfma_f32_16x16x32_f16 v[84:87], a[232:235], v[212:215], v[84:87]\n" \
  "ds_read_b128 v[204:207], v251 offset:3072\n" \
  "v_mfma_f32_16x16x32_f16 v[88:91], a[232:235], v[216:219], v[88:91]\n" \
  "v_mfma_f32_16x16x32_f16 v[92:95], a[232:235], v[220:223], v[92:95]\n" \
  "v_mfma_f32_16x16x32_f16 v[124:127], a[236:239], v[220:223], v[124:127]\n" \
  "v_mfma_f32_16x16x32_f16 v[120:123], a[236:239], v[216:219], v[120:123]\n" \
  "v_mfma_f32_16x16x32_f16 v[116:119], a[236:239], v[212:215], v[116:119]\n" \
  "v_mfma_f32_16x16x32_f16 v[112:115], a[236:239], v[208:211], v[112:115]\n" \
  "v_mfma_f32_16x16x32_f16 a[16:19], a[240:243], v[208:211], a[16:19]\n" \
  "v_mfma_f32_16x16x32_f16 a[20:23], a[240:243], v[212:215], a[20:23]\n" \
  "v_mfma_f32_16x16x32_f16 a[24:27], a[240:243], v[216:219], a[24:27]\n" \
  "v_mfma_f32_16x16x32_f16 a[28:31], a[240:243], v[220:223], a[28:31]\n" \
  "v_mfma_f32_16x16x32_f16 a[60:63], a[244:247], v[220:223], a[60:63]\n" \
  "v_mfma_f32_16x16x32_f16 a[56:59], a[244:247], v[216:219], a[56:59]\n" \
  "v_mfma_f32_16x16x32_f16 a[52:55], a[244:247], v[212:215], a[52:55]\n" \
  "v_mfma_f32_16x16x32_f16 a[48:51], a[244:247], v[208:211], a[48:51]\n" \
  "v_mfma_f32_16x16x32_f16 a[80:83], a[248:251], v[208:211], a[80:83]\n" \
  "v_mfma_f32_16x16x32_f16 a[84:87], a[248:251], v[212:215], a[84:87]\n" \
  "v_mfma_f32_16x16x32_f16 a[88:91], a[248:251], v[216:219], a[88:91]\n" \
  "v_mfma_f32_16x16x32_f16 a[92:95], a[248:251], v[220:223], a[92:95]\n" \
  "v_mfma_f32_16x16x32_f16 a[124:127], a[252:255], v[220:223], a[124:127]\n" \
  "v_mfma_f32_16x16x32_f16 a[120:123], a[252:255], v[216:219], a[120:123]\n" \
  "v_mfma_f32_16x16x32_f16 a[116:119], a[252:255], v[212:215], a[116:119]\n" \
  "v_mfma_f32_16x16x32_f16 a[112:115], a[252:255], v[208:211], a[112:115]\n" \
  "s_mov_b32 s92, s68\n" \
  "s_mov_b32 s68, s56\n" \
  "s_mov_b32 s69, s57\n" \
  "s_mov_b32 s70, s58\n" \
  "s_waitcnt lgkmcnt(0)\n" \
  "s_load_dwordx8 s[44:51], %6, s70\n" \
  "s_add_u32 s67, s67, 1\n" \
  "s_and_b32 s87, %7, 127\n" \
  "s_lshl_b32 s87, s87, 1\n" \
  "s_lshr_b32 s86, %7, 7\n" \
  "s_add_u32 s87, s87, s86\n" \
  "s_lshl_b32 s87, s87, 2\n" \
  "s_add_u32 s87, s87, %8\n" \
  "s_lshl_b32 s88, s87, 7\n" \
  "s_add_u32 s88, s88, 67174400\n" \
  "s_lshl_b32 s87, s87, 15\n" \
  "s_add_u32 s87, s87, 33554432\n" \
  "s_xor_b32 s86, s87, 0x20000\n" \
  "s_waitcnt vmcnt(27)\n" \
  "v_mfma_f32_16x16x32_f16 v[0:3], a[128:131], v[192:195], v[0:3]\n" \
  "ds_read_b128 v[208:211], v251 offset:4096\n" \
  "v_mfma_f32_16x16x32_f16 v[4:7], a[128:131], v[196:199], v[4:7]\n" \
  "ds_read_b128 v[212:215], v251 offset:5120\n" \
  "v_mfma_f32_16x16x32_f16 v[8:11], a[128:131], v[200:203], v[8:11]\n" \
  "ds_read_b128 v[216:219], v251 offset:6144\n" \
  "s_add_u32 s40, s87, 0\n" \
  "buffer_store_dwordx4 a[0:3], v248, s[36:39], s40 offen offset:0 sc1\n" \
  "v_mfma_f32_16x16x32_f16 v[12:15], a[128:131], v[204:207], v[12:15]\n" \
  "ds_read_b128 v[220:223], v251 offset:7168\n" \
  "s_waitcnt vmcnt(27)\n" \
  "v_mfma_f32_16x16x32_f16 v[44:47], a[132:135], v[204:207], v[44:47]\n" \
  "s_waitcnt vmcnt(13)\n" \
  "v_cvt_pk_f16_f32 v224, v160, v164\n" \
  "v_cvt_pk_f16_f32 v225, v168, v172\n" \
  "buffer_store_dwordx4 a[4:7], v248, s[36:39], s40 offen offset:1024 sc1\n" \
  "v_mfma_f32_16x16x32_f16 v[40:43], a[132:135], v[200:203], v[40:43]\n" \
  "v_cvt_pk_f16_f32 v226, v176, v180\n" \
  "v_cvt_pk_f16_f32 v227, v184, v188\n" \
  "v_mfma_f32_16x16x32_f16 v[36:39], a[132:135], v[196:199], v[36:39]\n" \
  "ds_write_b128 v252, v[224:227] offset:0\n" \
  "v_cvt_pk_f16_f32 v228, v161, v165\n" \
  "v_mfma_f32_16x16x32_f16 v[32:35], a[132:135], v[192:195], v[32:35]\n" \
  "v_cvt_pk_f16_f32 v229, v169, v173\n" \
  "v_cvt_pk_f16_f32 v230, v177, v181\n" \
  "buffer_store_dwordx4 a[8:11], v248, s[36:39], s40 offen offset:2048 sc1\n" \
  "s_waitcnt vmcnt(28)\n" \
  "v_mfma_f32_16x16x32_f16 v[64:67], a[136:139], v[192:195], v[64:67]\n" \
  "v_cvt_pk_f16_f32 v231, v185, v189\n" \
  "ds_write_b128 v252, v[228:231] offset:1024\n" \
  "v_mfma_f32_16x16x32_f16 v[68:71], a[136:139], v[196:199], v[68:71]\n" \
  "v_cvt_pk_f16_f32 v232, v162, v166\n" \
  "v_cvt_pk_f16_f32 v233, v170, v174\n" \
  "buffer_store_dwordx4 a[12:15], v248, s[36:39], s40 offen offset:3072 sc1\n" \
  "v_mfma_f32_16x16x32_f16 v[72:75], a[136:139], v[200:203], v[72:75]\n" \
  "v_cvt_pk_f16_f32 v234, v178, v182\n" \
  "v_cvt_pk_f16_f32 v235, v186, v190\n" \
  "v_mfma_f32_16x16x32_f16 v[76:79], a[136:139], v[204:207], v[76:79]\n" \
  "ds_write_b128 v252, v[232:235] offset:2048\n" \
  "v_cvt_pk_f16_f32 v236, v163, v167\n" \
  "s_waitcnt vmcnt(28)\n" \
  "v_mfma_f32_16x16x32_f16 v[108:111], a[140:143], v[204:207], v[108:111]\n" \
  "v_cvt_pk_f16_f32 v237, v171, v175\n" \
  "v_cvt_pk_f16_f32 v238, v179, v183\n" \
  "s_add_u32 s40, s87, 4096\n" \
  "buffer_store_dwordx4 a[16:19], v248, s[36:39], s40 offen offset:0 sc1\n" \
  "v_mfma_f32_16x16x32_f16 v[104:107], a[140:143], v[200:203], v[104:107]\n" \
  "v_cvt_pk_f16_f32 v239, v187, v191\n" \
  "ds_write_b128 v252, v[236:239] offset:3072\n" \
  "v_mfma_f32_16x16x32_f16 v[100:103], a[140:143], v[196:199], v[100:103]\n" \
  "buffer_store_dwordx4 a[20:23], v248, s[36:39], s40 offen offset:1024 sc1\n" \
  "v_mfma_f32_16x16x32_f16 v[96:99], a[140:143], v[192:195], v[96:99]\n" \
  "s_waitcnt lgkmcnt(0)\n" \
  "v_mfma_f32_16x16x32_f16 v[16:19], a[128:131], v[208:211], v[16:19]\n" \
  "s_min_u32 s44, s44, s45\n" \
  "s_min_u32 s46, s46, s47\n" \
  "s_min_u32 s48, s48, s49\n" \
  "s_min_u32 s50, s50, s51\n" \
  "v_mfma_f32_16x16x32_f16 v[20:23], a[128:131], v[212:215], v[20:23]\n" \
  "s_add_u32 s79, s92, s64\n" \
  "s_add_u32 s80, s79, s84\n" \
  "s_add_u32 s79, s79, s83\n" \
  "buffer_load_dwordx4 a[224:227], v248, s[24:27], s79 offen offset:0 sc1\n" \
  "s_min_u32 s44, s44, s46\n" \
  "s_min_u32 s48, s48, s50\n" \
  "s_min_u32 s44, s44, s48\n" \
  "s_cmp_lt_u32 s44, s82\n" \
  "s_cbranch_scc1 SLOW43_%=\n" \
  "OK42_%=:\n" \
  "buffer_store_dwordx4 a[24:27], v248, s[36:39], s40 offen offset:2048 sc1\n" \
  "v_mfma_f32_16x16x32_f16 v[24:27], a[128:131], v[216:219], v[24:27]\n" \
  "ds_read_b128 v[192:195], v251 offset:8192\n" \
  "v_mfma_f32_16x16x32_f16 v[28:31], a[128:131], v[220:223], v[28:31]\n" \
  "ds_read_b128 v[196:199], v251 offset:9216\n" \
  "buffer_store_dwordx4 a[28:31], v248, s[36:39], s40 offen offset:3072 sc1\n" \
  "v_mfma_f32_16x16x32_f16 v[60:63], a[132:135], v[220:223], v[60:63]\n" \
  "ds_read_b128 v[200:203], v251 offset:10240\n" \
  "v_mfma_f32_16x16x32_f16 v[56:59], a[132:135], v[216:219], v[56:59]\n" \
  "ds_read_b128 v[204:207], v251 offset:11264\n" \
  "buffer_load_dwordx4 a[228:231], v248, s[24:27], s79 offen offset:1024 sc1\n" \
  "v_mfma_f32_16x16x32_f16 v[52:55], a[132:135], v[212:215], v[52:55]\n" \
  "s_add_u32 s40, s87, 8192\n" \
  "buffer_store_dwordx4 a[32:35], v248, s[36:39], s40 offen offset:0 sc1\n" \
  "v_mfma_f32_16x16x32_f16 v[48:51], a[132:135], v[208:211], v[48:51]\n" \
  "v_mfma_f32_16x16x32_f16 v[80:83], a[136:139], v[208:211], v[80:83]\n" \
  "v_mfma_f32_16x16x32_f16 v[84:87], a[136:139], v[212:215], v[84:87]\n" \
  "buffer_load_dwordx4 a[232:235], v248, s[24:27], s79 offen offset:2048 sc1\n" \
  "buffer_store_dwordx4 a[36:39], v248, s[36:39], s40 offen offset:1024 sc1\n" \
  "v_mfma_f32_16x16x32_f16 v[88:91], a[136:139], v[216:219], v[88:91]\n" \
  "v_mfma_f32_16x16x32_f16 v[92:95], a[136:139], v[220:223], v[92:95]\n" \
  "buffer_store_dwordx4 a[40:43], v248, s[36:39], s40 offen offset:2048 sc1\n" \
  "v_mfma_f32_16x16x32_f16 v[124:127], a[140:143], v[220:223], v[124:127]\n" \
  "v_mfma_f32_16x16x32_f16 v[120:123], a[140:143], v[216:219], v[120:123]\n" \
  "buffer_load_dwordx4 a[236:239], v248, s[24:27], s79 offen offset:3072 sc1\n" \
  "v_mfma_f32_16x16x32_f16 v[116:119], a[140:143], v[212:215], v[116:119]\n" \
  "buffer_store_dwordx4 a[44:47], v248, s[36:39], s40 offen offset:3072 sc1\n" \
  "v_mfma_f32_16x16x32_f16 v[112:115], a[140:143], v[208:211], v[112:115]\n" \
  "s_waitcnt lgkmcnt(0)\n" \
  "s_waitcnt vmcnt(39)\n" \
  "v_mfma_f32_16x16x32_f16 v[0:3], a[160:163], v[192:195], v[0:3]\n" \
  "ds_read_b128 v[208:211], v251 offset:12288\n" \
  "s_add_u32 s79, s68, s61\n" \
  "s_add_u32 s80, s79, s84\n" \
  "s_add_u32 s79, s79, s83\n" \
  "s_add_u32 s40, s87, 12288\n" \
  "buffer_store_dwordx4 a[48:51], v248, s[36:39], s40 offen offset:0 sc1\n" \
  "v_mfma_f32_16x16x32_f16 v[4:7], a[160:163], v[196:199], v[4:7]\n" \
  "ds_read_b128 v[212:215], v251 offset:13312\n" \
  "buffer_load_dwordx4 a[128:131], v248, s[24:27], s79 offen offset:0 sc1\n" \
  "v_mfma_f32_16x16x32_f16 v[8:11], a[160:163], v[200:203], v[8:11]\n" \
  "ds_read_b128 v[216:219], v251 offset:14336\n" \
  "v_mfma_f32_16x16x32_f16 v[12:15], a[160:163], v[204:207], v[12:15]\n" \
  "ds_read_b128 v[220:223], v251 offset:15360\n" \
  "buffer_store_dwordx4 a[52:55], v248, s[36:39], s40 offen offset:1024 sc1\n" \
  "s_waitcnt vmcnt(41)\n" \
  "v_mfma_f32_16x16x32_f16 v[44:47], a[164:167], v[204:207], v[44:47]\n" \
  "buffer_load_dwordx4 a[132:135], v248, s[24:27], s79 offen offset:1024 sc1\n" \
  "v_mfma_f32_16x16x32_f16 v[40:43], a[164:167], v[200:203], v[40:43]\n" \
  "buffer_store_dwordx4 a[56:59], v248, s[36:39], s40 offen offset:2048 sc1\n" \
  "v_mfma_f32_16x16x32_f16 v[36:39], a[164:167], v[196:199], v[36:39]\n" \
  "v_mfma_f32_16x16x32_f16 v[32:35], a[164:167], v[192:195], v[32:35]\n" \
  "buffer_load_dwordx4 a[136:139], v248, s[24:27], s79 offen offset:2048 sc1\n" \
  "s_waitcnt vmcnt(43)\n" \
  "v_mfma_f32_16x16x32_f16 v[64:67], a[168:171], v[192:195], v[64:67]\n" \
  "buffer_store_dwordx4 a[60:63], v248, s[36:39], s40 offen offset:3072 sc1\n" \
  "v_mfma_f32_16x16x32_f16 v[68:71], a[168:171], v[196:199], v[68:71]\n" \
  "v_mfma_f32_16x16x32_f16 v[72:75], a[168:171], v[200:203], v[72:75]\n" \
  "buffer_load_dwordx4 a[140:143], v248, s[24:27], s79 offen offset:3072 sc1\n" \
  "v_mfma_f32_16x16x32_f16 v[76:79], a[168:171], v[204:207], v[76:79]\n" \
  "s_add_u32 s40, s87, 16384\n" \
  "buffer_store_dwordx4 a[64:67], v248, s[36:39], s40 offen offset:0 sc1\n" \
  "s_waitcnt vmcnt(45)\n" \
  "v_mfma_f32_16x16x32_f16 v[108:111], a[172:175], v[204:207], v[108:111]\n" \
  "v_mfma_f32_16x16x32_f16 v[104:107], a[172:175], v[200:203], v[104:107]\n" \
  "buffer_store_dwordx4 a[68:71], v248, s[36:39], s40 offen offset:1024 sc1\n" \
  "v_mfma_f32_16x16x32_f16 v[100:103], a[172:175], v[196:199], v[100:103]\n" \
  "v_mfma_f32_16x16x32_f16 v[96:99], a[172:175], v[192:195], v[96:99]\n" \
  "s_waitcnt lgkmcnt(0)\n" \
  "s_barrier\n" \
  "v_mfma_f32_16x16x32_f16 v[16:19], a[160:163], v[208:211], v[16:19]\n" \
  "s_add_u32 s81, s81, 1\n" \
  "s_cmp_eq_u32 s81, 3\n" \
  "s_cselect_b32 s81, 0, s81\n" \
  "s_movk_i32 s74, 0x4000\n" \
  "s_cmov_b32 s74, 0xffff8000\n" \
  "buffer_store_dwordx4 a[72:75], v248, s[36:39], s40 offen offset:2048 sc1\n" \
  "v_mfma_f32_16x16x32_f16 v[20:23], a[160:163], v[212:215], v[20:23]\n" \
  "v_add_u32 v251, s74, v251\n" \
  "s_cmp_eq_u32 s81, 2\n" \
  "s_movk_i32 s74, 0x4000\n" \
  "s_cmov_b32 s74, 0xffff8000\n" \
  "v_mfma_f32_16x16x32_f16 v[24:27], a[160:163], v[216:219], v[24:27]\n" \
  "v_add_u32 v252, s74, v252\n" \
  "ds_read_b128 v[192:195], v251 offset:0\n" \
  "s_add_u32 s79, s68, s62\n" \
  "s_add_u32 s80, s79, s84\n" \
  "s_add_u32 s79, s79, s83\n" \
  "buffer_store_dwordx4 a[76:79], v248, s[36:39], s40 offen offset:3072 sc1\n" \
  "v_mfma_f32_16x16x32_f16 v[28:31], a[160:163], v[220:223], v[28:31]\n" \
  "ds_read_b128 v[196:199], v251 offset:1024\n" \
  "buffer_load_dwordx4 a[160:163], v248, s[24:27], s79 offen offset:0 sc1\n" \
  "v_mfma_f32_16x16x32_f16 v[60:63], a[164:167], v[220:223], v[60:63]\n" \
  "ds_read_b128 v[200:203], v251 offset:2048\n" \
  "v_mfma_f32_16x16x32_f16 v[56:59], a[164:167], v[216:219], v[56:59]\n" \
  "ds_read_b128 v[204:207], v251 offset:3072\n" \
  "s_add_u32 s40, s87, 20480\n" \
  "buffer_store_dwordx4 a[80:83], v248, s[36:39], s40 offen offset:0 sc1\n" \
  "v_mfma_f32_16x16x32_f16 v[52:55], a[164:167], v[212:215], v[52:55]\n" \
  "v_mfma_f32_16x16x32_f16 v[48:51], a[164:167], v[208:211], v[48:51]\n" \
  "buffer_load_dwordx4 a[164:167], v248, s[24:27], s79 offen offset:1024 sc1\n" \
  "buffer_store_dwordx4 a[84:87], v248, s[36:39], s40 offen offset:1024 sc1\n" \
  "v_mfma_f32_16x16x32_f16 v[80:83], a[168:171], v[208:211], v[80:83]\n" \
  "v_mfma_f32_16x16x32_f16 v[84:87], a[168:171], v[212:215], v[84:87]\n" \
  "v_mfma_f32_16x16x32_f16 v[88:91], a[168:171], v[216:219], v[88:91]\n" \
  "buffer_store_dwordx4 a[88:91], v248, s[36:39], s40 offen offset:2048 sc1\n" \
  "v_mfma_f32_16x16x32_f16 v[92:95], a[168:171], v[220:223], v[92:95]\n" \
  "buffer_load_dwordx4 a[168:171], v248, s[24:27], s79 offen offset:2048 sc1\n" \
  "v_mfma_f32_16x16x32_f16 v[124:127], a[172:175], v[220:223], v[124:127]\n" \
  "buffer_store_dwordx4 a[92:95], v248, s[36:39], s40 offen offset:3072 sc1\n" \
  "v_mfma_f32_16x16x32_f16 v[120:123], a[172:175], v[216:219], v[120:123]\n" \
  "v_mfma_f32_16x16x32_f16 v[116:119], a[172:175], v[212:215], v[116:119]\n" \
  "v_mfma_f32_16x16x32_f16 v[112:115], a[172:175], v[208:211], v[112:115]\n" \
  "buffer_load_dwordx4 a[172:175], v248, s[24:27], s79 offen offset:3072 sc1\n" \
  "s_waitcnt lgkmcnt(0)\n" \
  "s_add_u32 s40, s87, 24576\n" \
  "buffer_store_dwordx4 a[96:99], v248, s[36:39], s40 offen offset:0 sc1\n" \
  "s_waitcnt vmcnt(48)\n" \
  "v_mfma_f32_16x16x32_f16 v[0:3], a[192:195], v[192:195], v[0:3]\n" \
  "ds_read_b128 v[208:211], v251 offset:4096\n" \
  "s_add_u32 s78, s69, s66\n" \
  "s_add_u32 s72, s78, 0\n" \
  "buffer_load_dwordx4 v[160:163], v249, s[28:31], s72 offen\n" \
  "v_mfma_f32_16x16x32_f16 v[4:7], a[192:195], v[196:199], v[4:7]\n" \
  "ds_read_b128 v[212:215], v251 offset:5120\n" \
  "v_mfma_f32_16x16x32_f16 v[8:11], a[192:195], v[200:203], v[8:11]\n" \
  "ds_read_b128 v[216:219], v251 offset:6144\n" \
  "s_add_u32 s73, s78, 32768\n" \
  "buffer_load_dwordx4 v[164:167], v249, s[28:31], s73 offen\n" \
  "buffer_store_dwordx4 a[100:103], v248, s[36:39], s40 offen offset:1024 sc1\n" \
  "v_mfma_f32_16x16x32_f16 v[12:15], a[192:195], v[204:207], v[12:15]\n" \
  "ds_read_b128 v[220:223], v251 offset:7168\n" \
  "s_waitcnt vmcnt(50)\n" \
  "v_mfma_f32_16x16x32_f16 v[44:47], a[196:199], v[204:207], v[44:47]\n" \
  "s_waitcnt vmcnt(40)\n" \
  "v_cvt_pk_f16_f32 v224, v128, v132\n" \
  "v_cvt_pk_f16_f32 v225, v136, v140\n" \
  "s_add_u32 s72, s78, 65536\n" \
  "buffer_load_dwordx4 v[168:171], v249, s[28:31], s72 offen\n" \
  "buffer_store_dwordx4 a[104:107], v248, s[36:39], s40 offen offset:2048 sc1\n" \
  "v_mfma_f32_16x16x32_f16 v[40:43], a[196:199], v[200:203], v[40:43]\n" \
  "v_cvt_pk_f16_f32 v226, v144, v148\n" \
  "v_cvt_pk_f16_f32 v227, v152, v156\n" \
  "v_mfma_f32_16x16x32_f16 v[36:39], a[196:199], v[196:199], v[36:39]\n" \
  "ds_write_b128 v252, v[224:227] offset:0\n" \
  "v_cvt_pk_f16_f32 v228, v129, v133\n" \
  "s_add_u32 s73, s78, 98304\n" \
  "buffer_load_dwordx4 v[172:175], v249, s[28:31], s73 offen\n" \
  "v_mfma_f32_16x16x32_f16 v[32:35], a[196:199], v[192:195], v[32:35]\n" \
  "v_cvt_pk_f16_f32 v229, v137, v141\n" \
  "v_cvt_pk_f16_f32 v230, v145, v149\n" \
  "buffer_store_dwordx4 a[108:111], v248, s[36:39], s40 offen offset:3072 sc1\n" \
  "s_waitcnt vmcnt(53)\n" \
  "v_mfma_f32_16x16x32_f16 v[64:67], a[200:203], v[192:195], v[64:67]\n" \
  "v_cvt_pk_f16_f32 v231, v153, v157\n" \
  "ds_write_b128 v252, v[228:231] offset:1024\n" \
  "s_add_u32 s72, s78, 131072\n" \
  "buffer_load_dwordx4 v[176:179], v249, s[28:31], s72 offen\n" \
  "v_mfma_f32_16x16x32_f16 v[68:71], a[200:203], v[196:199], v[68:71]\n" \
  "v_cvt_pk_f16_f32 v232, v130, v134\n" \
  "v_cvt_pk_f16_f32 v233, v138, v142\n" \
  "s_add_u32 s40, s87, 28672\n" \
  "buffer_store_dwordx4 a[112:115], v248, s[36:39], s40 offen offset:0 sc1\n" \
  "v_mfma_f32_16x16x32_f16 v[72:75], a[200:203], v[200:203], v[72:75]\n" \
  "v_cvt_pk_f16_f32 v234, v146, v150\n" \
  "v_cvt_pk_f16_f32 v235, v154, v158\n" \
  "s_add_u32 s73, s78, 163840\n" \
  "buffer_load_dwordx4 v[180:183], v249, s[28:31], s73 offen\n" \
  "v_mfma_f32_16x16x32_f16 v[76:79], a[200:203], v[204:207], v[76:79]\n" \
  "ds_write_b128 v252, v[232:235] offset:2048\n" \
  "v_cvt_pk_f16_f32 v236, v131, v135\n" \
  "s_waitcnt vmcnt(55)\n" \
  "v_mfma_f32_16x16x32_f16 v[108:111], a[204:207], v[204:207], v[108:111]\n" \
  "v_cvt_pk_f16_f32 v237, v139, v143\n" \
  "v_cvt_pk_f16_f32 v238, v147, v151\n" \
  "s_add_u32 s72, s78, 196608\n" \
  "buffer_load_dwordx4 v[184:187], v249, s[28:31], s72 offen\n" \
  "buffer_store_dwordx4 a[116:119], v248, s[36:39], s40 offen offset:1024 sc1\n" \
  "v_mfma_f32_16x16x32_f16 v[104:107], a[204:207], v[200:203], v[104:107]\n" \
  "v_cvt_pk_f16_f32 v239, v155, v159\n" \
  "ds_write_b128 v252, v[236:239] offset:3072\n" \
  "v_mfma_f32_16x16x32_f16 v[100:103], a[204:207], v[196:199], v[100:103]\n" \
  "s_add_u32 s73, s78, 229376\n" \
  "buffer_load_dwordx4 v[188:191], v249, s[28:31], s73 offen\n" \
  "buffer_store_dwordx4 a[120:123], v248, s[36:39], s40 offen offset:2048 sc1\n" \
  "v_mfma_f32_16x16x32_f16 v[96:99], a[204:207], v[192:195], v[96:99]\n" \
  "s_waitcnt lgkmcnt(0)\n" \
  "v_mfma_f32_16x16x32_f16 v[16:19], a[192:195], v[208:211], v[16:19]\n" \
  "v_mfma_f32_16x16x32_f16 v[20:23], a[192:195], v[212:215], v[20:23]\n" \
  "buffer_store_dwordx4 a[124:127], v248, s[36:39], s40 offen offset:3072 sc1\n" \
  "v_mfma_f32_16x16x32_f16 v[24:27], a[192:195], v[216:219], v[24:27]\n" \
  "ds_read_b128 v[192:195], v251 offset:8192\n" \
  "s_add_u32 s79, s68, s63\n" \
  "s_add_u32 s80, s79, s84\n" \
  "s_add_u32 s79, s79, s83\n" \
  "v_mfma_f32_16x16x32_f16 v[28:31], a[192:195], v[220:223], v[28:31]\n" \
  "ds_read_b128 v[196:199], v251 offset:9216\n" \
  "buffer_load_dwordx4 a[192:195], v248, s[24:27], s79 offen offset:0 sc1\n" \
  "v_mfma_f32_16x16x32_f16 v[60:63], a[196:199], v[220:223], v[60:63]\n" \
  "ds_read_b128 v[200:203], v251 offset:10240\n" \
  "v_mfma_f32_16x16x32_f16 v[56:59], a[196:199], v[216:219], v[56:59]\n" \
  "ds_read_b128 v[204:207], v251 offset:11264\n" \
  "v_mfma_f32_16x16x32_f16 v[52:55], a[196:199], v[212:215], v[52:55]\n" \
  "v_mfma_f32_16x16x32_f16 v[48:51], a[196:199], v[208:211], v[48:51]\n" \
  "buffer_load_dwordx4 a[196:199], v248, s[24:27], s79 offen offset:1024 sc1\n" \
  "v_mfma_f32_16x16x32_f16 v[80:83], a[200:203], v[208:211], v[80:83]\n" \
  "v_mfma_f32_16x16x32_f16 v[84:87], a[200:203], v[212:215], v[84:87]\n" \
  "v_mfma_f32_16x16x32_f16 v[88:91], a[200:203], v[216:219], v[88:91]\n" \
  "v_mfma_f32_16x16x32_f16 v[92:95], a[200:203], v[220:223], v[92:95]\n" \
  "buffer_load_dwordx4 a[200:203], v248, s[24:27], s79 offen offset:2048 sc1\n" \
  "v_mfma_f32_16x16x32_f16 v[124:127], a[204:207], v[220:223], v[124:127]\n" \
  "v_mfma_f32_16x16x32_f16 v[120:123], a[204:207], v[216:219], v[120:123]\n" \
  "v_mfma_f32_16x16x32_f16 v[116:119], a[204:207], v[212:215], v[116:119]\n" \
  "v_mfma_f32_16x16x32_f16 v[112:115], a[204:207], v[208:211], v[112:115]\n" \
  "buffer_load_dwordx4 a[204:207], v248, s[24:27], s79 offen offset:3072 sc1\n" \
  "s_waitcnt lgkmcnt(0)\n" \
  "s_waitcnt vmcnt(49)\n" \
  "v_mfma_f32_16x16x32_f16 v[0:3], a[224:227], v[192:195], v[0:3]\n" \
  "ds_read_b128 v[208:211], v251 offset:12288\n" \
  "v_mfma_f32_16x16x32_f16 v[4:7], a[224:227], v[196:199], v[4:7]\n" \
  "ds_read_b128 v[212:215], v251 offset:13312\n" \
  "v_mfma_f32_16x16x32_f16 v[8:11], a[224:227], v[200:203], v[8:11]\n" \
  "ds_read_b128 v[216:219], v251 offset:14336\n" \
  "v_mfma_f32_16x16x32_f16 v[12:15], a[224:227], v[204:207], v[12:15]\n" \
  "ds_read_b128 v[220:223], v251 offset:15360\n" \
  "s_waitcnt vmcnt(46)\n" \
  "v_mfma_f32_16x16x32_f16 v[44:47], a[228:231], v[204:207], v[44:47]\n" \
  "v_mfma_f32_16x16x32_f16 v[40:43], a[228:231], v[200:203], v[40:43]\n" \
  "v_mfma_f32_16x16x32_f16 v[36:39], a[228:231], v[196:199], v[36:39]\n" \
  "v_mfma_f32_16x16x32_f16 v[32:35], a[228:231], v[192:195], v[32:35]\n" \
  "s_waitcnt vmcnt(44)\n" \
  "v_mfma_f32_16x16x32_f16 v[64:67], a[232:235], v[192:195], v[64:67]\n" \
  "v_mfma_f32_16x16x32_f16 v[68:71], a[232:235], v[196:199], v[68:71]\n" \
  "v_mfma_f32_16x16x32_f16 v[72:75], a[232:235], v[200:203], v[72:75]\n" \
  "v_mfma_f32_16x16x32_f16 v[76:79], a[232:235], v[204:207], v[76:79]\n" \
  "s_waitcnt vmcnt(41)\n" \
  "v_mfma_f32_16x16x32_f16 v[108:111], a[236:239], v[204:207], v[108:111]\n" \
  "v_mfma_f32_16x16x32_f16 v[104:107], a[236:239], v[200:203], v[104:107]\n" \
  "v_mfma_f32_16x16x32_f16 v[100:103], a[236:239], v[196:199], v[100:103]\n" \
  "v_mfma_f32_16x16x32_f16 v[96:99], a[236:239], v[192:195], v[96:99]\n" \
  "s_waitcnt lgkmcnt(0)\n" \
  "s_barrier\n" \
  "v_mfma_f32_16x16x32_f16 v[16:19], a[224:227], v[208:211], v[16:19]\n" \
  "s_add_u32 s81, s81, 1\n" \
  "s_cmp_eq_u32 s81, 3\n" \
  "s_cselect_b32 s81, 0, s81\n" \
  "s_movk_i32 s74, 0x4000\n" \
  "s_cmov_b32 s74, 0xffff8000\n" \
  "v_mfma_f32_16x16x32_f16 v[20:23], a[224:227], v[212:215], v[20:23]\n" \
  "v_add_u32 v251, s74, v251\n" \
  "s_cmp_eq_u32 s81, 2\n" \
  "s_movk_i32 s74, 0x4000\n" \
  "s_cmov_b32 s74, 0xffff8000\n" \
  "v_mfma_f32_16x16x32_f16 v[24:27], a[224:227], v[216:219], v[24:27]\n" \
  "v_add_u32 v252, s74, v252\n" \
  "ds_read_b128 v[192:195], v251 offset:0\n" \
  "v_mfma_f32_16x16x32_f16 v[28:31], a[224:227], v[220:223], v[28:31]\n" \
  "ds_read_b128 v[196:199], v251 offset:1024\n" \
  "v_mfma_f32_16x16x32_f16 v[60:63], a[228:231], v[220:223], v[60:63]\n" \
  "ds_read_b128 v[200:203], v251 offset:2048\n" \
  "v_mfma_f32_16x16x32_f16 v[56:59], a[228:231], v[216:219], v[56:59]\n" \
  "ds_read_b128 v[204:207], v251 offset:3072\n" \
  "v_mfma_f32_16x16x32_f16 v[52:55], a[228:231], v[212:215], v[52:55]\n" \
  "v_mfma_f32_16x16x32_f16 v[48:51], a[228:231], v[208:211], v[48:51]\n" \
  "v_mfma_f32_16x16x32_f16 v[80:83], a[232:235], v[208:211], v[80:83]\n" \
  "v_mfma_f32_16x16x32_f16 v[84:87], a[232:235], v[212:215], v[84:87]\n" \
  "v_mfma_f32_16x16x32_f16 v[88:91], a[232:235], v[216:219], v[88:91]\n" \
  "v_mfma_f32_16x16x32_f16 v[92:95], a[232:235], v[220:223], v[92:95]\n" \
  "v_mfma_f32_16x16x32_f16 v[124:127], a[236:239], v[220:223], v[124:127]\n" \
  "v_mfma_f32_16x16x32_f16 v[120:123], a[236:239], v[216:219], v[120:123]\n" \
  "v_mfma_f32_16x16x32_f16 v[116:119], a[236:239], v[212:215], v[116:119]\n" \
  "v_mfma_f32_16x16x32_f16 v[112:115], a[236:239], v[208:211], v[112:115]\n" \
  "s_mov_b32 s92, s68\n" \
  "s_waitcnt lgkmcnt(0)\n" \
  "s_add_u32 s67, s67, 1\n" \
  "s_waitcnt vmcnt(38)\n" \
  "v_mfma_f32_16x16x32_f16 v[0:3], a[128:131], v[192:195], v[0:3]\n" \
  "ds_read_b128 v[208:211], v251 offset:4096\n" \
  "v_mfma_f32_16x16x32_f16 v[4:7], a[128:131], v[196:199], v[4:7]\n" \
  "ds_read_b128 v[212:215], v251 offset:5120\n" \
  "v_mfma_f32_16x16x32_f16 v[8:11], a[128:131], v[200:203], v[8:11]\n" \
  "ds_read_b128 v[216:219], v251 offset:6144\n" \
  "v_mfma_f32_16x16x32_f16 v[12:15], a[128:131], v[204:207], v[12:15]\n" \
  "ds_read_b128 v[220:223], v251 offset:7168\n" \
  "s_waitcnt vmcnt(36)\n" \
  "v_mfma_f32_16x16x32_f16 v[44:47], a[132:135], v[204:207], v[44:47]\n" \
  "s_waitcnt vmcnt(6)\n" \
  "v_cvt_pk_f16_f32 v224, v160, v164\n" \
  "v_cvt_pk_f16_f32 v225, v168, v172\n" \
  "v_mfma_f32_16x16x32_f16 v[40:43], a[132:135], v[200:203], v[40:43]\n" \
  "v_cvt_pk_f16_f32 v226, v176, v180\n" \
  "v_cvt_pk_f16_f32 v227, v184, v188\n" \
  "v_mfma_f32_16x16x32_f16 v[36:39], a[132:135], v[196:199], v[36:39]\n" \
  "ds_write_b128 v252, v[224:227] offset:0\n" \
  "v_cvt_pk_f16_f32 v228, v161, v165\n" \
  "v_mfma_f32_16x16x32_f16 v[32:35], a[132:135], v[192:195], v[32:35]\n" \
  "v_cvt_pk_f16_f32 v229, v169, v173\n" \
  "v_cvt_pk_f16_f32 v230, v177, v181\n" \
  "s_waitcnt vmcnt(34)\n" \
  "v_mfma_f32_16x16x32_f16 v[64:67], a[136:139], v[192:195], v[64:67]\n" \
  "v_cvt_pk_f16_f32 v231, v185, v189\n" \
  "ds_write_b128 v252, v[228:231] offset:1024\n" \
  "v_mfma_f32_16x16x32_f16 v[68:71], a[136:139], v[196:199], v[68:71]\n" \
  "v_cvt_pk_f16_f32 v232, v162, v166\n" \
  "v_cvt_pk_f16_f32 v233, v170, v174\n" \
  "v_mfma_f32_16x16x32_f16 v[72:75], a[136:139], v[200:203], v[72:75]\n" \
  "v_cvt_pk_f16_f32 v234, v178, v182\n" \
  "v_cvt_pk_f16_f32 v235, v186, v190\n" \
  "v_mfma_f32_16x16x32_f16 v[76:79], a[136:139], v[204:207], v[76:79]\n" \
  "ds_write_b128 v252, v[232:235] offset:2048\n" \
  "v_cvt_pk_f16_f32 v236, v163, v167\n" \
  "s_waitcnt vmcnt(32)\n" \
  "v_mfma_f32_16x16x32_f16 v[108:111], a[140:143], v[204:207], v[108:111]\n" \
  "v_cvt_pk_f16_f32 v237, v171, v175\n" \
  "v_cvt_pk_f16_f32 v238, v179, v183\n" \
  "v_mfma_f32_16x16x32_f16 v[104:107], a[140:143], v[200:203], v[104:107]\n" \
  "v_cvt_pk_f16_f32 v239, v187, v191\n" \
  "ds_write_b128 v252, v[236:239] offset:3072\n" \
  "v_mfma_f32_16x16x32_f16 v[100:103], a[140:143], v[196:199], v[100:103]\n" \
  "v_mfma_f32_16x16x32_f16 v[96:99], a[140:143], v[192:195], v[96:99]\n" \
  "s_waitcnt lgkmcnt(0)\n" \
  "v_mfma_f32_16x16x32_f16 v[16:19], a[128:131], v[208:211], v[16:19]\n" \
  "v_mfma_f32_16x16x32_f16 v[20:23], a[128:131], v[212:215], v[20:23]\n" \
  "s_add_u32 s79, s92, s64\n" \
  "s_add_u32 s80, s79, s84\n" \
  "s_add_u32 s79, s79, s83\n" \
  "buffer_load_dwordx4 a[224:227], v248, s[24:27], s79 offen offset:0 sc1\n" \
  "v_mfma_f32_16x16x32_f16 v[24:27], a[128:131], v[216:219], v[24:27]\n" \
  "ds_read_b128 v[192:195], v251 offset:8192\n" \
  "v_mfma_f32_16x16x32_f16 v[28:31], a[128:131], v[220:223], v[28:31]\n" \
  "ds_read_b128 v[196:199], v251 offset:9216\n" \
  "s_waitcnt vmcnt(5)\n" \
  "v_mov_b32 v240, s89\n" \
  "v_lshlrev_b32 v240, 31, v240\n" \
  "v_or_b32 v240, s82, v240\n" \
  "s_mov_b64 exec, 1\n" \
  "buffer_store_dword v240, off, s[36:39], s88 sc1\n" \
  "s_mov_b64 exec, -1\n" \
  "v_mfma_f32_16x16x32_f16 v[60:63], a[132:135], v[220:223], v[60:63]\n" \
  "ds_read_b128 v[200:203], v251 offset:10240\n" \
  "s_xor_b32 s88, s88, 0x200\n" \
  "v_mfma_f32_16x16x32_f16 v[56:59], a[132:135], v[216:219], v[56:59]\n" \
  "ds_read_b128 v[204:207], v251 offset:11264\n" \
  "buffer_load_dwordx4 a[228:231], v248, s[24:27], s79 offen offset:1024 sc1\n" \
  "v_mfma_f32_16x16x32_f16 v[52:55], a[132:135], v[212:215], v[52:55]\n" \
  "v_mfma_f32_16x16x32_f16 v[48:51], a[132:135], v[208:211], v[48:51]\n" \
  "v_mfma_f32_16x16x32_f16 v[80:83], a[136:139], v[208:211], v[80:83]\n" \
  "v_mfma_f32_16x16x32_f16 v[84:87], a[136:139], v[212:215], v[84:87]\n" \
  "buffer_load_dwordx4 a[232:235], v248, s[24:27], s79 offen offset:2048 sc1\n" \
  "v_mfma_f32_16x16x32_f16 v[88:91], a[136:139], v[216:219], v[88:91]\n" \
  "v_mfma_f32_16x16x32_f16 v[92:95], a[136:139], v[220:223], v[92:95]\n" \
  "v_mfma_f32_16x16x32_f16 v[124:127], a[140:143], v[220:223], v[124:127]\n" \
  "v_mfma_f32_16x16x32_f16 v[120:123], a[140:143], v[216:219], v[120:123]\n" \
  "buffer_load_dwordx4 a[236:239], v248, s[24:27], s79 offen offset:3072 sc1\n" \
  "v_mfma_f32_16x16x32_f16 v[116:119], a[140:143], v[212:215], v[116:119]\n" \
  "v_mfma_f32_16x16x32_f16 v[112:115], a[140:143], v[208:211], v[112:115]\n" \
  "s_waitcnt lgkmcnt(0)\n" \
  "s_waitcnt vmcnt(32)\n" \
  "v_mfma_f32_16x16x32_f16 v[0:3], a[160:163], v[192:195], v[0:3]\n" \
  "ds_read_b128 v[208:211], v251 offset:12288\n" \
  "v_mfma_f32_16x16x32_f16 v[4:7], a[160:163], v[196:199], v[4:7]\n" \
  "ds_read_b128 v[212:215], v251 offset:13312\n" \
  "v_mfma_f32_16x16x32_f16 v[8:11], a[160:163], v[200:203], v[8:11]\n" \
  "ds_read_b128 v[216:219], v251 offset:14336\n" \
  "v_mfma_f32_16x16x32_f16 v[12:15], a[160:163], v[204:207], v[12:15]\n" \
  "ds_read_b128 v[220:223], v251 offset:15360\n" \
  "s_waitcnt vmcnt(30)\n" \
  "v_mfma_f32_16x16x32_f16 v[44:47], a[164:167], v[204:207], v[44:47]\n" \
  "v_mfma_f32_16x16x32_f16 v[40:43], a[164:167], v[200:203], v[40:43]\n" \
  "v_mfma_f32_16x16x32_f16 v[36:39], a[164:167], v[196:199], v[36:39]\n" \
  "v_mfma_f32_16x16x32_f16 v[32:35], a[164:167], v[192:195], v[32:35]\n" \
  "s_waitcnt vmcnt(27)\n" \
  "v_mfma_f32_16x16x32_f16 v[64:67], a[168:171], v[192:195], v[64:67]\n" \
  "v_mfma_f32_16x16x32_f16 v[68:71], a[168:171], v[196:199], v[68:71]\n" \
  "v_mfma_f32_16x16x32_f16 v[72:75], a[168:171], v[200:203], v[72:75]\n" \
  "v_mfma_f32_16x16x32_f16 v[76:79], a[168:171], v[204:207], v[76:79]\n" \
  "s_waitcnt vmcnt(25)\n" \
  "v_mfma_f32_16x16x32_f16 v[108:111], a[172:175], v[204:207], v[108:111]\n" \
  "v_mfma_f32_16x16x32_f16 v[104:107], a[172:175], v[200:203], v[104:107]\n" \
  "v_mfma_f32_16x16x32_f16 v[100:103], a[172:175], v[196:199], v[100:103]\n" \
  "v_mfma_f32_16x16x32_f16 v[96:99], a[172:175], v[192:195], v[96:99]\n" \
  "s_waitcnt lgkmcnt(0)\n" \
  "s_barrier\n" \
  "v_mfma_f32_16x16x32_f16 v[16:19], a[160:163], v[208:211], v[16:19]\n" \
  "s_add_u32 s81, s81, 1\n" \
  "s_cmp_eq_u32 s81, 3\n" \
  "s_cselect_b32 s81, 0, s81\n" \
  "s_movk_i32 s74, 0x4000\n" \
  "s_cmov_b32 s74, 0xffff8000\n" \
  "v_mfma_f32_16x16x32_f16 v[20:23], a[160:163], v[212:215], v[20:23]\n" \
  "v_add_u32 v251, s74, v251\n" \
  "s_cmp_eq_u32 s81, 2\n" \
  "s_movk_i32 s74, 0x4000\n" \
  "s_cmov_b32 s74, 0xffff8000\n" \
  "v_mfma_f32_16x16x32_f16 v[24:27], a[160:163], v[216:219], v[24:27]\n" \
  "v_add_u32 v252, s74, v252\n" \
  "ds_read_b128 v[192:195], v251 offset:0\n" \
  "v_mfma_f32_16x16x32_f16 v[28:31], a[160:163], v[220:223], v[28:31]\n" \
  "ds_read_b128 v[196:199], v251 offset:1024\n" \
  "v_mfma_f32_16x16x32_f16 v[60:63], a[164:167], v[220:223], v[60:63]\n" \
  "ds_read_b128 v[200:203], v251 offset:2048\n" \
  "v_mfma_f32_16x16x32_f16 v[56:59], a[164:167], v[216:219], v[56:59]\n" \
  "ds_read_b128 v[204:207], v251 offset:3072\n" \
  "v_mfma_f32_16x16x32_f16 v[52:55], a[164:167], v[212:215], v[52:55]\n" \
  "v_mfma_f32_16x16x32_f16 v[48:51], a[164:167], v[208:211], v[48:51]\n" \
  "v_mfma_f32_16x16x32_f16 v[80:83], a[168:171], v[208:211], v[80:83]\n" \
  "v_mfma_f32_16x16x32_f16 v[84:87], a[168:171], v[212:215], v[84:87]\n" \
  "v_mfma_f32_16x16x32_f16 v[88:91], a[168:171], v[216:219], v[88:91]\n" \
  "v_mfma_f32_16x16x32_f16 v[92:95], a[168:171], v[220:223], v[92:95]\n" \
  "v_mfma_f32_16x16x32_f16 v[124:127], a[172:175], v[220:223], v[124:127]\n" \
  "v_mfma_f32_16x16x32_f16 v[120:123], a[172:175], v[216:219], v[120:123]\n" \
  "v_mfma_f32_16x16x32_f16 v[116:119], a[172:175], v[212:215], v[116:119]\n" \
  "v_mfma_f32_16x16x32_f16 v[112:115], a[172:175], v[208:211], v[112:115]\n" \
  "s_waitcnt lgkmcnt(0)\n" \
  "s_waitcnt vmcnt(8)\n" \
  "v_mfma_f32_16x16x32_f16 v[0:3], a[192:195], v[192:195], v[0:3]\n" \
  "ds_read_b128 v[208:211], v251 offset:4096\n" \
  "v_mfma_f32_16x16x32_f16 v[4:7], a[192:195], v[196:199], v[4:7]\n" \
  "ds_read_b128 v[212:215], v251 offset:5120\n" \
  "v_mfma_f32_16x16x32_f16 v[8:11], a[192:195], v[200:203], v[8:11]\n" \
  "ds_read_b128 v[216:219], v251 offset:6144\n" \
  "v_mfma_f32_16x16x32_f16 v[12:15], a[192:195], v[204:207], v[12:15]\n" \
  "ds_read_b128 v[220:223], v251 offset:7168\n" \
  "s_waitcnt vmcnt(7)\n" \
  "v_mfma_f32_16x16x32_f16 v[44:47], a[196:199], v[204:207], v[44:47]\n" \
  "v_mfma_f32_16x16x32_f16 v[40:43], a[196:199], v[200:203], v[40:43]\n" \
  "v_mfma_f32_16x16x32_f16 v[36:39], a[196:199], v[196:199], v[36:39]\n" \
  "v_mfma_f32_16x16x32_f16 v[32:35], a[196:199], v[192:195], v[32:35]\n" \
  "s_waitcnt vmcnt(6)\n" \
  "v_mfma_f32_16x16x32_f16 v[64:67], a[200:203], v[192:195], v[64:67]\n" \
  "v_mfma_f32_16x16x32_f16 v[68:71], a[200:203], v[196:199], v[68:71]\n" \
  "v_mfma_f32_16x16x32_f16 v[72:75], a[200:203], v[200:203], v[72:75]\n" \
  "v_mfma_f32_16x16x32_f16 v[76:79], a[200:203], v[204:207], v[76:79]\n" \
  "s_waitcnt vmcnt(5)\n" \
  "v_mfma_f32_16x16x32_f16 v[108:111], a[204:207], v[204:207], v[108:111]\n" \
  "v_mfma_f32_16x16x32_f16 v[104:107], a[204:207], v[200:203], v[104:107]\n" \
  "v_mfma_f32_16x16x32_f16 v[100:103], a[204:207], v[196:199], v[100:103]\n" \
  "v_mfma_f32_16x16x32_f16 v[96:99], a[204:207], v[192:195], v[96:99]\n" \
  "s_waitcnt lgkmcnt(0)\n" \
  "v_mfma_f32_16x16x32_f16 v[16:19], a[192:195], v[208:211], v[16:19]\n" \
  "s_load_dword s76, %6, s88\n" \
  "v_mfma_f32_16x16x32_f16 v[20:23], a[192:195], v[212:215], v[20:23]\n" \
  "v_mfma_f32_16x16x32_f16 v[24:27], a[192:195], v[216:219], v[24:27]\n" \
  "ds_read_b128 v[192:195], v251 offset:8192\n" \
  "v_mfma_f32_16x16x32_f16 v[28:31], a[192:195], v[220:223], v[28:31]\n" \
  "ds_read_b128 v[196:199], v251 offset:9216\n" \
  "v_mfma_f32_16x16x32_f16 v[60:63], a[196:199], v[220:223], v[60:63]\n" \
  "ds_read_b128 v[200:203], v251 offset:10240\n" \
  "v_mfma_f32_16x16x32_f16 v[56:59], a[196:199], v[216:219], v[56:59]\n" \
  "ds_read_b128 v[204:207], v251 offset:11264\n" \
  "v_mfma_f32_16x16x32_f16 v[52:55], a[196:199], v[212:215], v[52:55]\n" \
  "v_mfma_f32_16x16x32_f16 v[48:51], a[196:199], v[208:211], v[48:51]\n" \
  "v_mfma_f32_16x16x32_f16 v[80:83], a[200:203], v[208:211], v[80:83]\n" \
  "v_mfma_f32_16x16x32_f16 v[84:87], a[200:203], v[212:215], v[84:87]\n" \
  "v_mfma_f32_16x16x32_f16 v[88:91], a[200:203], v[216:219], v[88:91]\n" \
  "v_mfma_f32_16x16x32_f16 v[92:95], a[200:203], v[220:223], v[92:95]\n" \
  "v_mfma_f32_16x16x32_f16 v[124:127], a[204:207], v[220:223], v[124:127]\n" \
  "v_mfma_f32_16x16x32_f16 v[120:123], a[204:207], v[216:219], v[120:123]\n" \
  "v_mfma_f32_16x16x32_f16 v[116:119], a[204:207], v[212:215], v[116:119]\n" \
  "v_mfma_f32_16x16x32_f16 v[112:115], a[204:207], v[208:211], v[112:115]\n" \
  "s_waitcnt lgkmcnt(0)\n" \
  "s_cmp_lt_u32 s76, s82\n" \
  "s_cbranch_scc1 SLOW2_%=\n" \
  "OK1_%=:\n" \
  "s_lshr_b32 s76, s76, 31\n" \
  "s_or_b32 s89, s89, s76\n" \
  "s_waitcnt vmcnt(4)\n" \
  "v_mfma_f32_16x16x32_f16 v[0:3], a[224:227], v[192:195], v[0:3]\n" \
  "ds_read_b128 v[208:211], v251 offset:12288\n" \
  "s_add_u32 s40, s86, 0\n" \
  "buffer_load_dwordx4 a[0:3], v248, s[36:39], s40 offen offset:0 sc1 nt\n" \
  "buffer_load_dwordx4 a[4:7], v248, s[36:39], s40 offen offset:1024 sc1 nt\n" \
  "v_mfma_f32_16x16x32_f16 v[4:7], a[224:227], v[196:199], v[4:7]\n" \
  "ds_read_b128 v[212:215], v251 offset:13312\n" \
  "buffer_load_dwordx4 a[8:11], v248, s[36:39], s40 offen offset:2048 sc1 nt\n" \
  "v_mfma_f32_16x16x32_f16 v[8:11], a[224:227], v[200:203], v[8:11]\n" \
  "ds_read_b128 v[216:219], v251 offset:14336\n" \
  "buffer_load_dwordx4 a[12:15], v248, s[36:39], s40 offen offset:3072 sc1 nt\n" \
  "v_mfma_f32_16x16x32_f16 v[12:15], a[224:227], v[204:207], v[12:15]\n" \
  "ds_read_b128 v[220:223], v251 offset:15360\n" \
  "s_add_u32 s40, s86, 4096\n" \
  "buffer_load_dwordx4 a[16:19], v248, s[36:39], s40 offen offset:0 sc1 nt\n" \
  "s_waitcnt vmcnt(7)\n" \
  "v_mfma_f32_16x16x32_f16 v[44:47], a[228:231], v[204:207], v[44:47]\n" \
  "buffer_load_dwordx4 a[20:23], v248, s[36:39], s40 offen offset:1024 sc1 nt\n" \
  "v_mfma_f32_16x16x32_f16 v[40:43], a[228:231], v[200:203], v[40:43]\n" \
  "buffer_load_dwordx4 a[24:27], v248, s[36:39], s40 offen offset:2048 sc1 nt\n" \
  "v_mfma_f32_16x16x32_f16 v[36:39], a[228:231], v[196:199], v[36:39]\n" \
  "buffer_load_dwordx4 a[28:31], v248, s[36:39], s40 offen offset:3072 sc1 nt\n" \
  "v_mfma_f32_16x16x32_f16 v[32:35], a[228:231], v[192:195], v[32:35]\n" \
  "s_add_u32 s40, s86, 8192\n" \
  "buffer_load_dwordx4 a[32:35], v248, s[36:39], s40 offen offset:0 sc1 nt\n" \
  "s_waitcnt vmcnt(10)\n" \
  "v_mfma_f32_16x16x32_f16 v[64:67], a[232:235], v[192:195], v[64:67]\n" \
  "buffer_load_dwordx4 a[36:39], v248, s[36:39], s40 offen offset:1024 sc1 nt\n" \
  "v_mfma_f32_16x16x32_f16 v[68:71], a[232:235], v[196:199], v[68:71]\n" \
  "buffer_load_dwordx4 a[40:43], v248, s[36:39], s40 offen offset:2048 sc1 nt\n" \
  "v_mfma_f32_16x16x32_f16 v[72:75], a[232:235], v[200:203], v[72:75]\n" \
  "buffer_load_dwordx4 a[44:47], v248, s[36:39], s40 offen offset:3072 sc1 nt\n" \
  "v_mfma_f32_16x16x32_f16 v[76:79], a[232:235], v[204:207], v[76:79]\n" \
  "s_add_u32 s40, s86, 12288\n" \
  "buffer_load_dwordx4 a[48:51], v248, s[36:39], s40 offen offset:0 sc1 nt\n" \
  "s_waitcnt vmcnt(13)\n" \
  "v_mfma_f32_16x16x32_f16 v[108:111], a[236:239], v[204:207], v[108:111]\n" \
  "buffer_load_dwordx4 a[52:55], v248, s[36:39], s40 offen offset:1024 sc1 nt\n" \
  "v_mfma_f32_16x16x32_f16 v[104:107], a[236:239], v[200:203], v[104:107]\n" \
  "buffer_load_dwordx4 a[56:59], v248, s[36:39], s40 offen offset:2048 sc1 nt\n" \
  "v_mfma_f32_16x16x32_f16 v[100:103], a[236:239], v[196:199], v[100:103]\n" \
  "buffer_load_dwordx4 a[60:63], v248, s[36:39], s40 offen offset:3072 sc1 nt\n" \
  "v_mfma_f32_16x16x32_f16 v[96:99], a[236:239], v[192:195], v[96:99]\n" \
  "s_waitcnt lgkmcnt(0)\n" \
  "s_add_u32 s40, s86, 16384\n" \
  "buffer_load_dwordx4 a[64:67], v248, s[36:39], s40 offen offset:0 sc1 nt\n" \
  "buffer_load_dwordx4 a[68:71], v248, s[36:39], s40 offen offset:1024 sc1 nt\n" \
  "v_mfma_f32_16x16x32_f16 v[16:19], a[224:227], v[208:211], v[16:19]\n" \
  "buffer_load_dwordx4 a[72:75], v248, s[36:39], s40 offen offset:2048 sc1 nt\n" \
  "v_mfma_f32_16x16x32_f16 v[20:23], a[224:227], v[212:215], v[20:23]\n" \
  "buffer_load_dwordx4 a[76:79], v248, s[36:39], s40 offen offset:3072 sc1 nt\n" \
  "v_mfma_f32_16x16x32_f16 v[24:27], a[224:227], v[216:219], v[24:27]\n" \
  "s_add_u32 s40, s86, 20480\n" \
  "buffer_load_dwordx4 a[80:83], v248, s[36:39], s40 offen offset:0 sc1 nt\n" \
  "v_mfma_f32_16x16x32_f16 v[28:31], a[224:227], v[220:223], v[28:31]\n" \
  "buffer_load_dwordx4 a[84:87], v248, s[36:39], s40 offen offset:1024 sc1 nt\n" \
  "v_mfma_f32_16x16x32_f16 v[60:63], a[228:231], v[220:223], v[60:63]\n" \
  "buffer_load_dwordx4 a[88:91], v248, s[36:39], s40 offen offset:2048 sc1 nt\n" \
  "v_mfma_f32_16x16x32_f16 v[56:59], a[228:231], v[216:219], v[56:59]\n" \
  "buffer_load_dwordx4 a[92:95], v248, s[36:39], s40 offen offset:3072 sc1 nt\n" \
  "v_mfma_f32_16x16x32_f16 v[52:55], a[228:231], v[212:215], v[52:55]\n" \
  "s_add_u32 s40, s86, 24576\n" \
  "buffer_load_dwordx4 a[96:99], v248, s[36:39], s40 offen offset:0 sc1 nt\n" \
  "v_mfma_f32_16x16x32_f16 v[48:51], a[228:231], v[208:211], v[48:51]\n" \
  "buffer_load_dwordx4 a[100:103], v248, s[36:39], s40 offen offset:1024 sc1 nt\n" \
  "v_mfma_f32_16x16x32_f16 v[80:83], a[232:235], v[208:211], v[80:83]\n" \
  "buffer_load_dwordx4 a[104:107], v248, s[36:39], s40 offen offset:2048 sc1 nt\n" \
  "v_mfma_f32_16x16x32_f16 v[84:87], a[232:235], v[212:215], v[84:87]\n" \
  "buffer_load_dwordx4 a[108:111], v248, s[36:39], s40 offen offset:3072 sc1 nt\n" \
  "v_mfma_f32_16x16x32_f16 v[88:91], a[232:235], v[216:219], v[88:91]\n" \
  "s_add_u32 s40, s86, 28672\n" \
  "buffer_load_dwordx4 a[112:115], v248, s[36:39], s40 offen offset:0 sc1 nt\n" \
  "v_mfma_f32_16x16x32_f16 v[92:95], a[232:235], v[220:223], v[92:95]\n" \
  "buffer_load_dwordx4 a[116:119], v248, s[36:39], s40 offen offset:1024 sc1 nt\n" \
  "v_mfma_f32_16x16x32_f16 v[124:127], a[236:239], v[220:223], v[124:127]\n" \
  "buffer_load_dwordx4 a[120:123], v248, s[36:39], s40 offen offset:2048 sc1 nt\n" \
  "v_mfma_f32_16x16x32_f16 v[120:123], a[236:239], v[216:219], v[120:123]\n" \
  "buffer_load_dwordx4 a[124:127], v248, s[36:39], s40 offen offset:3072 sc1 nt\n" \
  "v_mfma_f32_16x16x32_f16 v[116:119], a[236:239], v[212:215], v[116:119]\n" \
  "v_mfma_f32_16x16x32_f16 v[112:115], a[236:239], v[208:211], v[112:115]\n" \
  "s_waitcnt lgkmcnt(0)\n" \
  "s_add_u32 s67, s67, 1\n" \
  "s_nop 7\n" \
  "s_mov_b64 s[32:33], %4\n" \
  "s_mov_b32 s34, 0x2000000\n" \
  "s_mov_b32 s35, 0x20000\n" \
  "v_lshrrev_b32 v241, 8, v248\n" \
  "v_and_b32 v242, 0xf0, v248\n" \
  "s_and_b32 s72, %7, 7\n" \
  "s_lshl_b32 s72, s72, 12\n" \
  "s_bfe_u32 s73, %7, 0x20004\n" \
  "s_lshl_b32 s73, s73, 10\n" \
  "s_add_u32 s72, s72, s73\n" \
  "s_bfe_u32 s73, %7, 0x10003\n" \
  "s_lshl_b32 s73, s73, 8\n" \
  "s_add_u32 s72, s72, s73\n" \
  "s_lshl_b32 s73, 0, 9\n" \
  "s_add_u32 s72, s72, s73\n" \
  "v_add_u32 v240, s72, v242\n" \
  "v_lshl_add_u32 v240, v241, 17, v240\n" \
  "s_and_b32 s72, %7, 7\n" \
  "s_lshl_b32 s72, s72, 12\n" \
  "s_bfe_u32 s73, %7, 0x20004\n" \
  "s_lshl_b32 s73, s73, 10\n" \
  "s_add_u32 s72, s72, s73\n" \
  "s_bfe_u32 s73, %7, 0x10003\n" \
  "s_lshl_b32 s73, s73, 8\n" \
  "s_add_u32 s72, s72, s73\n" \
  "s_lshl_b32 s73, 1, 9\n" \
  "s_add_u32 s72, s72, s73\n" \
  "v_add_u32 v243, s72, v242\n" \
  "v_lshl_add_u32 v243, v241, 17, v243\n" \
  "s_bfe_u32 s72, %7, 0x10006\n" \
  "s_lshl_b32 s72, s72, 2\n" \
  "s_add_u32 s72, s72, %8\n" \
  "s_lshl_b32 s72, s72, 1\n" \
  "s_lshr_b32 s73, %7, 7\n" \
  "s_add_u32 s72, s72, s73\n" \
  "s_lshl_b32 s78, s72, 21\n" \
  "s_mov_b32 s74, 0xbfb8aa3b\n" \
  "s_mov_b32 s75, 0x42ad1f97\n" \
  "s_cmp_lg_u32 s89, 0\n" \
  "s_cselect_b32 s76, 0x7fc00000, 1.0\n" \
  "s_waitcnt vmcnt(28)\n" \
  "v_accvgpr_read_b32 v132, a0\n" \
  "v_accvgpr_read_b32 v133, a4\n" \
  "v_accvgpr_read_b32 v134, a8\n" \
  "v_accvgpr_read_b32 v135, a12\n" \
  "v_add_f32 v128, v0, v132\n" \
  "v_add_f32 v129, v4, v133\n" \
  "v_add_f32 v130, v8, v134\n" \
  "v_add_f32 v131, v12, v135\n" \
  "v_mul_f32 v132, s74, v128\n" \
  "v_mul_f32 v133, s74, v129\n" \
  "v_mul_f32 v134, s74, v130\n" \
  "v_mul_f32 v135, s74, v131\n" \
  "v_med3_f32 v132, v132, -s75, s75\n" \
  "v_med3_f32 v133, v133, -s75, s75\n" \
  "v_med3_f32 v134, v134, -s75, s75\n" \
  "v_med3_f32 v135, v135, -s75, s75\n" \
  "v_exp_f32 v132, v132\n" \
  "v_exp_f32 v133, v133\n" \
  "v_exp_f32 v134, v134\n" \
  "v_exp_f32 v135, v135\n" \
  "v_add_f32 v132, s76, v132\n" \
  "v_add_f32 v133, s76, v133\n" \
  "v_add_f32 v134, s76, v134\n" \
  "v_add_f32 v135, s76, v135\n" \
  "v_rcp_f32 v132, v132\n" \
  "v_rcp_f32 v133, v133\n" \
  "v_rcp_f32 v134, v134\n" \
  "v_rcp_f32 v135, v135\n" \
  "v_mul_f32 v128, v128, v132\n" \
  "v_mul_f32 v129, v129, v133\n" \
  "v_mul_f32 v130, v130, v134\n" \
  "v_mul_f32 v131, v131, v135\n" \
  "s_add_u32 s72, s78, 0\n" \
  "buffer_store_dwordx4 v[128:131], v240, s[32:35], s72 offen sc0 sc1\n" \
  "v_accvgpr_read_b32 v140, a1\n" \
  "v_accvgpr_read_b32 v141, a5\n" \
  "v_accvgpr_read_b32 v142, a9\n" \
  "v_accvgpr_read_b32 v143, a13\n" \
  "v_add_f32 v136, v1, v140\n" \
  "v_add_f32 v137, v5, v141\n" \
  "v_add_f32 v138, v9, v142\n" \
  "v_add_f32 v139, v13, v143\n" \
  "v_mul_f32 v140, s74, v136\n" \
  "v_mul_f32 v141, s74, v137\n" \
  "v_mul_f32 v142, s74, v138\n" \
  "v_mul_f32 v143, s74, v139\n" \
  "v_med3_f32 v140, v140, -s75, s75\n" \
  "v_med3_f32 v141, v141, -s75, s75\n" \
  "v_med3_f32 v142, v142, -s75, s75\n" \
  "v_med3_f32 v143, v143, -s75, s75\n" \
  "v_exp_f32 v140, v140\n" \
  "v_exp_f32 v141, v141\n" \
  "v_exp_f32 v142, v142\n" \
  "v_exp_f32 v143, v143\n" \
  "v_add_f32 v140, s76, v140\n" \
  "v_add_f32 v141, s76, v141\n" \
  "v_add_f32 v142, s76, v142\n" \
  "v_add_f32 v143, s76, v143\n" \
  "v_rcp_f32 v140, v140\n" \
  "v_rcp_f32 v141, v141\n" \
  "v_rcp_f32 v142, v142\n" \
  "v_rcp_f32 v143, v143\n" \
  "v_mul_f32 v136, v136, v140\n" \
  "v_mul_f32 v137, v137, v141\n" \
  "v_mul_f32 v138, v138, v142\n" \
  "v_mul_f32 v139, v139, v143\n" \
  "s_add_u32 s72, s78, 32768\n" \
  "buffer_store_dwordx4 v[136:139], v240, s[32:35], s72 offen sc0 sc1\n" \
  "v_accvgpr_read_b32 v148, a2\n" \
  "v_accvgpr_read_b32 v149, a6\n" \
  "v_accvgpr_read_b32 v150, a10\n" \
  "v_accvgpr_read_b32 v151, a14\n" \
  "v_add_f32 v144, v2, v148\n" \
  "v_add_f32 v145, v6, v149\n" \
  "v_add_f32 v146, v10, v150\n" \
  "v_add_f32 v147, v14, v151\n" \
  "v_mul_f32 v148, s74, v144\n" \
  "v_mul_f32 v149, s74, v145\n" \
  "v_mul_f32 v150, s74, v146\n" \
  "v_mul_f32 v151, s74, v147\n" \
  "v_med3_f32 v148, v148, -s75, s75\n" \
  "v_med3_f32 v149, v149, -s75, s75\n" \
  "v_med3_f32 v150, v150, -s75, s75\n" \
  "v_med3_f32 v151, v151, -s75, s75\n" \
  "v_exp_f32 v148, v148\n" \
  "v_exp_f32 v149, v149\n" \
  "v_exp_f32 v150, v150\n" \
  "v_exp_f32 v151, v151\n" \
  "v_add_f32 v148, s76, v148\n" \
  "v_add_f32 v149, s76, v149\n" \
  "v_add_f32 v150, s76, v150\n" \
  "v_add_f32 v151, s76, v151\n" \
  "v_rcp_f32 v148, v148\n" \
  "v_rcp_f32 v149, v149\n" \
  "v_rcp_f32 v150, v150\n" \
  "v_rcp_f32 v151, v151\n" \
  "v_mul_f32 v144, v144, v148\n" \
  "v_mul_f32 v145, v145, v149\n" \
  "v_mul_f32 v146, v146, v150\n" \
  "v_mul_f32 v147, v147, v151\n" \
  "s_add_u32 s72, s78, 65536\n" \
  "buffer_store_dwordx4 v[144:147], v240, s[32:35], s72 offen sc0 sc1\n" \
  "v_accvgpr_read_b32 v156, a3\n" \
  "v_accvgpr_read_b32 v157, a7\n" \
  "v_accvgpr_read_b32 v158, a11\n" \
  "v_accvgpr_read_b32 v159, a15\n" \
  "v_add_f32 v152, v3, v156\n" \
  "v_add_f32 v153, v7, v157\n" \
  "v_add_f32 v154, v11, v158\n" \
  "v_add_f32 v155, v15, v159\n" \
  "v_mul_f32 v156, s74, v152\n" \
  "v_mul_f32 v157, s74, v153\n" \
  "v_mul_f32 v158, s74, v154\n" \
  "v_mul_f32 v159, s74, v155\n" \
  "v_med3_f32 v156, v156, -s75, s75\n" \
  "v_med3_f32 v157, v157, -s75, s75\n" \
  "v_med3_f32 v158, v158, -s75, s75\n" \
  "v_med3_f32 v159, v159, -s75, s75\n" \
  "v_exp_f32 v156, v156\n" \
  "v_exp_f32 v157, v157\n" \
  "v_exp_f32 v158, v158\n" \
  "v_exp_f32 v159, v159\n" \
  "v_add_f32 v156, s76, v156\n" \
  "v_add_f32 v157, s76, v157\n" \
  "v_add_f32 v158, s76, v158\n" \
  "v_add_f32 v159, s76, v159\n" \
  "v_rcp_f32 v156, v156\n" \
  "v_rcp_f32 v157, v157\n" \
  "v_rcp_f32 v158, v158\n" \
  "v_rcp_f32 v159, v159\n" \
  "v_mul_f32 v152, v152, v156\n" \
  "v_mul_f32 v153, v153, v157\n" \
  "v_mul_f32 v154, v154, v158\n" \
  "v_mul_f32 v155, v155, v159\n" \
  "s_add_u32 s72, s78, 98304\n" \
  "buffer_store_dwordx4 v[152:155], v240, s[32:35], s72 offen sc0 sc1\n" \
  "s_waitcnt vmcnt(28)\n" \
  "v_accvgpr_read_b32 v164, a16\n" \
  "v_accvgpr_read_b32 v165, a20\n" \
  "v_accvgpr_read_b32 v166, a24\n" \
  "v_accvgpr_read_b32 v167, a28\n" \
  "v_add_f32 v160, v16, v164\n" \
  "v_add_f32 v161, v20, v165\n" \
  "v_add_f32 v162, v24, v166\n" \
  "v_add_f32 v163, v28, v167\n" \
  "v_mul_f32 v164, s74, v160\n" \
  "v_mul_f32 v165, s74, v161\n" \
  "v_mul_f32 v166, s74, v162\n" \
  "v_mul_f32 v167, s74, v163\n" \
  "v_med3_f32 v164, v164, -s75, s75\n" \
  "v_med3_f32 v165, v165, -s75, s75\n" \
  "v_med3_f32 v166, v166, -s75, s75\n" \
  "v_med3_f32 v167, v167, -s75, s75\n" \
  "v_exp_f32 v164, v164\n" \
  "v_exp_f32 v165, v165\n" \
  "v_exp_f32 v166, v166\n" \
  "v_exp_f32 v167, v167\n" \
  "v_add_f32 v164, s76, v164\n" \
  "v_add_f32 v165, s76, v165\n" \
  "v_add_f32 v166, s76, v166\n" \
  "v_add_f32 v167, s76, v167\n" \
  "v_rcp_f32 v164, v164\n" \
  "v_rcp_f32 v165, v165\n" \
  "v_rcp_f32 v166, v166\n" \
  "v_rcp_f32 v167, v167\n" \
  "v_mul_f32 v160, v160, v164\n" \
  "v_mul_f32 v161, v161, v165\n" \
  "v_mul_f32 v162, v162, v166\n" \
  "v_mul_f32 v163, v163, v167\n" \
  "s_add_u32 s72, s78, 0\n" \
  "buffer_store_dwordx4 v[160:163], v243, s[32:35], s72 offen sc0 sc1\n" \
  "v_accvgpr_read_b32 v172, a17\n" \
  "v_accvgpr_read_b32 v173, a21\n" \
  "v_accvgpr_read_b32 v174, a25\n" \
  "v_accvgpr_read_b32 v175, a29\n" \
  "v_add_f32 v168, v17, v172\n" \
  "v_add_f32 v169, v21, v173\n" \
  "v_add_f32 v170, v25, v174\n" \
  "v_add_f32 v171, v29, v175\n" \
  "v_mul_f32 v172, s74, v168\n" \
  "v_mul_f32 v173, s74, v169\n" \
  "v_mul_f32 v174, s74, v170\n" \
  "v_mul_f32 v175, s74, v171\n" \
  "v_med3_f32 v172, v172, -s75, s75\n" \
  "v_med3_f32 v173, v173, -s75, s75\n" \
  "v_med3_f32 v174, v174, -s75, s75\n" \
  "v_med3_f32 v175, v175, -s75, s75\n" \
  "v_exp_f32 v172, v172\n" \
  "v_exp_f32 v173, v173\n" \
  "v_exp_f32 v174, v174\n" \
  "v_exp_f32 v175, v175\n" \
  "v_add_f32 v172, s76, v172\n" \
  "v_add_f32 v173, s76, v173\n" \
  "v_add_f32 v174, s76, v174\n" \
  "v_add_f32 v175, s76, v175\n" \
  "v_rcp_f32 v172, v172\n" \
  "v_rcp_f32 v173, v173\n" \
  "v_rcp_f32 v174, v174\n" \
  "v_rcp_f32 v175, v175\n" \
  "v_mul_f32 v168, v168, v172\n" \
  "v_mul_f32 v169, v169, v173\n" \
  "v_mul_f32 v170, v170, v174\n" \
  "v_mul_f32 v171, v171, v175\n" \
  "s_add_u32 s72, s78, 32768\n" \
  "buffer_store_dwordx4 v[168:171], v243, s[32:35], s72 offen sc0 sc1\n" \
  "v_accvgpr_read_b32 v180, a18\n" \
  "v_accvgpr_read_b32 v181, a22\n" \
  "v_accvgpr_read_b32 v182, a26\n" \
  "v_accvgpr_read_b32 v183, a30\n" \
  "v_add_f32 v176, v18, v180\n" \
  "v_add_f32 v177, v22, v181\n" \
  "v_add_f32 v178, v26, v182\n" \
  "v_add_f32 v179, v30, v183\n" \
  "v_mul_f32 v180, s74, v176\n" \
  "v_mul_f32 v181, s74, v177\n" \
  "v_mul_f32 v182, s74, v178\n" \
  "v_mul_f32 v183, s74, v179\n" \
  "v_med3_f32 v180, v180, -s75, s75\n" \
  "v_med3_f32 v181, v181, -s75, s75\n" \
  "v_med3_f32 v182, v182, -s75, s75\n" \
  "v_med3_f32 v183, v183, -s75, s75\n" \
  "v_exp_f32 v180, v180\n" \
  "v_exp_f32 v181, v181\n" \
  "v_exp_f32 v182, v182\n" \
  "v_exp_f32 v183, v183\n" \
  "v_add_f32 v180, s76, v180\n" \
  "v_add_f32 v181, s76, v181\n" \
  "v_add_f32 v182, s76, v182\n" \
  "v_add_f32 v183, s76, v183\n" \
  "v_rcp_f32 v180, v180\n" \
  "v_rcp_f32 v181, v181\n" \
  "v_rcp_f32 v182, v182\n" \
  "v_rcp_f32 v183, v183\n" \
  "v_mul_f32 v176, v176, v180\n" \
  "v_mul_f32 v177, v177, v181\n" \
  "v_mul_f32 v178, v178, v182\n" \
  "v_mul_f32 v179, v179, v183\n" \
  "s_add_u32 s72, s78, 65536\n" \
  "buffer_store_dwordx4 v[176:179], v243, s[32:35], s72 offen sc0 sc1\n" \
  "v_accvgpr_read_b32 v188, a19\n" \
  "v_accvgpr_read_b32 v189, a23\n" \
  "v_accvgpr_read_b32 v190, a27\n" \
  "v_accvgpr_read_b32 v191, a31\n" \
  "v_add_f32 v184, v19, v188\n" \
  "v_add_f32 v185, v23, v189\n" \
  "v_add_f32 v186, v27, v190\n" \
  "v_add_f32 v187, v31, v191\n" \
  "v_mul_f32 v188, s74, v184\n" \
  "v_mul_f32 v189, s74, v185\n" \
  "v_mul_f32 v190, s74, v186\n" \
  "v_mul_f32 v191, s74, v187\n" \
  "v_med3_f32 v188, v188, -s75, s75\n" \
  "v_med3_f32 v189, v189, -s75, s75\n" \
  "v_med3_f32 v190, v190, -s75, s75\n" \
  "v_med3_f32 v191, v191, -s75, s75\n" \
  "v_exp_f32 v188, v188\n" \
  "v_exp_f32 v189, v189\n" \
  "v_exp_f32 v190, v190\n" \
  "v_exp_f32 v191, v191\n" \
  "v_add_f32 v188, s76, v188\n" \
  "v_add_f32 v189, s76, v189\n" \
  "v_add_f32 v190, s76, v190\n" \
  "v_add_f32 v191, s76, v191\n" \
  "v_rcp_f32 v188, v188\n" \
  "v_rcp_f32 v189, v189\n" \
  "v_rcp_f32 v190, v190\n" \
  "v_rcp_f32 v191, v191\n" \
  "v_mul_f32 v184, v184, v188\n" \
  "v_mul_f32 v185, v185, v189\n" \
  "v_mul_f32 v186, v186, v190\n" \
  "v_mul_f32 v187, v187, v191\n" \
  "s_add_u32 s72, s78, 98304\n" \
  "buffer_store_dwordx4 v[184:187], v243, s[32:35], s72 offen sc0 sc1\n" \
  "s_waitcnt vmcnt(28)\n" \
  "v_accvgpr_read_b32 v132, a32\n" \
  "v_accvgpr_read_b32 v133, a36\n" \
  "v_accvgpr_read_b32 v134, a40\n" \
  "v_accvgpr_read_b32 v135, a44\n" \
  "v_add_f32 v128, v32, v132\n" \
  "v_add_f32 v129, v36, v133\n" \
  "v_add_f32 v130, v40, v134\n" \
  "v_add_f32 v131, v44, v135\n" \
  "v_mul_f32 v132, s74, v128\n" \
  "v_mul_f32 v133, s74, v129\n" \
  "v_mul_f32 v134, s74, v130\n" \
  "v_mul_f32 v135, s74, v131\n" \
  "v_med3_f32 v132, v132, -s75, s75\n" \
  "v_med3_f32 v133, v133, -s75, s75\n" \
  "v_med3_f32 v134, v134, -s75, s75\n" \
  "v_med3_f32 v135, v135, -s75, s75\n" \
  "v_exp_f32 v132, v132\n" \
  "v_exp_f32 v133, v133\n" \
  "v_exp_f32 v134, v134\n" \
  "v_exp_f32 v135, v135\n" \
  "v_add_f32 v132, s76, v132\n" \
  "v_add_f32 v133, s76, v133\n" \
  "v_add_f32 v134, s76, v134\n" \
  "v_add_f32 v135, s76, v135\n" \
  "v_rcp_f32 v132, v132\n" \
  "v_rcp_f32 v133, v133\n" \
  "v_rcp_f32 v134, v134\n" \
  "v_rcp_f32 v135, v135\n" \
  "v_mul_f32 v128, v128, v132\n" \
  "v_mul_f32 v129, v129, v133\n" \
  "v_mul_f32 v130, v130, v134\n" \
  "v_mul_f32 v131, v131, v135\n" \
  "s_add_u32 s72, s78, 524288\n" \
  "buffer_store_dwordx4 v[128:131], v240, s[32:35], s72 offen sc0 sc1\n" \
  "v_accvgpr_read_b32 v140, a33\n" \
  "v_accvgpr_read_b32 v141, a37\n" \
  "v_accvgpr_read_b32 v142, a41\n" \
  "v_accvgpr_read_b32 v143, a45\n" \
  "v_add_f32 v136, v33, v140\n" \
  "v_add_f32 v137, v37, v141\n" \
  "v_add_f32 v138, v41, v142\n" \
  "v_add_f32 v139, v45, v143\n" \
  "v_mul_f32 v140, s74, v136\n" \
  "v_mul_f32 v141, s74, v137\n" \
  "v_mul_f32 v142, s74, v138\n" \
  "v_mul_f32 v143, s74, v139\n" \
  "v_med3_f32 v140, v140, -s75, s75\n" \
  "v_med3_f32 v141, v141, -s75, s75\n" \
  "v_med3_f32 v142, v142, -s75, s75\n" \
  "v_med3_f32 v143, v143, -s75, s75\n" \
  "v_exp_f32 v140, v140\n" \
  "v_exp_f32 v141, v141\n" \
  "v_exp_f32 v142, v142\n" \
  "v_exp_f32 v143, v143\n" \
  "v_add_f32 v140, s76, v140\n" \
  "v_add_f32 v141, s76, v141\n" \
  "v_add_f32 v142, s76, v142\n" \
  "v_add_f32 v143, s76, v143\n" \
  "v_rcp_f32 v140, v140\n" \
  "v_rcp_f32 v141, v141\n" \
  "v_rcp_f32 v142, v142\n" \
  "v_rcp_f32 v143, v143\n" \
  "v_mul_f32 v136, v136, v140\n" \
  "v_mul_f32 v137, v137, v141\n" \
  "v_mul_f32 v138, v138, v142\n" \
  "v_mul_f32 v139, v139, v143\n" \
  "s_add_u32 s72, s78, 557056\n" \
  "buffer_store_dwordx4 v[136:139], v240, s[32:35], s72 offen sc0 sc1\n" \
  "v_accvgpr_read_b32 v148, a34\n" \
  "v_accvgpr_read_b32 v149, a38\n" \
  "v_accvgpr_read_b32 v150, a42\n" \
  "v_accvgpr_read_b32 v151, a46\n" \
  "v_add_f32 v144, v34, v148\n" \
  "v_add_f32 v145, v38, v149\n" \
  "v_add_f32 v146, v42, v150\n" \
  "v_add_f32 v147, v46, v151\n" \
  "v_mul_f32 v148, s74, v144\n" \
  "v_mul_f32 v149, s74, v145\n" \
  "v_mul_f32 v150, s74, v146\n" \
  "v_mul_f32 v151, s74, v147\n" \
  "v_med3_f32 v148, v148, -s75, s75\n" \
  "v_med3_f32 v149, v149, -s75, s75\n" \
  "v_med3_f32 v150, v150, -s75, s75\n" \
  "v_med3_f32 v151, v151, -s75, s75\n" \
  "v_exp_f32 v148, v148\n" \
  "v_exp_f32 v149, v149\n" \
  "v_exp_f32 v150, v150\n" \
  "v_exp_f32 v151, v151\n" \
  "v_add_f32 v148, s76, v148\n" \
  "v_add_f32 v149, s76, v149\n" \
  "v_add_f32 v150, s76, v150\n" \
  "v_add_f32 v151, s76, v151\n" \
  "v_rcp_f32 v148, v148\n" \
  "v_rcp_f32 v149, v149\n" \
  "v_rcp_f32 v150, v150\n" \
  "v_rcp_f32 v151, v151\n" \
  "v_mul_f32 v144, v144, v148\n" \
  "v_mul_f32 v145, v145, v149\n" \
  "v_mul_f32 v146, v146, v150\n" \
  "v_mul_f32 v147, v147, v151\n" \
  "s_add_u32 s72, s78, 589824\n" \
  "buffer_store_dwordx4 v[144:147], v240, s[32:35], s72 offen sc0 sc1\n" \
  "v_accvgpr_read_b32 v156, a35\n" \
  "v_accvgpr_read_b32 v157, a39\n" \
  "v_accvgpr_read_b32 v158, a43\n" \
  "v_accvgpr_read_b32 v159, a47\n" \
  "v_add_f32 v152, v35, v156\n" \
  "v_add_f32 v153, v39, v157\n" \
  "v_add_f32 v154, v43, v158\n" \
  "v_add_f32 v155, v47, v159\n" \
  "v_mul_f32 v156, s74, v152\n" \
  "v_mul_f32 v157, s74, v153\n" \
  "v_mul_f32 v158, s74, v154\n" \
  "v_mul_f32 v159, s74, v155\n" \
  "v_med3_f32 v156, v156, -s75, s75\n" \
  "v_med3_f32 v157, v157, -s75, s75\n" \
  "v_med3_f32 v158, v158, -s75, s75\n" \
  "v_med3_f32 v159, v159, -s75, s75\n" \
  "v_exp_f32 v156, v156\n" \
  "v_exp_f32 v157, v157\n" \
  "v_exp_f32 v158, v158\n" \
  "v_exp_f32 v159, v159\n" \
  "v_add_f32 v156, s76, v156\n" \
  "v_add_f32 v157, s76, v157\n" \
  "v_add_f32 v158, s76, v158\n" \
  "v_add_f32 v159, s76, v159\n" \
  "v_rcp_f32 v156, v156\n" \
  "v_rcp_f32 v157, v157\n" \
  "v_rcp_f32 v158, v158\n" \
  "v_rcp_f32 v159, v159\n" \
  "v_mul_f32 v152, v152, v156\n" \
  "v_mul_f32 v153, v153, v157\n" \
  "v_mul_f32 v154, v154, v158\n" \
  "v_mul_f32 v155, v155, v159\n" \
  "s_add_u32 s72, s78, 622592\n" \
  "buffer_store_dwordx4 v[152:155], v240, s[32:35], s72 offen sc0 sc1\n" \
  "s_waitcnt vmcnt(28)\n" \
  "v_accvgpr_read_b32 v164, a48\n" \
  "v_accvgpr_read_b32 v165, a52\n" \
  "v_accvgpr_read_b32 v166, a56\n" \
  "v_accvgpr_read_b32 v167, a60\n" \
  "v_add_f32 v160, v48, v164\n" \
  "v_add_f32 v161, v52, v165\n" \
  "v_add_f32 v162, v56, v166\n" \
  "v_add_f32 v163, v60, v167\n" \
  "v_mul_f32 v164, s74, v160\n" \
  "v_mul_f32 v165, s74, v161\n" \
  "v_mul_f32 v166, s74, v162\n" \
  "v_mul_f32 v167, s74, v163\n" \
  "v_med3_f32 v164, v164, -s75, s75\n" \
  "v_med3_f32 v165, v165, -s75, s75\n" \
  "v_med3_f32 v166, v166, -s75, s75\n" \
  "v_med3_f32 v167, v167, -s75, s75\n" \
  "v_exp_f32 v164, v164\n" \
  "v_exp_f32 v165, v165\n" \
  "v_exp_f32 v166, v166\n" \
  "v_exp_f32 v167, v167\n" \
  "v_add_f32 v164, s76, v164\n" \
  "v_add_f32 v165, s76, v165\n" \
  "v_add_f32 v166, s76, v166\n" \
  "v_add_f32 v167, s76, v167\n" \
  "v_rcp_f32 v164, v164\n" \
  "v_rcp_f32 v165, v165\n" \
  "v_rcp_f32 v166, v166\n" \
  "v_rcp_f32 v167, v167\n" \
  "v_mul_f32 v160, v160, v164\n" \
  "v_mul_f32 v161, v161, v165\n" \
  "v_mul_f32 v162, v162, v166\n" \
  "v_mul_f32 v163, v163, v167\n" \
  "s_add_u32 s72, s78, 524288\n" \
  "buffer_store_dwordx4 v[160:163], v243, s[32:35], s72 offen sc0 sc1\n" \
  "v_accvgpr_read_b32 v172, a49\n" \
  "v_accvgpr_read_b32 v173, a53\n" \
  "v_accvgpr_read_b32 v174, a57\n" \
  "v_accvgpr_read_b32 v175, a61\n" \
  "v_add_f32 v168, v49, v172\n" \
  "v_add_f32 v169, v53, v173\n" \
  "v_add_f32 v170, v57, v174\n" \
  "v_add_f32 v171, v61, v175\n" \
  "v_mul_f32 v172, s74, v168\n" \
  "v_mul_f32 v173, s74, v169\n" \
  "v_mul_f32 v174, s74, v170\n" \
  "v_mul_f32 v175, s74, v171\n" \
  "v_med3_f32 v172, v172, -s75, s75\n" \
  "v_med3_f32 v173, v173, -s75, s75\n" \
  "v_med3_f32 v174, v174, -s75, s75\n" \
  "v_med3_f32 v175, v175, -s75, s75\n" \
  "v_exp_f32 v172, v172\n" \
  "v_exp_f32 v173, v173\n" \
  "v_exp_f32 v174, v174\n" \
  "v_exp_f32 v175, v175\n" \
  "v_add_f32 v172, s76, v172\n" \
  "v_add_f32 v173, s76, v173\n" \
  "v_add_f32 v174, s76, v174\n" \
  "v_add_f32 v175, s76, v175\n" \
  "v_rcp_f32 v172, v172\n" \
  "v_rcp_f32 v173, v173\n" \
  "v_rcp_f32 v174, v174\n" \
  "v_rcp_f32 v175, v175\n" \
  "v_mul_f32 v168, v168, v172\n" \
  "v_mul_f32 v169, v169, v173\n" \
  "v_mul_f32 v170, v170, v174\n" \
  "v_mul_f32 v171, v171, v175\n" \
  "s_add_u32 s72, s78, 557056\n" \
  "buffer_store_dwordx4 v[168:171], v243, s[32:35], s72 offen sc0 sc1\n" \
  "v_accvgpr_read_b32 v180, a50\n" \
  "v_accvgpr_read_b32 v181, a54\n" \
  "v_accvgpr_read_b32 v182, a58\n" \
  "v_accvgpr_read_b32 v183, a62\n" \
  "v_add_f32 v176, v50, v180\n" \
  "v_add_f32 v177, v54, v181\n" \
  "v_add_f32 v178, v58, v182\n" \
  "v_add_f32 v179, v62, v183\n" \
  "v_mul_f32 v180, s74, v176\n" \
  "v_mul_f32 v181, s74, v177\n" \
  "v_mul_f32 v182, s74, v178\n" \
  "v_mul_f32 v183, s74, v179\n" \
  "v_med3_f32 v180, v180, -s75, s75\n" \
  "v_med3_f32 v181, v181, -s75, s75\n" \
  "v_med3_f32 v182, v182, -s75, s75\n" \
  "v_med3_f32 v183, v183, -s75, s75\n" \
  "v_exp_f32 v180, v180\n" \
  "v_exp_f32 v181, v181\n" \
  "v_exp_f32 v182, v182\n" \
  "v_exp_f32 v183, v183\n" \
  "v_add_f32 v180, s76, v180\n" \
  "v_add_f32 v181, s76, v181\n" \
  "v_add_f32 v182, s76, v182\n" \
  "v_add_f32 v183, s76, v183\n" \
  "v_rcp_f32 v180, v180\n" \
  "v_rcp_f32 v181, v181\n" \
  "v_rcp_f32 v182, v182\n" \
  "v_rcp_f32 v183, v183\n" \
  "v_mul_f32 v176, v176, v180\n" \
  "v_mul_f32 v177, v177, v181\n" \
  "v_mul_f32 v178, v178, v182\n" \
  "v_mul_f32 v179, v179, v183\n" \
  "s_add_u32 s72, s78, 589824\n" \
  "buffer_store_dwordx4 v[176:179], v243, s[32:35], s72 offen sc0 sc1\n" \
  "v_accvgpr_read_b32 v188, a51\n" \
  "v_accvgpr_read_b32 v189, a55\n" \
  "v_accvgpr_read_b32 v190, a59\n" \
  "v_accvgpr_read_b32 v191, a63\n" \
  "v_add_f32 v184, v51, v188\n" \
  "v_add_f32 v185, v55, v189\n" \
  "v_add_f32 v186, v59, v190\n" \
  "v_add_f32 v187, v63, v191\n" \
  "v_mul_f32 v188, s74, v184\n" \
  "v_mul_f32 v189, s74, v185\n" \
  "v_mul_f32 v190, s74, v186\n" \
  "v_mul_f32 v191, s74, v187\n" \
  "v_med3_f32 v188, v188, -s75, s75\n" \
  "v_med3_f32 v189, v189, -s75, s75\n" \
  "v_med3_f32 v190, v190, -s75, s75\n" \
  "v_med3_f32 v191, v191, -s75, s75\n" \
  "v_exp_f32 v188, v188\n" \
  "v_exp_f32 v189, v189\n" \
  "v_exp_f32 v190, v190\n" \
  "v_exp_f32 v191, v191\n" \
  "v_add_f32 v188, s76, v188\n" \
  "v_add_f32 v189, s76, v189\n" \
  "v_add_f32 v190, s76, v190\n" \
  "v_add_f32 v191, s76, v191\n" \
  "v_rcp_f32 v188, v188\n" \
  "v_rcp_f32 v189, v189\n" \
  "v_rcp_f32 v190, v190\n" \
  "v_rcp_f32 v191, v191\n" \
  "v_mul_f32 v184, v184, v188\n" \
  "v_mul_f32 v185, v185, v189\n" \
  "v_mul_f32 v186, v186, v190\n" \
  "v_mul_f32 v187, v187, v191\n" \
  "s_add_u32 s72, s78, 622592\n" \
  "buffer_store_dwordx4 v[184:187], v243, s[32:35], s72 offen sc0 sc1\n" \
  "s_waitcnt vmcnt(28)\n" \
  "v_accvgpr_read_b32 v132, a64\n" \
  "v_accvgpr_read_b32 v133, a68\n" \
  "v_accvgpr_read_b32 v134, a72\n" \
  "v_accvgpr_read_b32 v135, a76\n" \
  "v_add_f32 v128, v64, v132\n" \
  "v_add_f32 v129, v68, v133\n" \
  "v_add_f32 v130, v72, v134\n" \
  "v_add_f32 v131, v76, v135\n" \
  "v_mul_f32 v132, s74, v128\n" \
  "v_mul_f32 v133, s74, v129\n" \
  "v_mul_f32 v134, s74, v130\n" \
  "v_mul_f32 v135, s74, v131\n" \
  "v_med3_f32 v132, v132, -s75, s75\n" \
  "v_med3_f32 v133, v133, -s75, s75\n" \
  "v_med3_f32 v134, v134, -s75, s75\n" \
  "v_med3_f32 v135, v135, -s75, s75\n" \
  "v_exp_f32 v132, v132\n" \
  "v_exp_f32 v133, v133\n" \
  "v_exp_f32 v134, v134\n" \
  "v_exp_f32 v135, v135\n" \
  "v_add_f32 v132, s76, v132\n" \
  "v_add_f32 v133, s76, v133\n" \
  "v_add_f32 v134, s76, v134\n" \
  "v_add_f32 v135, s76, v135\n" \
  "v_rcp_f32 v132, v132\n" \
  "v_rcp_f32 v133, v133\n" \
  "v_rcp_f32 v134, v134\n" \
  "v_rcp_f32 v135, v135\n" \
  "v_mul_f32 v128, v128, v132\n" \
  "v_mul_f32 v129, v129, v133\n" \
  "v_mul_f32 v130, v130, v134\n" \
  "v_mul_f32 v131, v131, v135\n" \
  "s_add_u32 s72, s78, 1048576\n" \
  "buffer_store_dwordx4 v[128:131], v240, s[32:35], s72 offen sc0 sc1\n" \
  "v_accvgpr_read_b32 v140, a65\n" \
  "v_accvgpr_read_b32 v141, a69\n" \
  "v_accvgpr_read_b32 v142, a73\n" \
  "v_accvgpr_read_b32 v143, a77\n" \
  "v_add_f32 v136, v65, v140\n" \
  "v_add_f32 v137, v69, v141\n" \
  "v_add_f32 v138, v73, v142\n" \
  "v_add_f32 v139, v77, v143\n" \
  "v_mul_f32 v140, s74, v136\n" \
  "v_mul_f32 v141, s74, v137\n" \
  "v_mul_f32 v142, s74, v138\n" \
  "v_mul_f32 v143, s74, v139\n" \
  "v_med3_f32 v140, v140, -s75, s75\n" \
  "v_med3_f32 v141, v141, -s75, s75\n" \
  "v_med3_f32 v142, v142, -s75, s75\n" \
  "v_med3_f32 v143, v143, -s75, s75\n" \
  "v_exp_f32 v140, v140\n" \
  "v_exp_f32 v141, v141\n" \
  "v_exp_f32 v142, v142\n" \
  "v_exp_f32 v143, v143\n" \
  "v_add_f32 v140, s76, v140\n" \
  "v_add_f32 v141, s76, v141\n" \
  "v_add_f32 v142, s76, v142\n" \
  "v_add_f32 v143, s76, v143\n" \
  "v_rcp_f32 v140, v140\n" \
  "v_rcp_f32 v141, v141\n" \
  "v_rcp_f32 v142, v142\n" \
  "v_rcp_f32 v143, v143\n" \
  "v_mul_f32 v136, v136, v140\n" \
  "v_mul_f32 v137, v137, v141\n" \
  "v_mul_f32 v138, v138, v142\n" \
  "v_mul_f32 v139, v139, v143\n" \
  "s_add_u32 s72, s78, 1081344\n" \
  "buffer_store_dwordx4 v[136:139], v240, s[32:35], s72 offen sc0 sc1\n" \
  "v_accvgpr_read_b32 v148, a66\n" \
  "v_accvgpr_read_b32 v149, a70\n" \
  "v_accvgpr_read_b32 v150, a74\n" \
  "v_accvgpr_read_b32 v151, a78\n" \
  "v_add_f32 v144, v66, v148\n" \
  "v_add_f32 v145, v70, v149\n" \
  "v_add_f32 v146, v74, v150\n" \
  "v_add_f32 v147, v78, v151\n" \
  "v_mul_f32 v148, s74, v144\n" \
  "v_mul_f32 v149, s74, v145\n" \
  "v_mul_f32 v150, s74, v146\n" \
  "v_mul_f32 v151, s74, v147\n" \
  "v_med3_f32 v148, v148, -s75, s75\n" \
  "v_med3_f32 v149, v149, -s75, s75\n" \
  "v_med3_f32 v150, v150, -s75, s75\n" \
  "v_med3_f32 v151, v151, -s75, s75\n" \
  "v_exp_f32 v148, v148\n" \
  "v_exp_f32 v149, v149\n" \
  "v_exp_f32 v150, v150\n" \
  "v_exp_f32 v151, v151\n" \
  "v_add_f32 v148, s76, v148\n" \
  "v_add_f32 v149, s76, v149\n" \
  "v_add_f32 v150, s76, v150\n" \
  "v_add_f32 v151, s76, v151\n" \
  "v_rcp_f32 v148, v148\n" \
  "v_rcp_f32 v149, v149\n" \
  "v_rcp_f32 v150, v150\n" \
  "v_rcp_f32 v151, v151\n" \
  "v_mul_f32 v144, v144, v148\n" \
  "v_mul_f32 v145, v145, v149\n" \
  "v_mul_f32 v146, v146, v150\n" \
  "v_mul_f32 v147, v147, v151\n" \
  "s_add_u32 s72, s78, 1114112\n" \
  "buffer_store_dwordx4 v[144:147], v240, s[32:35], s72 offen sc0 sc1\n" \
  "v_accvgpr_read_b32 v156, a67\n" \
  "v_accvgpr_read_b32 v157, a71\n" \
  "v_accvgpr_read_b32 v158, a75\n" \
  "v_accvgpr_read_b32 v159, a79\n" \
  "v_add_f32 v152, v67, v156\n" \
  "v_add_f32 v153, v71, v157\n" \
  "v_add_f32 v154, v75, v158\n" \
  "v_add_f32 v155, v79, v159\n" \
  "v_mul_f32 v156, s74, v152\n" \
  "v_mul_f32 v157, s74, v153\n" \
  "v_mul_f32 v158, s74, v154\n" \
  "v_mul_f32 v159, s74, v155\n" \
  "v_med3_f32 v156, v156, -s75, s75\n" \
  "v_med3_f32 v157, v157, -s75, s75\n" \
  "v_med3_f32 v158, v158, -s75, s75\n" \
  "v_med3_f32 v159, v159, -s75, s75\n" \
  "v_exp_f32 v156, v156\n" \
  "v_exp_f32 v157, v157\n" \
  "v_exp_f32 v158, v158\n" \
  "v_exp_f32 v159, v159\n" \
  "v_add_f32 v156, s76, v156\n" \
  "v_add_f32 v157, s76, v157\n" \
  "v_add_f32 v158, s76, v158\n" \
  "v_add_f32 v159, s76, v159\n" \
  "v_rcp_f32 v156, v156\n" \
  "v_rcp_f32 v157, v157\n" \
  "v_rcp_f32 v158, v158\n" \
  "v_rcp_f32 v159, v159\n" \
  "v_mul_f32 v152, v152, v156\n" \
  "v_mul_f32 v153, v153, v157\n" \
  "v_mul_f32 v154, v154, v158\n" \
  "v_mul_f32 v155, v155, v159\n" \
  "s_add_u32 s72, s78, 1146880\n" \
  "buffer_store_dwordx4 v[152:155], v240, s[32:35], s72 offen sc0 sc1\n" \
  "s_waitcnt vmcnt(28)\n" \
  "v_accvgpr_read_b32 v164, a80\n" \
  "v_accvgpr_read_b32 v165, a84\n" \
  "v_accvgpr_read_b32 v166, a88\n" \
  "v_accvgpr_read_b32 v167, a92\n" \
  "v_add_f32 v160, v80, v164\n" \
  "v_add_f32 v161, v84, v165\n" \
  "v_add_f32 v162, v88, v166\n" \
  "v_add_f32 v163, v92, v167\n" \
  "v_mul_f32 v164, s74, v160\n" \
  "v_mul_f32 v165, s74, v161\n" \
  "v_mul_f32 v166, s74, v162\n" \
  "v_mul_f32 v167, s74, v163\n" \
  "v_med3_f32 v164, v164, -s75, s75\n" \
  "v_med3_f32 v165, v165, -s75, s75\n" \
  "v_med3_f32 v166, v166, -s75, s75\n" \
  "v_med3_f32 v167, v167, -s75, s75\n" \
  "v_exp_f32 v164, v164\n" \
  "v_exp_f32 v165, v165\n" \
  "v_exp_f32 v166, v166\n" \
  "v_exp_f32 v167, v167\n" \
  "v_add_f32 v164, s76, v164\n" \
  "v_add_f32 v165, s76, v165\n" \
  "v_add_f32 v166, s76, v166\n" \
  "v_add_f32 v167, s76, v167\n" \
  "v_rcp_f32 v164, v164\n" \
  "v_rcp_f32 v165, v165\n" \
  "v_rcp_f32 v166, v166\n" \
  "v_rcp_f32 v167, v167\n" \
  "v_mul_f32 v160, v160, v164\n" \
  "v_mul_f32 v161, v161, v165\n" \
  "v_mul_f32 v162, v162, v166\n" \
  "v_mul_f32 v163, v163, v167\n" \
  "s_add_u32 s72, s78, 1048576\n" \
  "buffer_store_dwordx4 v[160:163], v243, s[32:35], s72 offen sc0 sc1\n" \
  "v_accvgpr_read_b32 v172, a81\n" \
  "v_accvgpr_read_b32 v173, a85\n" \
  "v_accvgpr_read_b32 v174, a89\n" \
  "v_accvgpr_read_b32 v175, a93\n" \
  "v_add_f32 v168, v81, v172\n" \
  "v_add_f32 v169, v85, v173\n" \
  "v_add_f32 v170, v89, v174\n" \
  "v_add_f32 v171, v93, v175\n" \
  "v_mul_f32 v172, s74, v168\n" \
  "v_mul_f32 v173, s74, v169\n" \
  "v_mul_f32 v174, s74, v170\n" \
  "v_mul_f32 v175, s74, v171\n" \
  "v_med3_f32 v172, v172, -s75, s75\n" \
  "v_med3_f32 v173, v173, -s75, s75\n" \
  "v_med3_f32 v174, v174, -s75, s75\n" \
  "v_med3_f32 v175, v175, -s75, s75\n" \
  "v_exp_f32 v172, v172\n" \
  "v_exp_f32 v173, v173\n" \
  "v_exp_f32 v174, v174\n" \
  "v_exp_f32 v175, v175\n" \
  "v_add_f32 v172, s76, v172\n" \
  "v_add_f32 v173, s76, v173\n" \
  "v_add_f32 v174, s76, v174\n" \
  "v_add_f32 v175, s76, v175\n" \
  "v_rcp_f32 v172, v172\n" \
  "v_rcp_f32 v173, v173\n" \
  "v_rcp_f32 v174, v174\n" \
  "v_rcp_f32 v175, v175\n" \
  "v_mul_f32 v168, v168, v172\n" \
  "v_mul_f32 v169, v169, v173\n" \
  "v_mul_f32 v170, v170, v174\n" \
  "v_mul_f32 v171, v171, v175\n" \
  "s_add_u32 s72, s78, 1081344\n" \
  "buffer_store_dwordx4 v[168:171], v243, s[32:35], s72 offen sc0 sc1\n" \
  "v_accvgpr_read_b32 v180, a82\n" \
  "v_accvgpr_read_b32 v181, a86\n" \
  "v_accvgpr_read_b32 v182, a90\n" \
  "v_accvgpr_read_b32 v183, a94\n" \
  "v_add_f32 v176, v82, v180\n" \
  "v_add_f32 v177, v86, v181\n" \
  "v_add_f32 v178, v90, v182\n" \
  "v_add_f32 v179, v94, v183\n" \
  "v_mul_f32 v180, s74, v176\n" \
  "v_mul_f32 v181, s74, v177\n" \
  "v_mul_f32 v182, s74, v178\n" \
  "v_mul_f32 v183, s74, v179\n" \
  "v_med3_f32 v180, v180, -s75, s75\n" \
  "v_med3_f32 v181, v181, -s75, s75\n" \
  "v_med3_f32 v182, v182, -s75, s75\n" \
  "v_med3_f32 v183, v183, -s75, s75\n" \
  "v_exp_f32 v180, v180\n" \
  "v_exp_f32 v181, v181\n" \
  "v_exp_f32 v182, v182\n" \
  "v_exp_f32 v183, v183\n" \
  "v_add_f32 v180, s76, v180\n" \
  "v_add_f32 v181, s76, v181\n" \
  "v_add_f32 v182, s76, v182\n" \
  "v_add_f32 v183, s76, v183\n" \
  "v_rcp_f32 v180, v180\n" \
  "v_rcp_f32 v181, v181\n" \
  "v_rcp_f32 v182, v182\n" \
  "v_rcp_f32 v183, v183\n" \
  "v_mul_f32 v176, v176, v180\n" \
  "v_mul_f32 v177, v177, v181\n" \
  "v_mul_f32 v178, v178, v182\n" \
  "v_mul_f32 v179, v179, v183\n" \
  "s_add_u32 s72, s78, 1114112\n" \
  "buffer_store_dwordx4 v[176:179], v243, s[32:35], s72 offen sc0 sc1\n" \
  "v_accvgpr_read_b32 v188, a83\n" \
  "v_accvgpr_read_b32 v189, a87\n" \
  "v_accvgpr_read_b32 v190, a91\n" \
  "v_accvgpr_read_b32 v191, a95\n" \
  "v_add_f32 v184, v83, v188\n" \
  "v_add_f32 v185, v87, v189\n" \
  "v_add_f32 v186, v91, v190\n" \
  "v_add_f32 v187, v95, v191\n" \
  "v_mul_f32 v188, s74, v184\n" \
  "v_mul_f32 v189, s74, v185\n" \
  "v_mul_f32 v190, s74, v186\n" \
  "v_mul_f32 v191, s74, v187\n" \
  "v_med3_f32 v188, v188, -s75, s75\n" \
  "v_med3_f32 v189, v189, -s75, s75\n" \
  "v_med3_f32 v190, v190, -s75, s75\n" \
  "v_med3_f32 v191, v191, -s75, s75\n" \
  "v_exp_f32 v188, v188\n" \
  "v_exp_f32 v189, v189\n" \
  "v_exp_f32 v190, v190\n" \
  "v_exp_f32 v191, v191\n" \
  "v_add_f32 v188, s76, v188\n" \
  "v_add_f32 v189, s76, v189\n" \
  "v_add_f32 v190, s76, v190\n" \
  "v_add_f32 v191, s76, v191\n" \
  "v_rcp_f32 v188, v188\n" \
  "v_rcp_f32 v189, v189\n" \
  "v_rcp_f32 v190, v190\n" \
  "v_rcp_f32 v191, v191\n" \
  "v_mul_f32 v184, v184, v188\n" \
  "v_mul_f32 v185, v185, v189\n" \
  "v_mul_f32 v186, v186, v190\n" \
  "v_mul_f32 v187, v187, v191\n" \
  "s_add_u32 s72, s78, 1146880\n" \
  "buffer_store_dwordx4 v[184:187], v243, s[32:35], s72 offen sc0 sc1\n" \
  "s_waitcnt vmcnt(28)\n" \
  "v_accvgpr_read_b32 v132, a96\n" \
  "v_accvgpr_read_b32 v133, a100\n" \
  "v_accvgpr_read_b32 v134, a104\n" \
  "v_accvgpr_read_b32 v135, a108\n" \
  "v_add_f32 v128, v96, v132\n" \
  "v_add_f32 v129, v100, v133\n" \
  "v_add_f32 v130, v104, v134\n" \
  "v_add_f32 v131, v108, v135\n" \
  "v_mul_f32 v132, s74, v128\n" \
  "v_mul_f32 v133, s74, v129\n" \
  "v_mul_f32 v134, s74, v130\n" \
  "v_mul_f32 v135, s74, v131\n" \
  "v_med3_f32 v132, v132, -s75, s75\n" \
  "v_med3_f32 v133, v133, -s75, s75\n" \
  "v_med3_f32 v134, v134, -s75, s75\n" \
  "v_med3_f32 v135, v135, -s75, s75\n" \
  "v_exp_f32 v132, v132\n" \
  "v_exp_f32 v133, v133\n" \
  "v_exp_f32 v134, v134\n" \
  "v_exp_f32 v135, v135\n" \
  "v_add_f32 v132, s76, v132\n" \
  "v_add_f32 v133, s76, v133\n" \
  "v_add_f32 v134, s76, v134\n" \
  "v_add_f32 v135, s76, v135\n" \
  "v_rcp_f32 v132, v132\n" \
  "v_rcp_f32 v133, v133\n" \
  "v_rcp_f32 v134, v134\n" \
  "v_rcp_f32 v135, v135\n" \
  "v_mul_f32 v128, v128, v132\n" \
  "v_mul_f32 v129, v129, v133\n" \
  "v_mul_f32 v130, v130, v134\n" \
  "v_mul_f32 v131, v131, v135\n" \
  "s_add_u32 s72, s78, 1572864\n" \
  "buffer_store_dwordx4 v[128:131], v240, s[32:35], s72 offen sc0 sc1\n" \
  "v_accvgpr_read_b32 v140, a97\n" \
  "v_accvgpr_read_b32 v141, a101\n" \
  "v_accvgpr_read_b32 v142, a105\n" \
  "v_accvgpr_read_b32 v143, a109\n" \
  "v_add_f32 v136, v97, v140\n" \
  "v_add_f32 v137, v101, v141\n" \
  "v_add_f32 v138, v105, v142\n" \
  "v_add_f32 v139, v109, v143\n" \
  "v_mul_f32 v140, s74, v136\n" \
  "v_mul_f32 v141, s74, v137\n" \
  "v_mul_f32 v142, s74, v138\n" \
  "v_mul_f32 v143, s74, v139\n" \
  "v_med3_f32 v140, v140, -s75, s75\n" \
  "v_med3_f32 v141, v141, -s75, s75\n" \
  "v_med3_f32 v142, v142, -s75, s75\n" \
  "v_med3_f32 v143, v143, -s75, s75\n" \
  "v_exp_f32 v140, v140\n" \
  "v_exp_f32 v141, v141\n" \
  "v_exp_f32 v142, v142\n" \
  "v_exp_f32 v143, v143\n" \
  "v_add_f32 v140, s76, v140\n" \
  "v_add_f32 v141, s76, v141\n" \
  "v_add_f32 v142, s76, v142\n" \
  "v_add_f32 v143, s76, v143\n" \
  "v_rcp_f32 v140, v140\n" \
  "v_rcp_f32 v141, v141\n" \
  "v_rcp_f32 v142, v142\n" \
  "v_rcp_f32 v143, v143\n" \
  "v_mul_f32 v136, v136, v140\n" \
  "v_mul_f32 v137, v137, v141\n" \
  "v_mul_f32 v138, v138, v142\n" \
  "v_mul_f32 v139, v139, v143\n" \
  "s_add_u32 s72, s78, 1605632\n" \
  "buffer_store_dwordx4 v[136:139], v240, s[32:35], s72 offen sc0 sc1\n" \
  "v_accvgpr_read_b32 v148, a98\n" \
  "v_accvgpr_read_b32 v149, a102\n" \
  "v_accvgpr_read_b32 v150, a106\n" \
  "v_accvgpr_read_b32 v151, a110\n" \
  "v_add_f32 v144, v98, v148\n" \
  "v_add_f32 v145, v102, v149\n" \
  "v_add_f32 v146, v106, v150\n" \
  "v_add_f32 v147, v110, v151\n" \
  "v_mul_f32 v148, s74, v144\n" \
  "v_mul_f32 v149, s74, v145\n" \
  "v_mul_f32 v150, s74, v146\n" \
  "v_mul_f32 v151, s74, v147\n" \
  "v_med3_f32 v148, v148, -s75, s75\n" \
  "v_med3_f32 v149, v149, -s75, s75\n" \
  "v_med3_f32 v150, v150, -s75, s75\n" \
  "v_med3_f32 v151, v151, -s75, s75\n" \
  "v_exp_f32 v148, v148\n" \
  "v_exp_f32 v149, v149\n" \
  "v_exp_f32 v150, v150\n" \
  "v_exp_f32 v151, v151\n" \
  "v_add_f32 v148, s76, v148\n" \
  "v_add_f32 v149, s76, v149\n" \
  "v_add_f32 v150, s76, v150\n" \
  "v_add_f32 v151, s76, v151\n" \
  "v_rcp_f32 v148, v148\n" \
  "v_rcp_f32 v149, v149\n" \
  "v_rcp_f32 v150, v150\n" \
  "v_rcp_f32 v151, v151\n" \
  "v_mul_f32 v144, v144, v148\n" \
  "v_mul_f32 v145, v145, v149\n" \
  "v_mul_f32 v146, v146, v150\n" \
  "v_mul_f32 v147, v147, v151\n" \
  "s_add_u32 s72, s78, 1638400\n" \
  "buffer_store_dwordx4 v[144:147], v240, s[32:35], s72 offen sc0 sc1\n" \
  "v_accvgpr_read_b32 v156, a99\n" \
  "v_accvgpr_read_b32 v157, a103\n" \
  "v_accvgpr_read_b32 v158, a107\n" \
  "v_accvgpr_read_b32 v159, a111\n" \
  "v_add_f32 v152, v99, v156\n" \
  "v_add_f32 v153, v103, v157\n" \
  "v_add_f32 v154, v107, v158\n" \
  "v_add_f32 v155, v111, v159\n" \
  "v_mul_f32 v156, s74, v152\n" \
  "v_mul_f32 v157, s74, v153\n" \
  "v_mul_f32 v158, s74, v154\n" \
  "v_mul_f32 v159, s74, v155\n" \
  "v_med3_f32 v156, v156, -s75, s75\n" \
  "v_med3_f32 v157, v157, -s75, s75\n" \
  "v_med3_f32 v158, v158, -s75, s75\n" \
  "v_med3_f32 v159, v159, -s75, s75\n" \
  "v_exp_f32 v156, v156\n" \
  "v_exp_f32 v157, v157\n" \
  "v_exp_f32 v158, v158\n" \
  "v_exp_f32 v159, v159\n" \
  "v_add_f32 v156, s76, v156\n" \
  "v_add_f32 v157, s76, v157\n" \
  "v_add_f32 v158, s76, v158\n" \
  "v_add_f32 v159, s76, v159\n" \
  "v_rcp_f32 v156, v156\n" \
  "v_rcp_f32 v157, v157\n" \
  "v_rcp_f32 v158, v158\n" \
  "v_rcp_f32 v159, v159\n" \
  "v_mul_f32 v152, v152, v156\n" \
  "v_mul_f32 v153, v153, v157\n" \
  "v_mul_f32 v154, v154, v158\n" \
  "v_mul_f32 v155, v155, v159\n" \
  "s_add_u32 s72, s78, 1671168\n" \
  "buffer_store_dwordx4 v[152:155], v240, s[32:35], s72 offen sc0 sc1\n" \
  "s_waitcnt vmcnt(28)\n" \
  "v_accvgpr_read_b32 v164, a112\n" \
  "v_accvgpr_read_b32 v165, a116\n" \
  "v_accvgpr_read_b32 v166, a120\n" \
  "v_accvgpr_read_b32 v167, a124\n" \
  "v_add_f32 v160, v112, v164\n" \
  "v_add_f32 v161, v116, v165\n" \
  "v_add_f32 v162, v120, v166\n" \
  "v_add_f32 v163, v124, v167\n" \
  "v_mul_f32 v164, s74, v160\n" \
  "v_mul_f32 v165, s74, v161\n" \
  "v_mul_f32 v166, s74, v162\n" \
  "v_mul_f32 v167, s74, v163\n" \
  "v_med3_f32 v164, v164, -s75, s75\n" \
  "v_med3_f32 v165, v165, -s75, s75\n" \
  "v_med3_f32 v166, v166, -s75, s75\n" \
  "v_med3_f32 v167, v167, -s75, s75\n" \
  "v_exp_f32 v164, v164\n" \
  "v_exp_f32 v165, v165\n" \
  "v_exp_f32 v166, v166\n" \
  "v_exp_f32 v167, v167\n" \
  "v_add_f32 v164, s76, v164\n" \
  "v_add_f32 v165, s76, v165\n" \
  "v_add_f32 v166, s76, v166\n" \
  "v_add_f32 v167, s76, v167\n" \
  "v_rcp_f32 v164, v164\n" \
  "v_rcp_f32 v165, v165\n" \
  "v_rcp_f32 v166, v166\n" \
  "v_rcp_f32 v167, v167\n" \
  "v_mul_f32 v160, v160, v164\n" \
  "v_mul_f32 v161, v161, v165\n" \
  "v_mul_f32 v162, v162, v166\n" \
  "v_mul_f32 v163, v163, v167\n" \
  "s_add_u32 s72, s78, 1572864\n" \
  "buffer_store_dwordx4 v[160:163], v243, s[32:35], s72 offen sc0 sc1\n" \
  "v_accvgpr_read_b32 v172, a113\n" \
  "v_accvgpr_read_b32 v173, a117\n" \
  "v_accvgpr_read_b32 v174, a121\n" \
  "v_accvgpr_read_b32 v175, a125\n" \
  "v_add_f32 v168, v113, v172\n" \
  "v_add_f32 v169, v117, v173\n" \
  "v_add_f32 v170, v121, v174\n" \
  "v_add_f32 v171, v125, v175\n" \
  "v_mul_f32 v172, s74, v168\n" \
  "v_mul_f32 v173, s74, v169\n" \
  "v_mul_f32 v174, s74, v170\n" \
  "v_mul_f32 v175, s74, v171\n" \
  "v_med3_f32 v172, v172, -s75, s75\n" \
  "v_med3_f32 v173, v173, -s75, s75\n" \
  "v_med3_f32 v174, v174, -s75, s75\n" \
  "v_med3_f32 v175, v175, -s75, s75\n" \
  "v_exp_f32 v172, v172\n" \
  "v_exp_f32 v173, v173\n" \
  "v_exp_f32 v174, v174\n" \
  "v_exp_f32 v175, v175\n" \
  "v_add_f32 v172, s76, v172\n" \
  "v_add_f32 v173, s76, v173\n" \
  "v_add_f32 v174, s76, v174\n" \
  "v_add_f32 v175, s76, v175\n" \
  "v_rcp_f32 v172, v172\n" \
  "v_rcp_f32 v173, v173\n" \
  "v_rcp_f32 v174, v174\n" \
  "v_rcp_f32 v175, v175\n" \
  "v_mul_f32 v168, v168, v172\n" \
  "v_mul_f32 v169, v169, v173\n" \
  "v_mul_f32 v170, v170, v174\n" \
  "v_mul_f32 v171, v171, v175\n" \
  "s_add_u32 s72, s78, 1605632\n" \
  "buffer_store_dwordx4 v[168:171], v243, s[32:35], s72 offen sc0 sc1\n" \
  "v_accvgpr_read_b32 v180, a114\n" \
  "v_accvgpr_read_b32 v181, a118\n" \
  "v_accvgpr_read_b32 v182, a122\n" \
  "v_accvgpr_read_b32 v183, a126\n" \
  "v_add_f32 v176, v114, v180\n" \
  "v_add_f32 v177, v118, v181\n" \
  "v_add_f32 v178, v122, v182\n" \
  "v_add_f32 v179, v126, v183\n" \
  "v_mul_f32 v180, s74, v176\n" \
  "v_mul_f32 v181, s74, v177\n" \
  "v_mul_f32 v182, s74, v178\n" \
  "v_mul_f32 v183, s74, v179\n" \
  "v_med3_f32 v180, v180, -s75, s75\n" \
  "v_med3_f32 v181, v181, -s75, s75\n" \
  "v_med3_f32 v182, v182, -s75, s75\n" \
  "v_med3_f32 v183, v183, -s75, s75\n" \
  "v_exp_f32 v180, v180\n" \
  "v_exp_f32 v181, v181\n" \
  "v_exp_f32 v182, v182\n" \
  "v_exp_f32 v183, v183\n" \
  "v_add_f32 v180, s76, v180\n" \
  "v_add_f32 v181, s76, v181\n" \
  "v_add_f32 v182, s76, v182\n" \
  "v_add_f32 v183, s76, v183\n" \
  "v_rcp_f32 v180, v180\n" \
  "v_rcp_f32 v181, v181\n" \
  "v_rcp_f32 v182, v182\n" \
  "v_rcp_f32 v183, v183\n" \
  "v_mul_f32 v176, v176, v180\n" \
  "v_mul_f32 v177, v177, v181\n" \
  "v_mul_f32 v178, v178, v182\n" \
  "v_mul_f32 v179, v179, v183\n" \
  "s_add_u32 s72, s78, 1638400\n" \
  "buffer_store_dwordx4 v[176:179], v243, s[32:35], s72 offen sc0 sc1\n" \
  "v_accvgpr_read_b32 v188, a115\n" \
  "v_accvgpr_read_b32 v189, a119\n" \
  "v_accvgpr_read_b32 v190, a123\n" \
  "v_accvgpr_read_b32 v191, a127\n" \
  "v_add_f32 v184, v115, v188\n" \
  "v_add_f32 v185, v119, v189\n" \
  "v_add_f32 v186, v123, v190\n" \
  "v_add_f32 v187, v127, v191\n" \
  "v_mul_f32 v188, s74, v184\n" \
  "v_mul_f32 v189, s74, v185\n" \
  "v_mul_f32 v190, s74, v186\n" \
  "v_mul_f32 v191, s74, v187\n" \
  "v_med3_f32 v188, v188, -s75, s75\n" \
  "v_med3_f32 v189, v189, -s75, s75\n" \
  "v_med3_f32 v190, v190, -s75, s75\n" \
  "v_med3_f32 v191, v191, -s75, s75\n" \
  "v_exp_f32 v188, v188\n" \
  "v_exp_f32 v189, v189\n" \
  "v_exp_f32 v190, v190\n" \
  "v_exp_f32 v191, v191\n" \
  "v_add_f32 v188, s76, v188\n" \
  "v_add_f32 v189, s76, v189\n" \
  "v_add_f32 v190, s76, v190\n" \
  "v_add_f32 v191, s76, v191\n" \
  "v_rcp_f32 v188, v188\n" \
  "v_rcp_f32 v189, v189\n" \
  "v_rcp_f32 v190, v190\n" \
  "v_rcp_f32 v191, v191\n" \
  "v_mul_f32 v184, v184, v188\n" \
  "v_mul_f32 v185, v185, v189\n" \
  "v_mul_f32 v186, v186, v190\n" \
  "v_mul_f32 v187, v187, v191\n" \
  "s_add_u32 s72, s78, 1671168\n" \
  "buffer_store_dwordx4 v[184:187], v243, s[32:35], s72 offen sc0 sc1\n" \
  "v_mov_b32 v224, s82\n" \
  "s_lshl_b32 s72, %7, 7\n" \
  "s_add_u32 s72, s72, 67371008\n" \
  "s_cmp_lg_u32 %8, 0\n" \
  "s_cbranch_scc1 W047_%=\n" \
  "s_mov_b64 exec, 1\n" \
  "buffer_store_dword v224, off, s[36:39], s72 sc1\n" \
  "s_mov_b64 exec, -1\n" \
  "W047_%=:\n" \
  "s_branch END_%=\n" \
  "SLOW2_%=:\n" \
  "s_cmp_lg_u32 s89, 0\n" \
  "s_cbranch_scc1 OK1_%=\n" \
  "s_memrealtime s[52:53]\n" \
  "s_mov_b32 s77, 0\n" \
  "v_lshrrev_b32 v225, 2, v248\n" \
  "s_mov_b64 exec, 1\n" \
  "SPIN3_%=:\n" \
  "buffer_load_dword v224, v225, s[36:39], s88 offen sc0 sc1\n" \
  "s_waitcnt vmcnt(0)\n" \
  "v_cmp_gt_u32 vcc, s82, v224\n" \
  "v_readfirstlane_b32 s76, v224\n" \
  "s_nop 1\n" \
  "s_cmp_eq_u64 vcc, 0\n" \
  "s_cbranch_scc1 DN5_%=\n" \
  "s_sleep 8\n" \
  "s_add_u32 s77, s77, 1\n" \
  "s_and_b32 s77, s77, 63\n" \
  "s_cbranch_scc1 SPIN3_%=\n" \
  "s_mov_b32 s76, 67436544\n" \
  "buffer_load_dword v224, off, s[36:39], s76 sc1\n" \
  "s_memrealtime s[54:55]\n" \
  "s_waitcnt vmcnt(0) lgkmcnt(0)\n" \
  "v_readfirstlane_b32 s76, v224\n" \
  "s_nop 1\n" \
  "s_cmp_eq_u32 s76, s82\n" \
  "s_cbranch_scc1 TMO4_%=\n" \
  "s_sub_u32 s54, s54, s52\n" \
  "s_cmp_lt_u32 s54, 30000000\n" \
  "s_cbranch_scc1 SPIN3_%=\n" \
  "TMO4_%=:\n" \
  "s_mov_b32 s89, 1\n" \
  "s_mov_b32 s76, 67436544\n" \
  "v_mov_b32 v224, s82\n" \
  "s_mov_b64 exec, 1\n" \
  "buffer_store_dword v224, off, s[36:39], s76 sc1\n" \
  "s_mov_b32 s76, 0\n" \
  "DN5_%=:\n" \
  "s_mov_b64 exec, -1\n" \
  "s_waitcnt lgkmcnt(0)\n" \
  "s_branch OK1_%=\n" \
  "SLOW8_%=:\n" \
  "s_cmp_lg_u32 s89, 0\n" \
  "s_cbranch_scc1 OK7_%=\n" \
  "s_memrealtime s[52:53]\n" \
  "s_mov_b32 s77, 0\n" \
  "v_lshrrev_b32 v226, 7, v248\n" \
  "v_lshrrev_b32 v225, 2, v248\n" \
  "v_and_b32 v225, 31, v225\n" \
  "v_lshl_add_u32 v225, v226, 9, v225\n" \
  "s_mov_b64 exec, 65535\n" \
  "SPIN9_%=:\n" \
  "buffer_load_dword v224, v225, s[36:39], s74 offen sc0 sc1\n" \
  "s_waitcnt vmcnt(0)\n" \
  "v_cmp_gt_u32 vcc, s82, v224\n" \
  "v_readfirstlane_b32 s76, v224\n" \
  "s_nop 1\n" \
  "s_cmp_eq_u64 vcc, 0\n" \
  "s_cbranch_scc1 DN11_%=\n" \
  "s_sleep 8\n" \
  "s_add_u32 s77, s77, 1\n" \
  "s_and_b32 s77, s77, 63\n" \
  "s_cbranch_scc1 SPIN9_%=\n" \
  "s_mov_b32 s76, 67436544\n" \
  "buffer_load_dword v224, off, s[36:39], s76 sc1\n" \
  "s_memrealtime s[54:55]\n" \
  "s_waitcnt vmcnt(0) lgkmcnt(0)\n" \
  "v_readfirstlane_b32 s76, v224\n" \
  "s_nop 1\n" \
  "s_cmp_eq_u32 s76, s82\n" \
  "s_cbranch_scc1 TMO10_%=\n" \
  "s_sub_u32 s54, s54, s52\n" \
  "s_cmp_lt_u32 s54, 30000000\n" \
  "s_cbranch_scc1 SPIN9_%=\n" \
  "TMO10_%=:\n" \
  "s_mov_b32 s89, 1\n" \
  "s_mov_b32 s76, 67436544\n" \
  "v_mov_b32 v224, s82\n" \
  "s_mov_b64 exec, 1\n" \
  "buffer_store_dword v224, off, s[36:39], s76 sc1\n" \
  "s_mov_b32 s76, 0\n" \
  "DN11_%=:\n" \
  "s_mov_b64 exec, -1\n" \
  "s_waitcnt lgkmcnt(0)\n" \
  "s_branch OK7_%=\n" \
  "DSLOW14_%=:\n" \
  "s_mov_b32 s74, s75\n" \
  "s_cmp_lg_u32 s89, 0\n" \
  "s_cbranch_scc1 DOK13_%=\n" \
  "s_memrealtime s[52:53]\n" \
  "s_mov_b32 s77, 0\n" \
  "v_bfe_u32 v232, %1, 8, 16\n" \
  "DSPIN15_%=:\n" \
  "buffer_load_dword v253, v232, s[36:39], s90 offen sc0 sc1\n" \
  "s_waitcnt vmcnt(0)\n" \
  "v_cmp_le_u32 vcc, s82, v253\n" \
  "s_nop 1\n" \
  "s_and_b32 s74, vcc_lo, s75\n" \
  "s_cbranch_scc1 DOK13_%=\n" \
  "s_sleep 8\n" \
  "s_add_u32 s77, s77, 1\n" \
  "s_and_b32 s77, s77, 63\n" \
  "s_cbranch_scc1 DSPIN15_%=\n" \
  "s_mov_b32 s76, 67436544\n" \
  "buffer_load_dword v224, off, s[36:39], s76 sc1\n" \
  "s_memrealtime s[54:55]\n" \
  "s_waitcnt vmcnt(0) lgkmcnt(0)\n" \
  "v_readfirstlane_b32 s76, v224\n" \
  "s_nop 1\n" \
  "s_cmp_eq_u32 s76, s82\n" \
  "s_cbranch_scc1 DTMO16_%=\n" \
  "s_sub_u32 s54, s54, s52\n" \
  "s_cmp_lt_u32 s54, 30000000\n" \
  "s_cbranch_scc1 DSPIN15_%=\n" \
  "DTMO16_%=:\n" \
  "s_mov_b32 s74, s75\n" \
  "s_branch DOK13_%=\n" \
  "SLOW23_%=:\n" \
  "s_cmp_lg_u32 s89, 0\n" \
  "s_cbranch_scc1 OK22_%=\n" \
  "s_memrealtime s[52:53]\n" \
  "s_mov_b32 s77, 0\n" \
  "v_lshrrev_b32 v225, 2, v248\n" \
  "s_mov_b64 exec, 255\n" \
  "SPIN24_%=:\n" \
  "buffer_load_dword v224, v225, s[36:39], s70 offen sc0 sc1\n" \
  "s_waitcnt vmcnt(0)\n" \
  "v_cmp_gt_u32 vcc, s82, v224\n" \
  "v_readfirstlane_b32 s76, v224\n" \
  "s_nop 1\n" \
  "s_cmp_eq_u64 vcc, 0\n" \
  "s_cbranch_scc1 DN26_%=\n" \
  "s_sleep 8\n" \
  "s_add_u32 s77, s77, 1\n" \
  "s_and_b32 s77, s77, 63\n" \
  "s_cbranch_scc1 SPIN24_%=\n" \
  "s_mov_b32 s76, 67436544\n" \
  "buffer_load_dword v224, off, s[36:39], s76 sc1\n" \
  "s_memrealtime s[54:55]\n" \
  "s_waitcnt vmcnt(0) lgkmcnt(0)\n" \
  "v_readfirstlane_b32 s76, v224\n" \
  "s_nop 1\n" \
  "s_cmp_eq_u32 s76, s82\n" \
  "s_cbranch_scc1 TMO25_%=\n" \
  "s_sub_u32 s54, s54, s52\n" \
  "s_cmp_lt_u32 s54, 30000000\n" \
  "s_cbranch_scc1 SPIN24_%=\n" \
  "TMO25_%=:\n" \
  "s_mov_b32 s89, 1\n" \
  "s_mov_b32 s76, 67436544\n" \
  "v_mov_b32 v224, s82\n" \
  "s_mov_b64 exec, 1\n" \
  "buffer_store_dword v224, off, s[36:39], s76 sc1\n" \
  "s_mov_b32 s76, 0\n" \
  "DN26_%=:\n" \
  "s_mov_b64 exec, -1\n" \
  "s_waitcnt lgkmcnt(0)\n" \
  "s_branch OK22_%=\n" \
  "DSLOW28_%=:\n" \
  "s_mov_b32 s74, s75\n" \
  "s_cmp_lg_u32 s89, 0\n" \
  "s_cbranch_scc1 DOK27_%=\n" \
  "s_memrealtime s[52:53]\n" \
  "s_mov_b32 s77, 0\n" \
  "v_bfe_u32 v232, %1, 8, 16\n" \
  "DSPIN29_%=:\n" \
  "buffer_load_dword v253, v232, s[36:39], s90 offen sc0 sc1\n" \
  "s_waitcnt vmcnt(0)\n" \
  "v_cmp_le_u32 vcc, s82, v253\n" \
  "s_nop 1\n" \
  "s_and_b32 s74, vcc_lo, s75\n" \
  "s_cbranch_scc1 DOK27_%=\n" \
  "s_sleep 8\n" \
  "s_add_u32 s77, s77, 1\n" \
  "s_and_b32 s77, s77, 63\n" \
  "s_cbranch_scc1 DSPIN29_%=\n" \
  "s_mov_b32 s76, 67436544\n" \
  "buffer_load_dword v224, off, s[36:39], s76 sc1\n" \
  "s_memrealtime s[54:55]\n" \
  "s_waitcnt vmcnt(0) lgkmcnt(0)\n" \
  "v_readfirstlane_b32 s76, v224\n" \
  "s_nop 1\n" \
  "s_cmp_eq_u32 s76, s82\n" \
  "s_cbranch_scc1 DTMO30_%=\n" \
  "s_sub_u32 s54, s54, s52\n" \
  "s_cmp_lt_u32 s54, 30000000\n" \
  "s_cbranch_scc1 DSPIN29_%=\n" \
  "DTMO30_%=:\n" \
  "s_mov_b32 s74, s75\n" \
  "s_branch DOK27_%=\n" \
  "SLOW37_%=:\n" \
  "s_cmp_lg_u32 s89, 0\n" \
  "s_cbranch_scc1 OK36_%=\n" \
  "s_memrealtime s[52:53]\n" \
  "s_mov_b32 s77, 0\n" \
  "v_lshrrev_b32 v225, 2, v248\n" \
  "s_mov_b64 exec, 255\n" \
  "SPIN38_%=:\n" \
  "buffer_load_dword v224, v225, s[36:39], s70 offen sc0 sc1\n" \
  "s_waitcnt vmcnt(0)\n" \
  "v_cmp_gt_u32 vcc, s82, v224\n" \
  "v_readfirstlane_b32 s76, v224\n" \
  "s_nop 1\n" \
  "s_cmp_eq_u64 vcc, 0\n" \
  "s_cbranch_scc1 DN40_%=\n" \
  "s_sleep 8\n" \
  "s_add_u32 s77, s77, 1\n" \
  "s_and_b32 s77, s77, 63\n" \
  "s_cbranch_scc1 SPIN38_%=\n" \
  "s_mov_b32 s76, 67436544\n" \
  "buffer_load_dword v224, off, s[36:39], s76 sc1\n" \
  "s_memrealtime s[54:55]\n" \
  "s_waitcnt vmcnt(0) lgkmcnt(0)\n" \
  "v_readfirstlane_b32 s76, v224\n" \
  "s_nop 1\n" \
  "s_cmp_eq_u32 s76, s82\n" \
  "s_cbranch_scc1 TMO39_%=\n" \
  "s_sub_u32 s54, s54, s52\n" \
  "s_cmp_lt_u32 s54, 30000000\n" \
  "s_cbranch_scc1 SPIN38_%=\n" \
  "TMO39_%=:\n" \
  "s_mov_b32 s89, 1\n" \
  "s_mov_b32 s76, 67436544\n" \
  "v_mov_b32 v224, s82\n" \
  "s_mov_b64 exec, 1\n" \
  "buffer_store_dword v224, off, s[36:39], s76 sc1\n" \
  "s_mov_b32 s76, 0\n" \
  "DN40_%=:\n" \
  "s_mov_b64 exec, -1\n" \
  "s_waitcnt lgkmcnt(0)\n" \
  "s_branch OK36_%=\n" \
  "SLOW43_%=:\n" \
  "s_cmp_lg_u32 s89, 0\n" \
  "s_cbranch_scc1 OK42_%=\n" \
  "s_memrealtime s[52:53]\n" \
  "s_mov_b32 s77, 0\n" \
  "v_lshrrev_b32 v225, 2, v248\n" \
  "s_mov_b64 exec, 255\n" \
  "SPIN44_%=:\n" \
  "buffer_load_dword v224, v225, s[36:39], s70 offen sc0 sc1\n" \
  "s_waitcnt vmcnt(0)\n" \
  "v_cmp_gt_u32 vcc, s82, v224\n" \
  "v_readfirstlane_b32 s76, v224\n" \
  "s_nop 1\n" \
  "s_cmp_eq_u64 vcc, 0\n" \
  "s_cbranch_scc1 DN46_%=\n" \
  "s_sleep 8\n" \
  "s_add_u32 s77, s77, 1\n" \
  "s_and_b32 s77, s77, 63\n" \
  "s_cbranch_scc1 SPIN44_%=\n" \
  "s_mov_b32 s76, 67436544\n" \
  "buffer_load_dword v224, off, s[36:39], s76 sc1\n" \
  "s_memrealtime s[54:55]\n" \
  "s_waitcnt vmcnt(0) lgkmcnt(0)\n" \
  "v_readfirstlane_b32 s76, v224\n" \
  "s_nop 1\n" \
  "s_cmp_eq_u32 s76, s82\n" \
  "s_cbranch_scc1 TMO45_%=\n" \
  "s_sub_u32 s54, s54, s52\n" \
  "s_cmp_lt_u32 s54, 30000000\n" \
  "s_cbranch_scc1 SPIN44_%=\n" \
  "TMO45_%=:\n" \
  "s_mov_b32 s89, 1\n" \
  "s_mov_b32 s76, 67436544\n" \
  "v_mov_b32 v224, s82\n" \
  "s_mov_b64 exec, 1\n" \
  "buffer_store_dword v224, off, s[36:39], s76 sc1\n" \
  "s_mov_b32 s76, 0\n" \
  "DN46_%=:\n" \
  "s_mov_b64 exec, -1\n" \
  "s_waitcnt lgkmcnt(0)\n" \
  "s_branch OK42_%=\n" \
  "END_%=:\n" \
  ""
#define FUSED_CLOB "v0","v1","v2","v3","v4","v5","v6","v7","v8","v9","v10","v11","v12","v13","v14","v15","v16","v17","v18","v19","v20","v21","v22","v23","v24","v25","v26","v27","v28","v29","v30","v31","v32","v33","v34","v35","v36","v37","v38","v39","v40","v41","v42","v43","v44","v45","v46","v47","v48","v49","v50","v51","v52","v53","v54","v55","v56","v57","v58","v59","v60","v61","v62","v63","v64","v65","v66","v67","v68","v69","v70","v71","v72","v73","v74","v75","v76","v77","v78","v79","v80","v81","v82","v83","v84","v85","v86","v87","v88","v89","v90","v91","v92","v93","v94","v95","v96","v97","v98","v99","v100","v101","v102","v103","v104","v105","v106","v107","v108","v109","v110","v111","v112","v113","v114","v115","v116","v117","v118","v119","v120","v121","v122","v123","v124","v125","v126","v127","v128","v129","v130","v131","v132","v133","v134","v135","v136","v137","v138","v139","v140","v141","v142","v143","v144","v145","v146","v147","v148","v149","v150","v151","v152","v153","v154","v155","v156","v157","v158","v159","v160","v161","v162","v163","v164","v165","v166","v167","v168","v169","v170","v171","v172","v173","v174","v175","v176","v177","v178","v179","v180","v181","v182","v183","v184","v185","v186","v187","v188","v189","v190","v191","v192","v193","v194","v195","v196","v197","v198","v199","v200","v201","v202","v203","v204","v205","v206","v207","v208","v209","v210","v211","v212","v213","v214","v215","v216","v217","v218","v219","v220","v221","v222","v223","v224","v225","v226","v227","v228","v229","v230","v231","v232","v233","v234","v235","v236","v237","v238","v239","v240","v241","v242","v243","v244","v245","v246","v247","v248","v249","v250","v251","v252","v253","a0","a1","a2","a3","a4","a5","a6","a7","a8","a9","a10","a11","a12","a13","a14","a15","a16","a17","a18","a19","a20","a21","a22","a23","a24","a25","a26","a27","a28","a29","a30","a31","a32","a33","a34","a35","a36","a37","a38","a39","a40","a41","a42","a43","a44","a45","a46","a47","a48","a49","a50","a51","a52","a53","a54","a55","a56","a57","a58","a59","a60","a61","a62","a63","a64","a65","a66","a67","a68","a69","a70","a71","a72","a73","a74","a75","a76","a77","a78","a79","a80","a81","a82","a83","a84","a85","a86","a87","a88","a89","a90","a91","a92","a93","a94","a95","a96","a97","a98","a99","a100","a101","a102","a103","a104","a105","a106","a107","a108","a109","a110","a111","a112","a113","a114","a115","a116","a117","a118","a119","a120","a121","a122","a123","a124","a125","a126","a127","a128","a129","a130","a131","a132","a133","a134","a135","a136","a137","a138","a139","a140","a141","a142","a143","a144","a145","a146","a147","a148","a149","a150","a151","a152","a153","a154","a155","a156","a157","a158","a159","a160","a161","a162","a163","a164","a165","a166","a167","a168","a169","a170","a171","a172","a173","a174","a175","a176","a177","a178","a179","a180","a181","a182","a183","a184","a185","a186","a187","a188","a189","a190","a191","a192","a193","a194","a195","a196","a197","a198","a199","a200","a201","a202","a203","a204","a205","a206","a207","a208","a209","a210","a211","a212","a213","a214","a215","a216","a217","a218","a219","a220","a221","a222","a223","a224","a225","a226","a227","a228","a229","a230","a231","a232","a233","a234","a235","a236","a237","a238","a239","a240","a241","a242","a243","a244","a245","a246","a247","a248","a249","a250","a251","a252","a253","a254","a255","s24","s25","s26","s27","s28","s29","s30","s31","s32","s33","s34","s35","s36","s37","s38","s39","s40","s41","s42","s43","s44","s45","s46","s47","s48","s49","s50","s51","s52","s53","s54","s55","s56","s57","s58","s59","s60","s61","s62","s63","s64","s65","s66","s67","s68","s69","s70","s71","s72","s73","s74","s75","s76","s77","s78","s79","s80","s81","s82","s83","s84","s85","s86","s87","s88","s89","s90","s91","s92","vcc","scc","memory"
#define NFULL 31
#define NHALF 2
#define NLOC 4
#define XMAP 1
#define NRSLOW 8
#define R0 0
#define KGRP 2

constexpr int NR = 8;
constexpr size_t WS_USED = (64u << 20) + 0x60000;

struct Params {
    const float* x;
    const float* w;
    float* out;
    void* const* ws_tbl;
    char* ws;
    unsigned* dbg;
    unsigned rank;
};

__global__ __launch_bounds__(256, 1) void fused_kernel(Params p) {
    __shared__ __attribute__((aligned(16))) char lds[49152 + 64];
    const unsigned pb = blockIdx.x;
    const unsigned b = ((pb >> 1) & 3) * 2 + ((pb >> 3) & 1) + (((pb >> 4) & 7) << 3) + ((pb >> 7) << 6) + ((pb & 1) << 7);
    const unsigned rank = p.rank;
    const unsigned kh = b >> 7;
    const unsigned wave = __builtin_amdgcn_readfirstlane(threadIdx.x >> 6);
    const unsigned t = threadIdx.x & 63;
    const unsigned ldsb =
        __builtin_amdgcn_readfirstlane((unsigned)(size_t)(__attribute__((address_space(3))) char*)lds);
    const unsigned slow_parity = ((3u - (unsigned)((size_t)p.w >> 7)) >> 1) & 1;
    const bool slow = ((b >> 3) & 1) == slow_parity;
    const unsigned nr = slow ? NRSLOW : 28 - NRSLOW;
    const unsigned fidx = (b & 7) | ((b >> 4) << 3);
    unsigned dest = rank, s = 0, mb = 0;
    if (t < nr) {
        const unsigned U = t < NRSLOW ? t * 256 + b : NRSLOW * 256 + (t - NRSLOW) * 128 + fidx;
        s = U / 448;
        const unsigned idx = U - s * 448;
        mb = idx / 7;
        dest = (rank + 1 + (idx - mb * 7)) & 7;
    } else if (t >= 62) {
        const unsigned L = b + (t - 62) * 256;
        s = L >> 6;
        mb = L & 63;
    }
    const unsigned kap = (((b >> 5) & 1) | ((b & 1) << 1)) & (KGRP - 1);
    unsigned tc = t;
    if (t >= 2 && t < NLOC + 4) tc = t ^ (kap & 1);
    else if (t >= NLOC && t < NLOC + ((NFULL - NLOC) & ~(KGRP - 1u))) tc = NLOC + ((t - NLOC) ^ kap);
    unsigned bsrc = rank, bs = 0;
    if (tc < NLOC) {
        bs = 2 * tc + kh;
    } else if (tc < NFULL + NHALF) {
        const unsigned i = tc < NFULL ? 2 * (tc - NLOC) + kh : (56 - NHALF) + (tc - NFULL);
        bs = i / 7;
        bsrc = (rank + 1 + (i - bs * 7)) & 7;
    }
    const unsigned done_rounds = t + R0;
    const unsigned units = done_rounds <= NRSLOW ? done_rounds * 256 : NRSLOW * 256 + (done_rounds - NRSLOW) * 128;
    const unsigned sfull = units / 448;
    const unsigned nrem = (sfull * 7 + 1 - kh) / 2;
    unsigned lmax = NLOC + (nrem < 4 ? nrem & ~((KGRP - 1u) & 1u) : nrem & ~(KGRP - 1u));
    if (lmax > NFULL || sfull >= 8) lmax = NFULL;
    const unsigned mt = (b >> 6) & 1;
    unsigned p2 = dest | (mb << 4) | ((rank * 8 + s) << 10);
    unsigned p3 = (bsrc * 8 + bs) | ((((bsrc * 8 + bs) * 64 + mt * 32 + 31) * 4) << 8) | (lmax << 24);
    unsigned* dbg = p.dbg + (pb * 4 + wave) * 8;
    asm volatile(FUSED_ASM
                 : "+v"(p2), "+v"(p3)
                 : "s"(p.x), "s"(p.w), "s"(p.out), "s"(p.ws_tbl), "s"(p.ws), "s"(b), "s"(wave), "s"(ldsb),
                   "s"(dbg), "s"(nr - 1 - R0)
                 : FUSED_CLOB);
}

extern "C" void kernel_launch(int rank, int nranks, ncclComm_t comm, hipStream_t stream, void* const* d_in,
                              const int* in_sizes, int n_in, void* d_out, int out_size, const int* in_dtypes,
                              int out_dtype, void* const* d_out_all, void* const* d_ws_all, size_t ws_bytes) {
    if (nranks != NR || (unsigned)rank >= (unsigned)NR || ws_bytes < WS_USED || n_in < 2 ||
        in_sizes[0] != 8192 * 1024 || in_sizes[1] != 8192 * 8192 || out_size != 1024 * 8192) {
        fprintf(stderr, "unexpected configuration\n");
        return;
    }
    Params p;
    p.x = (const float*)d_in[0];
    p.w = (const float*)d_in[1];
    p.out = (float*)d_out;
    p.ws_tbl = d_ws_all;
    p.ws = (char*)d_ws_all[rank];
    p.dbg = nullptr;
    p.rank = (unsigned)rank;
    fused_kernel<<<256, 256, 0, stream>>>(p);
}
```
